```python
import numpy as np

kernel_name = "gpu_gemm_ar_m4096_k4096_n16384_e5m2e5m2_dqrelu"


ABSMAX_RTOL = 0.05

COMPUTE_DTYPE = "float32"
EPILOGUE = "dequant_relu"
IN_QUANT = None
OUT_QUANT = None
IN_FP8 = "e5m2e5m2"
IN_DTYPES = {"x": "e5m2", "w_mat": "e5m2",
             "scale_x": "float32", "scale_w": "float32"}
WORLD_SIZE = 8

def _snap_fp8(x, mant_bits, max_abs, min_normal):
    a = np.abs(x.astype(np.float64))
    _m, _e = np.frexp(a)
    q = np.where(a >= min_normal, np.ldexp(1.0, _e - 1 - mant_bits),
                 min_normal * 2.0 ** -mant_bits)
    snapped = np.minimum(np.round(a / q) * q, max_abs)
    return (np.sign(x) * snapped).astype(np.float32)

def setup_inputs():

    x = _snap_fp8(np.random.randn(4096, 4096).astype(np.float32), 2, 57344.0, 2.0 ** -14)
    w_mat = _snap_fp8(np.random.randn(4096, 16384).astype(np.float32), 2, 57344.0, 2.0 ** -14)
    scale_x = (0.02 * (1.0 + np.random.rand(1))).astype(np.float32)
    scale_w = (0.02 * (1.0 + np.random.rand(1))).astype(np.float32)
    return {"x": x, "w_mat": w_mat, "scale_x": scale_x, "scale_w": scale_w}

def reference(x, w_mat, scale_x, scale_w):
    acc = (x @ w_mat).astype(np.float32)
    y = (acc * (scale_x * scale_w)).astype(np.float32)
    return np.maximum(y, 0.0).astype(np.float32)
```

```cpp
#include <hip/hip_runtime.h>
#include <rccl/rccl.h>
#include <cstdio>

#define HIP_CHECK(expr) do { hipError_t _e = (expr); if (_e != hipSuccess) { \
    fprintf(stderr, "HIP error %s at %s:%d\n", hipGetErrorString(_e), __FILE__, __LINE__); \
    return; } } while (0)

#define LAS __attribute__((address_space(3)))
#define GAS __attribute__((address_space(1)))
typedef int i32x4 __attribute__((ext_vector_type(4)));
typedef unsigned u32x4 __attribute__((ext_vector_type(4)));
typedef int i32x8 __attribute__((ext_vector_type(8)));
typedef float f32x4 __attribute__((ext_vector_type(4)));

namespace {
constexpr int NR = 8;
constexpr int M = 4096, N = 16384, KS = 512;
constexpr unsigned BLK = 32768, HB = 16384;
constexpr unsigned NPANEL = 80;
constexpr unsigned SLAB = NPANEL * BLK;
constexpr unsigned DATA_BYTES = 32 * SLAB;
constexpr unsigned WS_FLAGS = 0;
constexpr unsigned FLAG_BYTES = NPANEL * 128;
constexpr unsigned WS_CTR = 16384;
constexpr unsigned WS_TMO = WS_CTR + 256 * 128;
constexpr size_t WS_DATA = 1u << 20;
constexpr size_t WS_NEED = WS_DATA + 2 * (size_t)DATA_BYTES;
constexpr int NCOMM = 16, NWG = 256;
constexpr unsigned long long DEADLINE_TICKS = 50000000ull;
constexpr int AUX_SYS = 17;
constexpr int AUX_NT = 2;
constexpr unsigned LDS_FLAGS = 8 * HB;
constexpr unsigned LDS_MISC = LDS_FLAGS + 4 * 5 * 256;
constexpr unsigned LDS_BYTES = LDS_MISC + 64;

struct Args {
    const unsigned char* x; const unsigned char* w; const float* sx; const float* sw; float* out;
    char* ws[NR];
    int rank;
};

__device__ __forceinline__ void block_of(int o, int& pnl, int& kt) {
    if (o < 64) { kt = o >> 4; const int i = o & 15; pnl = (i & 1) ? 16 + (i >> 1) : (i >> 1); }
    else if (o < 140) { const int r = o - 64; kt = r / 19; const int i = r - kt * 19; pnl = i < 8 ? 8 + i : 16 + i; }
    else { const int r = o - 140, g = r / 60, r2 = r - g * 60; kt = r2 / 15; pnl = 16 + 19 + g * 15 + (r2 - kt * 15); }
}
__device__ __forceinline__ void tile_of(int g, int i, int& mp, int& np) {
    const int xcd = g & 7, s = g >> 3;
    int grp = i + (s < 8 ? 0 : 1);
    if (grp == 0) { mp = (xcd & 1) * 4 + (s & 3); np = (xcd >> 1) * 2 + (s >> 2); }
    else if (grp == 1) {
        const int l = xcd * 30 + s;
        if (l < 88) { const int mg = l >= 44, r = l - mg * 44; mp = mg * 4 + (r & 3); np = 8 + (r >> 2); }
        else { const int l2 = l - 88, mg = l2 >= 76, r = l2 - mg * 76; mp = 8 + mg * 4 + (r & 3); np = r >> 2; }
    } else { const int r = (xcd >> 2) * 30 + s; mp = (xcd & 3) * 4 + (r & 3); np = 19 + (grp - 2) * 15 + (r >> 2); }
}

#define WAIT_V(n) asm volatile("s_waitcnt vmcnt(" #n ")" ::: "memory")
#define WAIT_L(n) asm volatile("s_waitcnt lgkmcnt(" #n ")" ::: "memory")
#define BAR __builtin_amdgcn_s_barrier()
#define SCHED __builtin_amdgcn_sched_barrier(0)

#define ACLOB "a0","a1","a2","a3","a4","a5","a6","a7","a8","a9","a10","a11","a12","a13","a14","a15","a16","a17","a18","a19","a20","a21","a22","a23","a24","a25","a26","a27","a28","a29","a30","a31","a32","a33","a34","a35","a36","a37","a38","a39","a40","a41","a42","a43","a44","a45","a46","a47","a48","a49","a50","a51","a52","a53","a54","a55","a56","a57","a58","a59","a60","a61","a62","a63","a64","a65","a66","a67","a68","a69","a70","a71","a72","a73","a74","a75","a76","a77","a78","a79","a80","a81","a82","a83","a84","a85","a86","a87","a88","a89","a90","a91","a92","a93","a94","a95","a96","a97","a98","a99","a100","a101","a102","a103","a104","a105","a106","a107","a108","a109","a110","a111","a112","a113","a114","a115","a116","a117","a118","a119","a120","a121","a122","a123","a124","a125","a126","a127","a128","a129","a130","a131","a132","a133","a134","a135","a136","a137","a138","a139","a140","a141","a142","a143","a144","a145","a146","a147","a148","a149","a150","a151","a152","a153","a154","a155","a156","a157","a158","a159","a160","a161","a162","a163","a164","a165","a166","a167","a168","a169","a170","a171","a172","a173","a174","a175","a176","a177","a178","a179","a180","a181","a182","a183","a184","a185","a186","a187","a188","a189","a190","a191","a192","a193","a194","a195","a196","a197","a198","a199","a200","a201","a202","a203","a204","a205","a206","a207","a208","a209","a210","a211","a212","a213","a214","a215","a216","a217","a218","a219","a220","a221","a222","a223","a224","a225","a226","a227","a228","a229","a230","a231","a232","a233","a234","a235","a236","a237","a238","a239","a240","a241","a242","a243","a244","a245","a246","a247","a248","a249","a250","a251","a252","a253","a254","a255"
constexpr int acc_idx(int ai, int bj, int m, int n) { return (((ai * 2 + bj) * 4 + m) * 4 + n) * 4; }
template <int IDX, bool ZERO> __device__ __forceinline__ void mfma(const i32x8& x, const i32x8& y) {
    if constexpr (ZERO) asm volatile("v_mfma_f32_16x16x128_f8f6f4 a[%c2:%c3], %0, %1, 0 cbsz:1 blgp:1" :: "v"(x), "v"(y), "i"(IDX), "i"(IDX + 3) : ACLOB);
    else asm volatile("v_mfma_f32_16x16x128_f8f6f4 a[%c2:%c3], %0, %1, a[%c2:%c3] cbsz:1 blgp:1" :: "v"(x), "v"(y), "i"(IDX), "i"(IDX + 3) : ACLOB);
}
template <int AI, int E, int NOP> __device__ __forceinline__ void epi(float sc, __amdgpu_buffer_rsrc_t ro, unsigned ovoff, unsigned obase) {
    constexpr int m = E >> 3, j = (E >> 1) & 3, g = E & 1;
    float v0, v1, v2, v3;
    asm volatile("s_nop %c9\n\tv_accvgpr_read_b32 %0, a%c5\n\tv_accvgpr_read_b32 %1, a%c6\n\tv_accvgpr_read_b32 %2, a%c7\n\tv_accvgpr_read_b32 %3, a%c8\n\t"
                 "v_mul_f32 %0, %4, %0\n\tv_mul_f32 %1, %4, %1\n\tv_mul_f32 %2, %4, %2\n\tv_mul_f32 %3, %4, %3\n\t"
                 "v_maximum3_f32 %0, %0, 0, 0\n\tv_maximum3_f32 %1, %1, 0, 0\n\tv_maximum3_f32 %2, %2, 0, 0\n\tv_maximum3_f32 %3, %3, 0, 0"
                 : "=&v"(v0), "=&v"(v1), "=&v"(v2), "=&v"(v3)
                 : "s"(sc), "i"(acc_idx(AI, 0, m, 2 * g) + j), "i"(acc_idx(AI, 0, m, 2 * g + 1) + j), "i"(acc_idx(AI, 1, m, 2 * g) + j), "i"(acc_idx(AI, 1, m, 2 * g + 1) + j), "i"(NOP));
    const f32x4 v = {v0, v1, v2, v3};
    __builtin_amdgcn_raw_buffer_store_b128(__builtin_bit_cast(u32x4, v), ro, ovoff + g * 256, obase + (unsigned)(AI * 128 + m * 16 + j) * 65536u, AUX_NT);
}

__global__ __launch_bounds__(256, 1) void fused_gemm(Args a) {
    __shared__ __attribute__((aligned(16))) char shm[LDS_BYTES];
    LAS char* lds = (LAS char*)shm;
    const int tid = threadIdx.x, wid = __builtin_amdgcn_readfirstlane(tid >> 6), lane = tid & 63;
    const int bid = blockIdx.x;
    const unsigned rank = (unsigned)a.rank & (NR - 1);
    char* const myws = a.ws[rank];
    if (tid == 0) {
        GAS unsigned* ctr = (GAS unsigned*)(myws + WS_CTR + bid * 128);
        const unsigned c = __hip_atomic_load(ctr, __ATOMIC_RELAXED, __HIP_MEMORY_SCOPE_AGENT) + 1u;
        __hip_atomic_store(ctr, c, __ATOMIC_RELAXED, __HIP_MEMORY_SCOPE_AGENT);
        *(LAS unsigned*)(lds + LDS_MISC) = c;
        *(LAS unsigned*)(lds + LDS_MISC + 4) = 0u;
    }
#pragma unroll
    for (int i = 0; i < 5; ++i) *(LAS unsigned*)(lds + LDS_FLAGS + (wid * 5 + i) * 256 + lane * 4) = 0u;
    __syncthreads();
    const unsigned call = __builtin_amdgcn_readfirstlane(*(LAS unsigned*)(lds + LDS_MISC));
    const size_t doff = WS_DATA + (size_t)(call & 1u) * DATA_BYTES;

    if (bid < NCOMM) {
        __amdgpu_buffer_rsrc_t rd[NR];
#pragma unroll
        for (int i = 0; i < NR; ++i) {
            const unsigned p = (rank + 1u + (unsigned)(i + 2 * wid + bid)) & (NR - 1);
            rd[i] = __builtin_amdgcn_make_buffer_rsrc(a.ws[p] + doff, 0, DATA_BYTES, 0x00020000);
        }
        const __amdgpu_buffer_rsrc_t rx = __builtin_amdgcn_make_buffer_rsrc((void*)a.x, 0, (unsigned)M * KS, 0x00020000);
        const __amdgpu_buffer_rsrc_t rw = __builtin_amdgcn_make_buffer_rsrc((void*)a.w, 0, (unsigned)KS * N, 0x00020000);
        const int r = lane & 15, g = lane >> 4;
        const unsigned xv = (unsigned)(wid * 64 + r) * KS + g * 16;
        const unsigned wv = (unsigned)(g * 16) * N + wid * 64 + 4 * r;
        GAS unsigned* fl = (GAS unsigned*)(a.ws[lane & (NR - 1)] + WS_FLAGS);
        unsigned raw[16];
#define COMM_LOAD(hs) do { int _p, _k; block_of(((hs) >> 1) * NCOMM + bid, _p, _k); const int _h = (hs) & 1; \
            if (_p < 16) { const unsigned so = (unsigned)(_p * 256) * KS + _k * 128 + _h * 64; \
                _Pragma("unroll") for (int i = 0; i < 4; ++i) { const u32x4 v = __builtin_amdgcn_raw_buffer_load_b128(rx, xv + i * 16 * KS, so, 0); \
                    raw[4 * i] = v[0]; raw[4 * i + 1] = v[1]; raw[4 * i + 2] = v[2]; raw[4 * i + 3] = v[3]; } } \
            else { const unsigned so = (unsigned)(_k * 128 + _h * 64) * N + (_p - 16) * 256; \
                _Pragma("unroll") for (int i = 0; i < 16; ++i) raw[i] = __builtin_amdgcn_raw_buffer_load_b32(rw, wv, so + i * N, 0); } } while (0)
        COMM_LOAD(0);
        int ppnl = 0, pkt = 0;
        for (int hs = 0; hs < 40; ++hs) {
            int pnl, kt; block_of((hs >> 1) * NCOMM + bid, pnl, kt);
            const int h = hs & 1;
            u32x4 o[4]; unsigned ov[4];
            if (pnl < 16) {
#pragma unroll
                for (int i = 0; i < 4; ++i) { o[i] = (u32x4){raw[4 * i], raw[4 * i + 1], raw[4 * i + 2], raw[4 * i + 3]}; ov[i] = (wid * 4 + i) * 2048 + lane * 16; }
            } else {
#pragma unroll
                for (int b = 0; b < 4; ++b) {
                    const unsigned r0 = raw[4 * b], r1 = raw[4 * b + 1], r2 = raw[4 * b + 2], r3 = raw[4 * b + 3];
                    const unsigned t0 = __builtin_amdgcn_perm(r1, r0, 0x05010400), t1 = __builtin_amdgcn_perm(r1, r0, 0x07030602);
                    const unsigned t2 = __builtin_amdgcn_perm(r3, r2, 0x05010400), t3 = __builtin_amdgcn_perm(r3, r2, 0x07030602);
                    o[0][b] = __builtin_amdgcn_perm(t2, t0, 0x05040100); o[1][b] = __builtin_amdgcn_perm(t2, t0, 0x07060302);
                    o[2][b] = __builtin_amdgcn_perm(t3, t1, 0x05040100); o[3][b] = __builtin_amdgcn_perm(t3, t1, 0x07060302);
                }
#pragma unroll
                for (int c = 0; c < 4; ++c) ov[c] = (c >> 1) * HB + wid * 4096 + (c & 1) * 2048 + lane * 16;
            }
            const unsigned so = (unsigned)(kt * NR + rank) * SLAB + pnl * BLK + h * 1024;
#pragma unroll
            for (int d = 0; d < NR; ++d) __builtin_amdgcn_raw_buffer_store_b128(o[0], rd[d], ov[0], so, AUX_SYS);
            if (h == 0 && hs > 0) {
                WAIT_V(8); BAR;
                if (tid < NR) __hip_atomic_store(fl + ppnl * 32 + pkt * NR + rank, call, __ATOMIC_RELAXED, __HIP_MEMORY_SCOPE_SYSTEM);
            }
            if (hs + 1 < 40) COMM_LOAD(hs + 1);
#pragma unroll
            for (int c = 1; c < 4; ++c)
#pragma unroll
                for (int d = 0; d < NR; ++d) __builtin_amdgcn_raw_buffer_store_b128(o[c], rd[d], ov[c], so, AUX_SYS);
            ppnl = pnl; pkt = kt;
        }
        WAIT_V(0); BAR;
        if (tid < NR) __hip_atomic_store(fl + ppnl * 32 + pkt * NR + rank, call, __ATOMIC_RELAXED, __HIP_MEMORY_SCOPE_SYSTEM);
        return;
    }

    const int gw = bid - NCOMM, ntiles = (gw >> 3) < 8 ? 5 : 4;
    const int wr = wid >> 1, wc = wid & 1, fr = lane & 15, fq = lane >> 4;
    const __amdgpu_buffer_rsrc_t rin = __builtin_amdgcn_make_buffer_rsrc(myws + doff, 0, DATA_BYTES, 0x00020000);
    const __amdgpu_buffer_rsrc_t rfl = __builtin_amdgcn_make_buffer_rsrc(myws + WS_FLAGS, 0, FLAG_BYTES, 0x00020000);
    GAS unsigned* tmo = (GAS unsigned*)(myws + WS_TMO);
    const float scale = __builtin_bit_cast(float, __builtin_amdgcn_readfirstlane(__builtin_bit_cast(int, a.sx[0] * a.sw[0])));
    const unsigned voff = tid * 16, ldsw = wid * 1024;
    int aoff = wr * 8192 + lane * 16, boff = 4 * HB + wc * 8192 + lane * 16;
    asm volatile("" : "+v"(aoff), "+v"(boff));
    const unsigned ovoff = (unsigned)(wr * 64 + 4 * fq) * 65536u + wc * 512 + fr * 16;
    unsigned long long tl = 0;
#pragma unroll
    for (int i = 0; i < 5; ++i) { int m_, n_; tile_of(gw, i < ntiles ? i : ntiles - 1, m_, n_); tl |= (unsigned long long)((n_ << 4) | m_) << (12 * i); }
#define TILE(i) ((unsigned)(tl >> (12 * ((i) < ntiles ? (i) : ntiles - 1))) & 0xfffu)
#define SA(b, h) (((b) * 2 + (h)) * HB)
#define SB(b, h) ((4 + (b) * 2 + (h)) * HB)
#define G1(bufoff, soff, i) __builtin_amdgcn_raw_ptr_buffer_load_lds(rin, (LAS void*)(lds + (bufoff) + ldsw + (i) * 4096), 16, voff, (soff) + (i) * 4096, 0, AUX_SYS)
#define STAGE4(bufoff, soff) do { G1(bufoff, soff, 0); G1(bufoff, soff, 1); G1(bufoff, soff, 2); G1(bufoff, soff, 3); } while (0)
#define RD(p) (*(const LAS i32x4*)(p))
#define CAT(lo, hi) __builtin_shufflevector(lo, hi, 0, 1, 2, 3, 4, 5, 6, 7)
#define MF(Z, ai, bj, m, n, RA, RB) mfma<acc_idx(ai, bj, m, n), Z>(CAT(RA[2 * (m)], RA[2 * (m) + 1]), CAT(RB[2 * (n)], RB[2 * (n) + 1]))
#define EP(eai, e) do { if constexpr ((eai) >= 0) epi<(eai) < 0 ? 0 : (eai), (e), 0>(esc, erout, ovoff, eobase); } while (0)
#define QUAD(Z, ai, bj, RA, RB, RN, rdoff, stoff, soff, vm, eai, e0) do { \
        if constexpr ((vm) >= 0) { asm volatile("s_waitcnt vmcnt(%0) lgkmcnt(0)" :: "n"((vm) < 0 ? 0 : (vm)) : "memory"); BAR; } SCHED; \
        MF(Z, ai, bj, 0, 0, RA, RB); RN[0] = RD(lds + (rdoff)); EP(eai, e0 + 0); SCHED; \
        MF(Z, ai, bj, 0, 1, RA, RB); RN[1] = RD(lds + (rdoff) + 1024); EP(eai, e0 + 1); SCHED; \
        MF(Z, ai, bj, 0, 2, RA, RB); RN[2] = RD(lds + (rdoff) + 2048); EP(eai, e0 + 2); SCHED; \
        MF(Z, ai, bj, 0, 3, RA, RB); RN[3] = RD(lds + (rdoff) + 3072); EP(eai, e0 + 3); SCHED; \
        MF(Z, ai, bj, 1, 0, RA, RB); RN[4] = RD(lds + (rdoff) + 4096); EP(eai, e0 + 4); SCHED; \
        MF(Z, ai, bj, 1, 1, RA, RB); RN[5] = RD(lds + (rdoff) + 5120); EP(eai, e0 + 5); SCHED; \
        MF(Z, ai, bj, 1, 2, RA, RB); RN[6] = RD(lds + (rdoff) + 6144); EP(eai, e0 + 6); SCHED; \
        MF(Z, ai, bj, 1, 3, RA, RB); RN[7] = RD(lds + (rdoff) + 7168); EP(eai, e0 + 7); SCHED; \
        MF(Z, ai, bj, 2, 0, RA, RB); G1(stoff, soff, 0); EP(eai, e0 + 8); SCHED; \
        MF(Z, ai, bj, 2, 1, RA, RB); EP(eai, e0 + 9); SCHED; \
        MF(Z, ai, bj, 2, 2, RA, RB); G1(stoff, soff, 1); EP(eai, e0 + 10); SCHED; \
        MF(Z, ai, bj, 2, 3, RA, RB); EP(eai, e0 + 11); SCHED; \
        MF(Z, ai, bj, 3, 0, RA, RB); G1(stoff, soff, 2); EP(eai, e0 + 12); SCHED; \
        MF(Z, ai, bj, 3, 1, RA, RB); EP(eai, e0 + 13); SCHED; \
        MF(Z, ai, bj, 3, 2, RA, RB); G1(stoff, soff, 3); EP(eai, e0 + 14); SCHED; \
        MF(Z, ai, bj, 3, 3, RA, RB); EP(eai, e0 + 15); SCHED; } while (0)

    int si = 0, ready = 0;
    unsigned sA = (TILE(0) & 15u) * BLK, sB = (16u + (TILE(0) >> 4)) * BLK;
#define FLAG_VOFF(tile) ((lane < 32 ? ((tile) & 15u) : 16u + ((tile) >> 4)) * 128u + (lane & 31) * 4u)
    unsigned fvoff = FLAG_VOFF(TILE(0));
#define FLAG_SLOT(i) (lds + LDS_FLAGS + (wid * 5 + (i)) * 256)
#define FLAG_FETCH(i, vo) __builtin_amdgcn_raw_ptr_buffer_load_lds(rfl, (LAS void*)FLAG_SLOT(i), 4, vo, 0, 0, AUX_SYS)
#define CHECK(ss) do { if ((ss) >= ready) { \
            const unsigned long long _t0 = __builtin_amdgcn_s_memrealtime(); \
            for (;;) { \
                const unsigned _v = *(volatile LAS unsigned*)(FLAG_SLOT(si) + lane * 4); \
                const unsigned long long _m = __builtin_amdgcn_ballot_w64(_v >= call); \
                const unsigned _both = ~((unsigned)_m & (unsigned)(_m >> 32)); \
                ready = _both ? __builtin_ctz(_both) : 32; \
                if ((ss) < ready) break; \
                if (__builtin_amdgcn_readfirstlane(__hip_atomic_load(tmo, __ATOMIC_RELAXED, __HIP_MEMORY_SCOPE_AGENT)) == call \
                    || __builtin_amdgcn_s_memrealtime() - _t0 > DEADLINE_TICKS) { \
                    __hip_atomic_store(tmo, call, __ATOMIC_RELAXED, __HIP_MEMORY_SCOPE_AGENT); \
                    *(volatile LAS unsigned*)(lds + LDS_MISC + 4) = 1u; ready = 32; break; } \
                __builtin_amdgcn_s_sleep(4); \
                FLAG_FETCH(si, fvoff); WAIT_V(0); \
            } \
            if (ready < 32) FLAG_FETCH(si, fvoff); \
        } } while (0)
#define ITER(KIND, ss) do { \
        CHECK(ss); \
        { const unsigned a2 = (unsigned)(ss) * SLAB + sA, b2 = (unsigned)(ss) * SLAB + sB; \
          QUAD(KIND == 0, 0, 0, RA0, RB0, RB1, SA(0, 1) + boff, SA(0, 0), a2, (KIND == 0 ? 48 : 16), (KIND == 0 ? 1 : -1), 0); \
          QUAD(KIND == 0, 0, 1, RA0, RB1, RA1, SA(0, 1) + aoff, SB(0, 0), b2, -1, (KIND == 0 ? 1 : -1), 16); \
          QUAD(KIND == 0, 1, 1, RA1, RB1, RA0, SA(1, 0) + aoff, SB(0, 1), b2 + HB, (KIND == 0 ? 63 : 16), -1, 0); \
          QUAD(KIND == 0, 1, 0, RA1, RB0, RB1, SA(1, 1) + boff, SA(0, 1), a2 + HB, -1, -1, 0); } \
        CHECK((ss) + 1); \
        { const unsigned a3 = (unsigned)((ss) + 1) * SLAB + sA, b3 = (unsigned)((ss) + 1) * SLAB + sB; \
          QUAD(false, 0, 1, RA0, RB1, RB0, SA(1, 0) + boff, SA(1, 0), a3, (KIND == 0 ? 48 : 16), -1, 0); \
          QUAD(false, 0, 0, RA0, RB0, RA1, SA(1, 1) + aoff, SB(1, 1), b3 + HB, -1, -1, 0); \
          if constexpr (KIND == 2) { esc = *(volatile LAS unsigned*)(lds + LDS_MISC + 4) != 0u ? __builtin_nanf("") : scale; \
              esc = __builtin_bit_cast(float, __builtin_amdgcn_readfirstlane(__builtin_bit_cast(int, esc))); } \
          QUAD(false, 1, 0, RA1, RB0, RA0, SA(0, 0) + aoff, SB(1, 0), b3, 16, (KIND == 2 ? 0 : -1), 0); \
          QUAD(false, 1, 1, RA1, RB1, RB0, SA(0, 0) + boff, SA(1, 1), a3 + HB, -1, (KIND == 2 ? 0 : -1), 16); } } while (0)

    i32x4 RA0[8], RA1[8], RB0[8], RB1[8];
    CHECK(0);
    STAGE4(SA(0, 0), sA); STAGE4(SB(0, 0), sB); STAGE4(SB(0, 1), sB + HB); STAGE4(SA(0, 1), sA + HB);
    CHECK(1);
    STAGE4(SA(1, 0), sA + SLAB); STAGE4(SB(1, 1), sB + SLAB + HB); STAGE4(SB(1, 0), sB + SLAB); STAGE4(SA(1, 1), sA + SLAB + HB);
    WAIT_V(0); BAR;
#pragma unroll
    for (int i = 0; i < 8; ++i) { RA0[i] = RD(lds + SA(0, 0) + aoff + i * 1024); RB0[i] = RD(lds + SA(0, 0) + boff + i * 1024); }
    __amdgpu_buffer_rsrc_t erout = __builtin_amdgcn_make_buffer_rsrc((void*)a.out, 0, 0, 0x00020000);
    unsigned eobase = 0; float esc = 0.f;
    for (int ui = 0; ui < ntiles; ++ui) {
        ITER(0, 2);
        for (int ss = 4; ss < 32; ss += 2) {
            if (ss == 24 && ui + 1 < ntiles) FLAG_FETCH(ui + 1, FLAG_VOFF(TILE(ui + 1)));
            ITER(1, ss);
        }
        erout = __builtin_amdgcn_make_buffer_rsrc((void*)a.out, 0, (unsigned)M * N * 4u, 0x00020000);
        eobase = (TILE(ui) & 15u) * (256u * 65536u) + (TILE(ui) >> 4) * 1024u;
        if (ui + 1 < ntiles) { si = ui + 1; ready = 0; sA = (TILE(si) & 15u) * BLK; sB = (16u + (TILE(si) >> 4)) * BLK; fvoff = FLAG_VOFF(TILE(si)); }
        ITER(2, 0);
    }
    asm volatile("s_nop 15" ::: "memory");
#define EP1(e) epi<1, (e), 1>(esc, erout, ovoff, eobase)
#define EP4(e) EP1(e); EP1(e + 1); EP1(e + 2); EP1(e + 3)
    EP4(0); EP4(4); EP4(8); EP4(12); EP4(16); EP4(20); EP4(24); EP4(28);
    WAIT_V(0);
}
}

extern "C" void kernel_launch(int rank, int nranks, ncclComm_t comm,
                              hipStream_t stream,
                              void* const* d_in, const int* in_sizes, int n_in,
                              void* d_out, int out_size,
                              const int* in_dtypes, int out_dtype,
                              void* const* d_out_all,
                              void* const* d_ws_all, size_t ws_bytes) {
    (void)comm; (void)in_dtypes; (void)out_dtype; (void)d_out_all;
    if (nranks != NR || (unsigned)rank >= (unsigned)NR || n_in != 4 || in_sizes[0] != M * KS || in_sizes[1] != KS * N ||
        out_size != M * N || ws_bytes < WS_NEED) {
        fprintf(stderr, "kernel_launch: unexpected shapes\n");
        return;
    }
    Args a;
    a.x = (const unsigned char*)d_in[0]; a.w = (const unsigned char*)d_in[1];
    a.sx = (const float*)d_in[2]; a.sw = (const float*)d_in[3];
    a.out = (float*)d_out;
    for (int p = 0; p < NR; ++p) a.ws[p] = (char*)d_ws_all[p];
    a.rank = rank;
    hipLaunchKernelGGL(fused_gemm, dim3(NWG), dim3(256), 0, stream, a);
    HIP_CHECK(hipGetLastError());
}
```

```cpp
#include <hip/hip_runtime.h>
#include <rccl/rccl.h>
#include <cstdio>

#define HIP_CHECK(expr) do { hipError_t _e = (expr); if (_e != hipSuccess) { \
    fprintf(stderr, "HIP error %s at %s:%d\n", hipGetErrorString(_e), __FILE__, __LINE__); \
    return; } } while (0)

#define LAS __attribute__((address_space(3)))
#define GAS __attribute__((address_space(1)))
typedef int i32x4 __attribute__((ext_vector_type(4)));
typedef unsigned u32x4 __attribute__((ext_vector_type(4)));
typedef int i32x8 __attribute__((ext_vector_type(8)));
typedef float f32x4 __attribute__((ext_vector_type(4)));

namespace {
constexpr int NR = 8;
constexpr int M = 4096, N = 16384, KS = 512;
constexpr unsigned BLK = 32768, HB = 16384;
constexpr unsigned NPANEL = 80;
constexpr unsigned SLAB = NPANEL * BLK;
constexpr unsigned DATA_BYTES = 32 * SLAB;
constexpr unsigned WS_FLAGS = 0;
constexpr unsigned FLAG_BYTES = NPANEL * 512;
constexpr unsigned WS_CTR = 65536;
constexpr unsigned WS_TMO = WS_CTR + 256 * 128;
constexpr unsigned WS_SLOT = WS_TMO + 128;
constexpr unsigned WS_TICKET = WS_SLOT + 8 * 128;
constexpr unsigned NUNIT = 1280;
constexpr size_t WS_DATA = 1u << 20;
constexpr size_t WS_NEED = WS_DATA + 2 * (size_t)DATA_BYTES;
constexpr int NCOMM = 16, NWG = 256;
constexpr unsigned long long DEADLINE_TICKS = 50000000ull;
constexpr int AUX_SYS = 17;
#define POLL_SLEEP 8
#define SLOT_D 0
#define SLOT_LEAD 290
#define OAUX 2
constexpr int AUX_NT = OAUX;
constexpr unsigned LDS_FLAGS = 8 * HB;
constexpr unsigned LDS_MISC = LDS_FLAGS + 3 * 1024;
constexpr unsigned LDS_BYTES = LDS_MISC + 64;

struct Args {
    const unsigned char* x; const unsigned char* w; const float* sx; const float* sw; float* out;
    char* myws; char* peer[NR - 1];
    int rank;
    unsigned order[320];
    unsigned long long tiles[NWG - NCOMM];
};

#define WAIT_V(n) asm volatile("s_waitcnt vmcnt(" #n ")" ::: "memory")
#define WAIT_L(n) asm volatile("s_waitcnt lgkmcnt(" #n ")" ::: "memory")
#define BAR __builtin_amdgcn_s_barrier()
#define SCHED __builtin_amdgcn_sched_barrier(0)

#define ACLOB "a0","a1","a2","a3","a4","a5","a6","a7","a8","a9","a10","a11","a12","a13","a14","a15","a16","a17","a18","a19","a20","a21","a22","a23","a24","a25","a26","a27","a28","a29","a30","a31","a32","a33","a34","a35","a36","a37","a38","a39","a40","a41","a42","a43","a44","a45","a46","a47","a48","a49","a50","a51","a52","a53","a54","a55","a56","a57","a58","a59","a60","a61","a62","a63","a64","a65","a66","a67","a68","a69","a70","a71","a72","a73","a74","a75","a76","a77","a78","a79","a80","a81","a82","a83","a84","a85","a86","a87","a88","a89","a90","a91","a92","a93","a94","a95","a96","a97","a98","a99","a100","a101","a102","a103","a104","a105","a106","a107","a108","a109","a110","a111","a112","a113","a114","a115","a116","a117","a118","a119","a120","a121","a122","a123","a124","a125","a126","a127","a128","a129","a130","a131","a132","a133","a134","a135","a136","a137","a138","a139","a140","a141","a142","a143","a144","a145","a146","a147","a148","a149","a150","a151","a152","a153","a154","a155","a156","a157","a158","a159","a160","a161","a162","a163","a164","a165","a166","a167","a168","a169","a170","a171","a172","a173","a174","a175","a176","a177","a178","a179","a180","a181","a182","a183","a184","a185","a186","a187","a188","a189","a190","a191","a192","a193","a194","a195","a196","a197","a198","a199","a200","a201","a202","a203","a204","a205","a206","a207","a208","a209","a210","a211","a212","a213","a214","a215","a216","a217","a218","a219","a220","a221","a222","a223","a224","a225","a226","a227","a228","a229","a230","a231","a232","a233","a234","a235","a236","a237","a238","a239","a240","a241","a242","a243","a244","a245","a246","a247","a248","a249","a250","a251","a252","a253","a254","a255"
constexpr int acc_idx(int ai, int bj, int m, int n) { return (((ai * 2 + bj) * 4 + m) * 4 + n) * 4; }
template <int IDX, bool ZERO> __device__ __forceinline__ void mfma(const i32x8& x, const i32x8& y) {
    if constexpr (ZERO) asm volatile("v_mfma_f32_16x16x128_f8f6f4 a[%c2:%c3], %0, %1, 0 cbsz:1 blgp:1" :: "v"(x), "v"(y), "i"(IDX), "i"(IDX + 3) : ACLOB);
    else asm volatile("v_mfma_f32_16x16x128_f8f6f4 a[%c2:%c3], %0, %1, a[%c2:%c3] cbsz:1 blgp:1" :: "v"(x), "v"(y), "i"(IDX), "i"(IDX + 3) : ACLOB);
}
template <int IDX, bool ZERO> __device__ __forceinline__ void mfma32(const i32x8& x, const i32x8& y) {
    if constexpr (ZERO) asm volatile("v_mfma_f32_32x32x64_f8f6f4 a[%c2:%c3], %0, %1, 0 cbsz:1 blgp:1" :: "v"(x), "v"(y), "i"(IDX), "i"(IDX + 15) : ACLOB);
    else asm volatile("v_mfma_f32_32x32x64_f8f6f4 a[%c2:%c3], %0, %1, a[%c2:%c3] cbsz:1 blgp:1" :: "v"(x), "v"(y), "i"(IDX), "i"(IDX + 15) : ACLOB);
}
struct Ev { float v0, v1, v2, v3; };
template <int AI, int E> __device__ __forceinline__ void epi_a(Ev& e, float sc) {
    constexpr int m = E >> 3, j = (E >> 1) & 3, g = E & 1;
    asm volatile("v_accvgpr_read_b32 %0, a%c5\n\tv_accvgpr_read_b32 %1, a%c6\n\tv_accvgpr_read_b32 %2, a%c7\n\tv_accvgpr_read_b32 %3, a%c8\n\t"
                 "v_mul_f32 %0, %4, %0\n\tv_mul_f32 %1, %4, %1\n\tv_mul_f32 %2, %4, %2"
                 : "=&v"(e.v0), "=&v"(e.v1), "=&v"(e.v2), "=&v"(e.v3)
                 : "s"(sc), "i"(acc_idx(AI, 0, m, 2 * g) + j), "i"(acc_idx(AI, 0, m, 2 * g + 1) + j), "i"(acc_idx(AI, 1, m, 2 * g) + j), "i"(acc_idx(AI, 1, m, 2 * g + 1) + j));
}
template <int AI, int E> __device__ __forceinline__ void epi_b(Ev& e, float sc, __amdgpu_buffer_rsrc_t ro, unsigned ovoff, unsigned obase) {
    constexpr int m = E >> 3, j = (E >> 1) & 3, g = E & 1;
    asm volatile("v_mul_f32 %3, %4, %3\n\tv_maximum3_f32 %0, %0, 0, 0\n\tv_maximum3_f32 %1, %1, 0, 0\n\tv_maximum3_f32 %2, %2, 0, 0\n\tv_maximum3_f32 %3, %3, 0, 0"
                 : "+v"(e.v0), "+v"(e.v1), "+v"(e.v2), "+v"(e.v3) : "s"(sc));
    const f32x4 v = {e.v0, e.v1, e.v2, e.v3};
    __builtin_amdgcn_raw_buffer_store_b128(__builtin_bit_cast(u32x4, v), ro, ovoff + g * 256, obase + (unsigned)(AI * 128 + m * 16 + j) * 65536u, AUX_NT);
}
__device__ __forceinline__ void epi_hold(const Ev& e) { asm volatile("" :: "v"(e.v0), "v"(e.v1), "v"(e.v2), "v"(e.v3)); }

__global__ __launch_bounds__(256, 1) void fused_gemm(Args a) {
    __shared__ __attribute__((aligned(16))) char shm[LDS_BYTES];
    LAS char* lds = (LAS char*)shm;
    const int tid = threadIdx.x, wid = __builtin_amdgcn_readfirstlane(tid >> 6), lane = tid & 63;
    const int bid = blockIdx.x;
    const unsigned rank = (unsigned)a.rank & (NR - 1);
    char* const myws = a.myws;
    GAS unsigned* const ctr = (GAS unsigned*)(myws + WS_CTR + bid * 128);
    const unsigned ctr_old = __hip_atomic_load(ctr, __ATOMIC_RELAXED, __HIP_MEMORY_SCOPE_AGENT);
#define CALL_INIT() const unsigned call = __builtin_amdgcn_readfirstlane(ctr_old) + 1u; const size_t doff = WS_DATA + (size_t)(call & 1u) * DATA_BYTES
#define CALL_ADVANCE() do { if (tid == 0) __hip_atomic_store(ctr, call, __ATOMIC_RELAXED, __HIP_MEMORY_SCOPE_AGENT); } while (0)
    const __amdgpu_buffer_rsrc_t rx = __builtin_amdgcn_make_buffer_rsrc((void*)a.x, 0, (unsigned)M * KS, 0x00020000);
    const __amdgpu_buffer_rsrc_t rw = __builtin_amdgcn_make_buffer_rsrc((void*)a.w, 0, (unsigned)KS * N, 0x00020000);
    const unsigned wv = (unsigned)((lane >> 4) * 16) * N + wid * 64 + 4 * (lane & 15);
    if (bid < NCOMM) {
        const unsigned xv = (unsigned)(wid * 32 + (lane & 15)) * KS + (lane >> 4) * 16;
        const unsigned ov = wid * 4096 + lane * 16;
        unsigned rawA[16], rawB[16], tick = 0; u32x4 xrA[2], xrB[2];
        unsigned u0 = (bid << 16) | a.order[bid >> 2], u1 = ((bid + NCOMM) << 16) | a.order[(bid + NCOMM) >> 2];
        unsigned u2 = ((bid + 2 * NCOMM) << 16) | a.order[(bid + 2 * NCOMM) >> 2], u3 = ((bid + 3 * NCOMM) << 16) | a.order[(bid + 3 * NCOMM) >> 2], pfl = 0;
#define COMM_LOAD(raw, xr, u) do { const unsigned _p = (u) & 255u, _k = ((u) >> 8) & 3u, _h = ((u) >> 16) & 1u, _f = ((u) >> 17) & 1u; \
            const bool _x = _p < 16; \
            const unsigned sox = _x ? (_p * 256u + _f * 128u) * KS + _k * 128u + _h * 64u : 0u, sow = _x ? 0u : (_k * 128u + _h * 64u) * N + (_p - 16u) * 256u; \
            const __amdgpu_buffer_rsrc_t rsx = __builtin_amdgcn_make_buffer_rsrc((void*)a.x, 0, _x ? (unsigned)M * KS : 0u, 0x00020000); \
            const __amdgpu_buffer_rsrc_t rsw = __builtin_amdgcn_make_buffer_rsrc((void*)a.w, 0, _x ? 0u : (unsigned)KS * N, 0x00020000); \
            _Pragma("unroll") for (int i = 0; i < 2; ++i) xr[i] = __builtin_amdgcn_raw_buffer_load_b128(rsx, xv + i * 16 * KS, sox, 0); \
            _Pragma("unroll") for (int i = 0; i < 16; ++i) raw[i] = __builtin_amdgcn_raw_buffer_load_b32(rsw, wv, sow + i * N, 0); } while (0)
        COMM_LOAD(rawA, xrA, u0); COMM_LOAD(rawB, xrB, u1);
        { const unsigned e0 = a.order[tid], e1 = a.order[256 + (tid < 64 ? tid : 63)];
          *(LAS unsigned*)(lds + tid * 4) = e0; *(LAS unsigned*)(lds + 1024 + tid * 4) = e1; }
        CALL_INIT();
        __amdgpu_buffer_rsrc_t rd[NR - 1];
#pragma unroll
        for (int i = 0; i < NR - 1; ++i) rd[i] = __builtin_amdgcn_make_buffer_rsrc(a.peer[(unsigned)(i + 2 * wid + bid) % (NR - 1)] + doff, 0, DATA_BYTES, 0x00020000);
        GAS unsigned* fl = (GAS unsigned*)(a.peer[lane < NR - 1 ? lane : 0] + WS_FLAGS) + rank * 4;
        char* const tickp = myws + WS_TICKET;
        const unsigned tbase = (call - 1u) * NUNIT - 4u * NCOMM, one = 1u;
#define COMM_ST(o, vo) _Pragma("unroll") for (int d = 0; d < NR - 1; ++d) __builtin_amdgcn_raw_buffer_store_b128(o, rd[d], vo, sd, AUX_SYS)
#define COMM_STEP(raw, xr, first) do { const unsigned _p0 = u0 & 255u, _k0 = (u0 >> 8) & 3u, _q0 = (u0 >> 16) & 3u; \
            u32x4 o0, o1; \
            { const unsigned sel = (_q0 & 2u) ? 0x07030602u : 0x05010400u; \
              _Pragma("unroll") for (int b = 0; b < 4; ++b) { \
                    const unsigned t0 = __builtin_amdgcn_perm(raw[4 * b + 1], raw[4 * b], sel), t2 = __builtin_amdgcn_perm(raw[4 * b + 3], raw[4 * b + 2], sel); \
                    o0[b] = __builtin_amdgcn_perm(t2, t0, 0x05040100u) | xr[0][b]; o1[b] = __builtin_amdgcn_perm(t2, t0, 0x07060302u) | xr[1][b]; } } \
            asm volatile("" : "+v"(o0), "+v"(o1)); \
            const unsigned sd = (_k0 * NR + rank) * SLAB + _p0 * BLK + (_q0 & 2u) * (HB / 2) + (_q0 & 1u) * 1024u; \
            COMM_LOAD(raw, xr, u2); SCHED; COMM_ST(o0, ov); WAIT_V(25); \
            if (!(first) && tid == 0) { asm volatile("" : "+v"(tick)); const unsigned _t = tick - tbase; \
                *(volatile LAS unsigned*)(lds + LDS_MISC + 8) = (_t << 16) | *(volatile LAS unsigned*)(lds + ((_t & 2047u) & ~3u)); } \
            WAIT_L(0); BAR; \
            unsigned un = 0; \
            if (!(first)) { un = __builtin_amdgcn_readfirstlane(*(volatile LAS unsigned*)(lds + LDS_MISC + 8)); \
                if (tid < NR - 1) __hip_atomic_store(fl + pfl, call, __ATOMIC_RELAXED, __HIP_MEMORY_SCOPE_SYSTEM); } \
            else CALL_ADVANCE(); \
            if (tid == 0) asm volatile("global_atomic_add %0, %1, %2, off sc0" : "=&v"(tick) : "v"(tickp), "v"(one) : "memory"); \
            COMM_ST(o1, ov + 2048); \
            BAR;     \
            pfl = _p0 * 128u + _k0 * (NR * 4u) + _q0; \
            u0 = u1; u1 = u2; if (first) { u2 = u3; } else { u2 = un; } } while (0)
        COMM_STEP(rawA, xrA, true);
        for (;;) {
            COMM_STEP(rawB, xrB, false); if ((u0 >> 16) >= NUNIT) break;
            COMM_STEP(rawA, xrA, false); if ((u0 >> 16) >= NUNIT) break;
        }
        WAIT_V(0); BAR;
        if (tid < NR - 1) __hip_atomic_store(fl + pfl, call, __ATOMIC_RELAXED, __HIP_MEMORY_SCOPE_SYSTEM);
        return;
    }

    const int gw = bid - NCOMM;
    const unsigned long long tl = a.tiles[gw];
    const int ntiles = (int)(tl >> 50) & 7;
    const unsigned koff0 = (unsigned)(tl >> 56) & 31u;
    const int wr = wid >> 1, wc = wid & 1, fr = lane & 15, fq = lane >> 4;
    constexpr int NLOC = 2 * NPANEL;
    const int lw = gw - (NWG - NCOMM - NLOC);
    unsigned raw[4][16];
    if (lw >= 0) {
        const unsigned h = lw & 1, p = lw >> 1;
        if (p < 16) {
            const unsigned xv = (unsigned)(wid * 64 + (lane & 15)) * KS + h * 64 + (lane >> 4) * 16;
#pragma unroll
            for (int i = 0; i < 4; ++i)
#pragma unroll
                for (int j = 0; j < 4; ++j) { const u32x4 v = __builtin_amdgcn_raw_buffer_load_b128(rx, xv + j * 16 * KS, p * (256u * KS) + i * 128u, 0);
                    raw[i][4 * j] = v[0]; raw[i][4 * j + 1] = v[1]; raw[i][4 * j + 2] = v[2]; raw[i][4 * j + 3] = v[3]; }
        } else {
#pragma unroll
            for (int i = 0; i < 4; ++i)
#pragma unroll
                for (int j = 0; j < 16; ++j) raw[i][j] = __builtin_amdgcn_raw_buffer_load_b32(rw, wv, (i * 128u + h * 64u + j) * N + (p - 16u) * 256u, 0);
        }
    }
    if (tid == 0) *(LAS unsigned*)(lds + LDS_MISC + 4) = 0u;
    CALL_INIT();
    const __amdgpu_buffer_rsrc_t rin = __builtin_amdgcn_make_buffer_rsrc(myws + doff, 0, DATA_BYTES, 0x00020000);
    const __amdgpu_buffer_rsrc_t rfl = __builtin_amdgcn_make_buffer_rsrc(myws + WS_FLAGS, 0, FLAG_BYTES, 0x00020000);
    GAS unsigned* tmo = (GAS unsigned*)(myws + WS_TMO);
    if (lw >= 0) {
        const unsigned h = lw & 1, p = lw >> 1;
#pragma unroll
        for (int i = 0; i < 4; ++i) {
            const unsigned so = (i * NR + rank) * SLAB + p * BLK + h * 1024;
            if (p < 16) {
#pragma unroll
                for (int j = 0; j < 4; ++j) __builtin_amdgcn_raw_buffer_store_b128((u32x4){raw[i][4 * j], raw[i][4 * j + 1], raw[i][4 * j + 2], raw[i][4 * j + 3]}, rin, (wid * 4 + j) * 2048 + lane * 16, so, AUX_SYS);
            } else {
                u32x4 o[4];
#pragma unroll
                for (int b = 0; b < 4; ++b) {
                    const unsigned r0 = raw[i][4 * b], r1 = raw[i][4 * b + 1], r2 = raw[i][4 * b + 2], r3 = raw[i][4 * b + 3];
                    const unsigned t0 = __builtin_amdgcn_perm(r1, r0, 0x05010400), t1 = __builtin_amdgcn_perm(r1, r0, 0x07030602);
                    const unsigned t2 = __builtin_amdgcn_perm(r3, r2, 0x05010400), t3 = __builtin_amdgcn_perm(r3, r2, 0x07030602);
                    o[0][b] = __builtin_amdgcn_perm(t2, t0, 0x05040100); o[1][b] = __builtin_amdgcn_perm(t2, t0, 0x07060302);
                    o[2][b] = __builtin_amdgcn_perm(t3, t1, 0x05040100); o[3][b] = __builtin_amdgcn_perm(t3, t1, 0x07060302);
                }
#pragma unroll
                for (int c = 0; c < 4; ++c) __builtin_amdgcn_raw_buffer_store_b128(o[c], rin, (c >> 1) * HB + wid * 4096 + (c & 1) * 2048 + lane * 16, so, AUX_SYS);
            }
        }
    }
    asm volatile("s_waitcnt vmcnt(0) lgkmcnt(0)" ::: "memory"); BAR;
    CALL_ADVANCE();
    if (lw >= 0 && tid < 8)
        __hip_atomic_store((GAS unsigned*)(myws + WS_FLAGS) + (lw >> 1) * 128u + ((tid >> 1) * NR + rank) * 4u + (tid & 1) * 2 + (lw & 1), call, __ATOMIC_RELAXED, __HIP_MEMORY_SCOPE_SYSTEM);
    const float scale = __builtin_bit_cast(float, __builtin_amdgcn_readfirstlane(__builtin_bit_cast(int, a.sx[0] * a.sw[0])));
    const unsigned voff = tid * 16, ldsw = wid * 1024;
    int aoff = wr * 8192 + lane * 16, boff = 4 * HB + wc * 8192 + lane * 16;
    asm volatile("" : "+v"(aoff), "+v"(boff));
    const unsigned ovoff = (unsigned)(wr * 64 + 4 * fq) * 65536u + wc * 512 + fr * 16;
#define TILE(i) ((unsigned)(tl >> (10 * (i))) & 0x3ffu)
#define SA(b, h) (((b) * 2 + (h)) * HB)
#define SB(b, h) ((4 + (b) * 2 + (h)) * HB)
#define G1(bufoff, soff, i) __builtin_amdgcn_raw_ptr_buffer_load_lds(rin, (LAS void*)(lds + (bufoff) + ldsw + (i) * 4096), 16, voff, (soff) + (i) * 4096, 0, AUX_SYS)
#define STAGE4(bufoff, soff) do { G1(bufoff, soff, 0); G1(bufoff, soff, 1); G1(bufoff, soff, 2); G1(bufoff, soff, 3); } while (0)
#define RD(p) (*(const LAS i32x4*)(p))
#define CAT(lo, hi) __builtin_shufflevector(lo, hi, 0, 1, 2, 3, 4, 5, 6, 7)
#define MF(Z, ai, bj, m, n, RA, RB) mfma<acc_idx(ai, bj, m, n), Z>(CAT(RA[2 * (m)], RA[2 * (m) + 1]), CAT(RB[2 * (n)], RB[2 * (n) + 1]))
#define EPA(eai, e, V) do { if constexpr ((eai) >= 0) epi_a<(eai) < 0 ? 0 : (eai), (e)>(V, esc); } while (0)
#define EPB(eai, e, V, W) do { if constexpr ((eai) >= 0) { epi_b<(eai) < 0 ? 0 : (eai), (e)>(V, esc, erout, ovoff, eobase); epi_hold(W); } } while (0)
#define G1Q(a_, b_, c_) G1(a_, b_, c_)
#define RDQ(d_, p_) d_ = RD(p_)
#define QUAD(Z, ai, bj, RA, RB, RN, rdoff, stoff, soff, vm, eai, e0, hook) do { u32x4 _fv; \
        if constexpr ((vm) >= 0) { asm volatile("s_waitcnt vmcnt(%0) lgkmcnt(0)" :: "n"((vm) < 0 ? 0 : (vm)) : "memory"); BAR; } SCHED; \
        MF(Z, ai, bj, 0, 0, RA, RB); if constexpr ((hook) == 1 || (hook) == 2) _fv = FLAG_LOAD((hook) - 1); RDQ(RN[0], lds + (rdoff)); EPA(eai, e0 + 0, ev0); SCHED; \
        MF(Z, ai, bj, 0, 1, RA, RB); RDQ(RN[1], lds + (rdoff) + 1024); EPB(eai, e0 + 0, ev0, ev1); SCHED; \
        MF(Z, ai, bj, 0, 2, RA, RB); RDQ(RN[2], lds + (rdoff) + 2048); EPA(eai, e0 + 1, ev1); SCHED; \
        MF(Z, ai, bj, 0, 3, RA, RB); RDQ(RN[3], lds + (rdoff) + 3072); EPB(eai, e0 + 1, ev1, ev0); SCHED; \
        MF(Z, ai, bj, 1, 0, RA, RB); RDQ(RN[4], lds + (rdoff) + 4096); EPA(eai, e0 + 2, ev0); SCHED; \
        MF(Z, ai, bj, 1, 1, RA, RB); RDQ(RN[5], lds + (rdoff) + 5120); EPB(eai, e0 + 2, ev0, ev1); if constexpr ((hook) == 1 || (hook) == 2) FLAG_MASK(_fv); SCHED; \
        MF(Z, ai, bj, 1, 2, RA, RB); RDQ(RN[6], lds + (rdoff) + 6144); EPA(eai, e0 + 3, ev1); if constexpr ((hook) == 1) PICK(kbA, kbB); SCHED; \
        MF(Z, ai, bj, 1, 3, RA, RB); RDQ(RN[7], lds + (rdoff) + 7168); EPB(eai, e0 + 3, ev1, ev0); SCHED; \
        MF(Z, ai, bj, 2, 0, RA, RB); G1Q(stoff, soff, 0); EPA(eai, e0 + 4, ev0); SCHED; \
        MF(Z, ai, bj, 2, 1, RA, RB); EPB(eai, e0 + 4, ev0, ev1); if constexpr ((hook) == 3 || (hook) == 4) FLAG_FETCH(4 - (hook)); SCHED; \
        MF(Z, ai, bj, 2, 2, RA, RB); G1Q(stoff, soff, 1); EPA(eai, e0 + 5, ev1); SCHED; \
        MF(Z, ai, bj, 2, 3, RA, RB); EPB(eai, e0 + 5, ev1, ev0); SCHED; \
        MF(Z, ai, bj, 3, 0, RA, RB); G1Q(stoff, soff, 2); EPA(eai, e0 + 6, ev0); SCHED; \
        MF(Z, ai, bj, 3, 1, RA, RB); EPB(eai, e0 + 6, ev0, ev1); SCHED; \
        MF(Z, ai, bj, 3, 2, RA, RB); G1Q(stoff, soff, 3); EPA(eai, e0 + 7, ev1); SCHED; \
        MF(Z, ai, bj, 3, 3, RA, RB); EPB(eai, e0 + 7, ev1, ev0); if constexpr ((hook) == 2) PICK(kaA, kaB); SCHED; } while (0)

    unsigned rmask = 0, done = 0, kaA, kaB, kbA, kbB;
#define BASE_A(tile) (((tile) & 15u) * BLK)
#define BASE_B(tile) ((16u + ((tile) >> 4)) * BLK)
#define FLAG_SOFF(tile) ((wid < 2 ? ((tile) & 15u) : 16u + ((tile) >> 4)) * 512u + (wid & 1) * 256u)
    unsigned pA = BASE_A(TILE(0)), pB = BASE_B(TILE(0)), psoff = FLAG_SOFF(TILE(0)), fsoff = psoff;
    const unsigned fvoff = lane * 4;
    int froff = LDS_FLAGS + ((lane & 32) + ((lane + koff0) & 31)) * 16; asm volatile("" : "+v"(froff));
#define FLAG_FETCH_AT(slot, so) __builtin_amdgcn_raw_ptr_buffer_load_lds(rfl, (LAS void*)(lds + LDS_FLAGS + (slot) * 1024 + wid * 256), 4, fvoff, so, 0, AUX_SYS)
#define FLAG_FETCH(slot) FLAG_FETCH_AT(slot, fsoff)
#define FLAG_LOAD(slot) (*(volatile LAS u32x4*)(lds + froff + (slot) * 1024))
#define FLAG_MASK(_v) do { const unsigned _a = _v[0] < _v[1] ? _v[0] : _v[1], _b = _v[2] < _v[3] ? _v[2] : _v[3]; \
            const unsigned long long _m = __builtin_amdgcn_ballot_w64((_a < _b ? _a : _b) >= call); \
            rmask |= (unsigned)_m & (unsigned)(_m >> 32); } while (0)
#define TWAIT(t0) do {} while (0)
#define PICK(oA, oB) do { \
            if (__builtin_expect((rmask & ~done) == 0u, 0)) {        \
                const unsigned long long _t0 = __builtin_amdgcn_s_memrealtime(); \
                for (;;) { \
                    FLAG_FETCH_AT(2, psoff); \
                    if (wid == 0 && __builtin_amdgcn_s_memrealtime() - _t0 > 10000ull        \
                        && (__builtin_amdgcn_readfirstlane(__hip_atomic_load(tmo, __ATOMIC_RELAXED, __HIP_MEMORY_SCOPE_AGENT)) == call \
                            || __builtin_amdgcn_s_memrealtime() - _t0 > DEADLINE_TICKS)) { \
                        __hip_atomic_store(tmo, call, __ATOMIC_RELAXED, __HIP_MEMORY_SCOPE_AGENT); \
                        *(volatile LAS unsigned*)(lds + LDS_MISC + 4) = 1u; } \
                    asm volatile("s_waitcnt vmcnt(0) lgkmcnt(0)" ::: "memory"); BAR; \
                    const u32x4 _pv = FLAG_LOAD(2); FLAG_MASK(_pv); \
                    if (__builtin_amdgcn_readfirstlane(*(volatile LAS unsigned*)(lds + LDS_MISC + 4)) != 0u) rmask = ~0u; \
                    WAIT_L(0); BAR; \
                    if ((rmask & ~done) != 0u) break; \
                    __builtin_amdgcn_s_sleep(POLL_SLEEP); \
                } \
                TWAIT(_t0); \
            } \
            const unsigned _i = __builtin_ctz(rmask & ~done), _q = (_i + koff0) & 31u; done |= 1u << _i; oA = _q * SLAB + pA; oB = _q * SLAB + pB; } while (0)
#define ITER(KIND, sw) do { \
        if (sw) fsoff = nsoff; \
        QUAD(KIND == 0, 0, 0, RX, B0a, RY, SA(1, 0) + aoff, SB(0, 1), kaB + HB, (KIND == 0 ? 49 : 17), (KIND == 0 ? 1 : -1), 0, 0); \
        QUAD(KIND == 0, 0, 1, RX, B1a, B1b, SA(1, 1) + boff, SA(0, 0), kaA, -1, (KIND == 0 ? 1 : -1), 8, 3); \
        QUAD(false, 0, 1, RY, B1b, B0b, SA(1, 0) + boff, SA(1, 0), kbA, (KIND == 0 ? 49 : 17), (KIND == 0 ? 1 : -1), 16, 1); \
        QUAD(false, 0, 0, RY, B0b, RX, SA(0, 1) + aoff, SB(1, 1), kbB + HB, -1, (KIND == 0 ? 1 : -1), 24, 0); \
        if constexpr (KIND == 2) { SLOT_WAIT(); \
            esc = *(volatile LAS unsigned*)(lds + LDS_MISC + 4) != 0u ? __builtin_nanf("") : scale; \
            esc = __builtin_bit_cast(float, __builtin_amdgcn_readfirstlane(__builtin_bit_cast(int, esc))); } \
        QUAD(KIND == 0, 1, 0, RX, B0a, RY, SA(1, 1) + aoff, SB(1, 0), kbB, (KIND == 0 ? 49 : 17), (KIND == 2 ? 0 : -1), 0, 0); \
        QUAD(KIND == 0, 1, 1, RX, B1a, B0a, SA(0, 0) + boff, SA(0, 1), kaA + HB, -1, (KIND == 2 ? 0 : -1), 8, 4); \
        if (sw) { done = 0; rmask = nmask; pA = nA; pB = nB; psoff = nsoff; } \
        QUAD(false, 1, 1, RY, B1b, B1a, SA(0, 1) + boff, SA(1, 1), kbA + HB, (KIND == 1 ? 17 : 33), (KIND == 2 ? 0 : -1), 16, 2); \
        QUAD(false, 1, 0, RY, B0b, RX, SA(0, 0) + aoff, SB(0, 0), kaB, -1, (KIND == 2 ? 0 : -1), 24, 0); } while (0)

#define STAMP(ui, it) do {} while (0)
    i32x4 RX[8], RY[8], B0a[8], B1a[8], B0b[8], B1b[8];
    PICK(kaA, kaB);
    STAGE4(SA(0, 0), kaA); STAGE4(SB(0, 0), kaB); STAGE4(SB(0, 1), kaB + HB); STAGE4(SA(0, 1), kaA + HB);
    PICK(kbA, kbB);
    STAGE4(SA(1, 0), kbA); STAGE4(SB(1, 1), kbB + HB); STAGE4(SB(1, 0), kbB); STAGE4(SA(1, 1), kbA + HB);
    WAIT_V(16); BAR;
#pragma unroll
    for (int i = 0; i < 8; ++i) { RX[i] = RD(lds + SA(0, 0) + aoff + i * 1024); B0a[i] = RD(lds + SA(0, 0) + boff + i * 1024); B1a[i] = RD(lds + SA(0, 1) + boff + i * 1024); }
    WAIT_L(0); BAR;
    PICK(kaA, kaB);
    STAGE4(SB(0, 0), kaB); FLAG_FETCH(0);
    WAIT_V(0);
    __amdgpu_buffer_rsrc_t erout = __builtin_amdgcn_make_buffer_rsrc((void*)a.out, 0, 0, 0x00020000);
    unsigned eobase = 0; float esc = 0.f;
    Ev ev0 = {0.f, 0.f, 0.f, 0.f}, ev1 = {0.f, 0.f, 0.f, 0.f};
    unsigned long long slot = 0; char* const slotp = myws + WS_SLOT + (bid & 7) * 128;
#define SLOT_WAIT() do { if (SLOT_D > 0 && wid == 0) { asm volatile("" : "+v"(slot));     \
                const unsigned _s = __builtin_amdgcn_readfirstlane((unsigned)slot); const unsigned long long _t0 = __builtin_amdgcn_s_memrealtime(); \
                for (;;) { const unsigned long long _n = __builtin_amdgcn_s_memrealtime(); if ((int)(_s - (unsigned)_n) <= 0 || _n - _t0 > 2000ull) break; __builtin_amdgcn_s_sleep(2); } TWAIT(_t0); } } while (0)
    for (int ui = 0; ui < ntiles; ++ui) {
        const bool more = ui + 1 < ntiles; const unsigned nt = TILE(more ? ui + 1 : ui);
        const unsigned nA = BASE_A(nt), nB = BASE_B(nt), nsoff = FLAG_SOFF(nt), nmask = more ? 0u : ~0u;
        STAMP(ui, 0);
        ITER(0, false);
        for (int it = 1; it < 14; ++it) { STAMP(ui, it); ITER(1, false); }
        if (SLOT_D > 0 && wid == 0) {
            const unsigned long long want = __builtin_amdgcn_s_memrealtime() + SLOT_LEAD, dd = SLOT_D;
            if (lane == 0) asm volatile("global_atomic_umax_x2 %1, %2, off sc1\n\tglobal_atomic_add_x2 %0, %1, %3, off sc0 sc1" : "=&v"(slot) : "v"(slotp), "v"(want), "v"(dd) : "memory");
        }
        STAMP(ui, 14);
        ITER(1, true);
        erout = __builtin_amdgcn_make_buffer_rsrc((void*)a.out, 0, (unsigned)M * N * 4u, 0x00020000);
        eobase = (TILE(ui) & 15u) * (256u * 65536u) + (TILE(ui) >> 4) * 1024u;
        STAMP(ui, 15);
        ITER(2, false);
    }
    STAMP(ntiles, 0);
    asm volatile("s_nop 15" ::: "memory");
#define EP2(e) EPA(1, e, ev0); EPB(1, e, ev0, ev1); EPA(1, e + 1, ev1); EPB(1, e + 1, ev1, ev0)
#define EP8(e) EP2(e); EP2(e + 2); EP2(e + 4); EP2(e + 6)
    EP8(0); EP8(8); EP8(16); EP8(24);
    WAIT_V(0);
}

#define P_KS0 2.0
#define P_KS1 0.95
#define P_SX -3.0
#define P_RX 1.18
#define P_SPS 12.0
#define P_SPE 12.0
#define P_LEN2 38.0
#define P_S3 44.0
#define P_SP3 16.0
#define P_G1X -2.0
void make_tables(Args& a, int rank) {
    double key[320]; int n = 0;
    auto koff = [](int p) { return p < 4 ? 0 : (int)(P_KS0 + (p - 4) * P_KS1 + 0.5); };
    auto add = [&](int panel, double start, double rate, int rot, int off) {
        for (int kt = 0; kt < 4; ++kt) {
            const double k = start + ((kt * NR + ((rank + rot) & (NR - 1)) - off) & 31) * rate + panel * 1e-6;
            int i = n++;
            for (; i > 0 && key[i - 1] > k; --i) { key[i] = key[i - 1]; a.order[i] = a.order[i - 1]; }
            key[i] = k; a.order[i] = (unsigned)((kt << 8) | panel);
        }
    };
    for (int i = 0; i < 8; ++i) { add(i, -1000.0 + P_G1X, 1.0, i, 0); add(16 + i, -1000.0, 1.0, i, 0); add(8 + i, P_SX, P_RX, i, 0); }
    for (int p = 4; p < 15; ++p) { const double f = (p - 4) / 10.0; add(16 + 8 + (p - 4), f * P_SPS, (P_LEN2 + f * (P_SPE - P_SPS)) / 32.0, p, koff(p)); }
    for (int g = 0; g < 3; ++g)
        for (int p = 0; p < 15; ++p) add(16 + 19 + g * 15 + p, P_S3 + g * 38.0 + p * P_SP3 / 15.0, 1.18, p, koff(p));
    for (int g = 0; g < NWG - NCOMM; ++g) {
        const int xcd = g & 7, r = (xcd >> 2) * 30 + (g >> 3), p = (2 * (r >> 2)) % 15, m = (xcd & 3) * 4 + (r & 3);
        int mp[5], np[5], nt = 0;
        if (p < 4) { mp[0] = m & 7; mp[1] = 8 + (m & 7); np[0] = np[1] = 2 * p + (m >> 3); nt = 2; }
        else { mp[0] = m; np[0] = 8 + p - 4; nt = 1; }
        for (int k = 0; k < 3; ++k) { mp[nt] = m; np[nt] = 19 + k * 15 + p; ++nt; }
        unsigned long long tl = ((unsigned long long)nt << 50) | ((unsigned long long)koff(p) << 56);
        for (int i = 0; i < nt; ++i) tl |= (unsigned long long)((np[i] << 4) | mp[i]) << (10 * i);
        a.tiles[g] = tl;
    }
}
}

extern "C" void kernel_launch(int rank, int nranks, ncclComm_t comm,
                              hipStream_t stream,
                              void* const* d_in, const int* in_sizes, int n_in,
                              void* d_out, int out_size,
                              const int* in_dtypes, int out_dtype,
                              void* const* d_out_all,
                              void* const* d_ws_all, size_t ws_bytes) {
    (void)comm; (void)in_dtypes; (void)out_dtype; (void)d_out_all;
    if (nranks != NR || (unsigned)rank >= (unsigned)NR || n_in != 4 || in_sizes[0] != M * KS || in_sizes[1] != KS * N ||
        out_size != M * N || ws_bytes < WS_NEED) {
        fprintf(stderr, "kernel_launch: unexpected shapes\n");
        return;
    }
    Args a;
    a.x = (const unsigned char*)d_in[0]; a.w = (const unsigned char*)d_in[1];
    a.sx = (const float*)d_in[2]; a.sw = (const float*)d_in[3];
    a.out = (float*)d_out;
    a.myws = (char*)d_ws_all[rank];
    for (int i = 0; i < NR - 1; ++i) a.peer[i] = (char*)d_ws_all[(rank + 1 + i) % NR];
    a.rank = rank;
    make_tables(a, rank);
    hipLaunchKernelGGL(fused_gemm, dim3(NWG), dim3(256), 0, stream, a);
    HIP_CHECK(hipGetLastError());
}
```

```cpp
#include <hip/hip_runtime.h>
#include <rccl/rccl.h>
#include <cstdio>

#define HIP_CHECK(expr) do { hipError_t _e = (expr); if (_e != hipSuccess) { \
    fprintf(stderr, "HIP error %s at %s:%d\n", hipGetErrorString(_e), __FILE__, __LINE__); \
    return; } } while (0)

#define LAS __attribute__((address_space(3)))
#define GAS __attribute__((address_space(1)))
typedef int i32x4 __attribute__((ext_vector_type(4)));
typedef unsigned u32x4 __attribute__((ext_vector_type(4)));
typedef int i32x8 __attribute__((ext_vector_type(8)));
typedef float f32x4 __attribute__((ext_vector_type(4)));

namespace {
constexpr int NR = 8;
constexpr int M = 4096, N = 16384, KS = 512;
constexpr unsigned BLK = 32768, HB = 16384;
constexpr unsigned NPANEL = 80;
constexpr unsigned SLAB = NPANEL * BLK;
constexpr unsigned DATA_BYTES = 32 * SLAB;
constexpr unsigned WS_FLAGS = 0;
constexpr unsigned FLAG_BYTES = NPANEL * 512;
constexpr unsigned WS_CTR = 65536;
constexpr unsigned WS_TMO = WS_CTR + 256 * 128;
constexpr unsigned WS_TICKET = WS_TMO + 128;
constexpr unsigned WS_TDONE = WS_TICKET + 128;
#define NZ 1
constexpr int NSHARE = 2 * NZ + 1;
constexpr unsigned WS_FWD = WS_TDONE + 128;
constexpr unsigned NBLOCK = 320, NUNIT = 4 * NBLOCK;
constexpr size_t WS_DATA = 1u << 20;
constexpr size_t WS_NEED = WS_DATA + 2 * (size_t)DATA_BYTES;
constexpr int NCOMM = 16, NWG = 256;
constexpr unsigned long long DEADLINE_TICKS = 50000000ull;
constexpr int AUX_SYS = 17;
#define POLL_SLEEP 8
#define OAUX 2
constexpr int AUX_NT = OAUX;
constexpr unsigned LDS_FLAGS = 8 * HB;
constexpr unsigned LDS_MISC = LDS_FLAGS + 2 * 1024;
constexpr unsigned LDS_BYTES = LDS_MISC + 64;

struct Args {
    const unsigned char* x; const unsigned char* w; const float* sx; const float* sw; float* out;
    char* myws; char* peer[NR - 1];
    char* peer_out[NR - 1];
    int rank;
    unsigned shared[NSHARE];
    unsigned order[NBLOCK];
    unsigned long long tiles[NWG - NCOMM];
};

#define WAIT_V(n) asm volatile("s_waitcnt vmcnt(" #n ")" ::: "memory")
#define WAIT_L(n) asm volatile("s_waitcnt lgkmcnt(" #n ")" ::: "memory")
#define BAR __builtin_amdgcn_s_barrier()
#define SCHED __builtin_amdgcn_sched_barrier(0)

#define ACLOB "a0","a1","a2","a3","a4","a5","a6","a7","a8","a9","a10","a11","a12","a13","a14","a15","a16","a17","a18","a19","a20","a21","a22","a23","a24","a25","a26","a27","a28","a29","a30","a31","a32","a33","a34","a35","a36","a37","a38","a39","a40","a41","a42","a43","a44","a45","a46","a47","a48","a49","a50","a51","a52","a53","a54","a55","a56","a57","a58","a59","a60","a61","a62","a63","a64","a65","a66","a67","a68","a69","a70","a71","a72","a73","a74","a75","a76","a77","a78","a79","a80","a81","a82","a83","a84","a85","a86","a87","a88","a89","a90","a91","a92","a93","a94","a95","a96","a97","a98","a99","a100","a101","a102","a103","a104","a105","a106","a107","a108","a109","a110","a111","a112","a113","a114","a115","a116","a117","a118","a119","a120","a121","a122","a123","a124","a125","a126","a127","a128","a129","a130","a131","a132","a133","a134","a135","a136","a137","a138","a139","a140","a141","a142","a143","a144","a145","a146","a147","a148","a149","a150","a151","a152","a153","a154","a155","a156","a157","a158","a159","a160","a161","a162","a163","a164","a165","a166","a167","a168","a169","a170","a171","a172","a173","a174","a175","a176","a177","a178","a179","a180","a181","a182","a183","a184","a185","a186","a187","a188","a189","a190","a191","a192","a193","a194","a195","a196","a197","a198","a199","a200","a201","a202","a203","a204","a205","a206","a207","a208","a209","a210","a211","a212","a213","a214","a215","a216","a217","a218","a219","a220","a221","a222","a223","a224","a225","a226","a227","a228","a229","a230","a231","a232","a233","a234","a235","a236","a237","a238","a239","a240","a241","a242","a243","a244","a245","a246","a247","a248","a249","a250","a251","a252","a253","a254","a255"
constexpr int acc_idx(int ai, int bj, int m, int n) { return (((ai * 2 + bj) * 4 + m) * 4 + n) * 4; }
template <int IDX, bool ZERO> __device__ __forceinline__ void mfma(const i32x8& x, const i32x8& y) {
    if constexpr ((IDX & 63) == 0) {
        if constexpr (ZERO) asm volatile("v_mfma_f32_16x16x128_f8f6f4 a[%c2:%c3], %0, %1, 0 cbsz:1 blgp:1" :: "v"(x), "v"(y), "i"(IDX), "i"(IDX + 3) : ACLOB);
        else asm volatile("v_mfma_f32_16x16x128_f8f6f4 a[%c2:%c3], %0, %1, a[%c2:%c3] cbsz:1 blgp:1" :: "v"(x), "v"(y), "i"(IDX), "i"(IDX + 3) : ACLOB);
    } else {
        if constexpr (ZERO) asm volatile("v_mfma_f32_16x16x128_f8f6f4 a[%c2:%c3], %0, %1, 0 cbsz:1 blgp:1" :: "v"(x), "v"(y), "i"(IDX), "i"(IDX + 3));
        else asm volatile("v_mfma_f32_16x16x128_f8f6f4 a[%c2:%c3], %0, %1, a[%c2:%c3] cbsz:1 blgp:1" :: "v"(x), "v"(y), "i"(IDX), "i"(IDX + 3));
    }
}
typedef float f32x2 __attribute__((ext_vector_type(2)));
struct Ev { float v0, v1, v2, v3; };
#define EPI_MODE 1
template <int Q, int G> __device__ __forceinline__ void epi_gap(Ev& e, const Ev& other, float sc, __amdgpu_buffer_rsrc_t ro, unsigned ovoff, unsigned obase) {
    constexpr int E = G / 3, part = G % 3, m = E >> 2, j = E & 3, ai = Q >> 1, bj = Q & 1;
    if constexpr (part == 0) {
        asm volatile("v_accvgpr_read_b32 %0, a%c4\n\tv_accvgpr_read_b32 %1, a%c5\n\tv_accvgpr_read_b32 %2, a%c6\n\tv_accvgpr_read_b32 %3, a%c7"
                     : "=v"(e.v0), "=v"(e.v1), "=v"(e.v2), "=v"(e.v3)
                     : "i"(acc_idx(ai, bj, m, 0) + j), "i"(acc_idx(ai, bj, m, 1) + j), "i"(acc_idx(ai, bj, m, 2) + j), "i"(acc_idx(ai, bj, m, 3) + j));
        if constexpr (EPI_MODE == 1) asm volatile("v_mul_f32 %0, %2, %0\n\tv_mul_f32 %1, %2, %1" : "+v"(e.v0), "+v"(e.v1) : "s"(sc));
    } else if constexpr (part == 1) {
        if constexpr (EPI_MODE == 0)
            asm volatile("v_mul_f32 %0, %4, %0\n\tv_mul_f32 %1, %4, %1\n\tv_mul_f32 %2, %4, %2\n\tv_mul_f32 %3, %4, %3\n\tv_maximum3_f32 %0, %0, 0, 0"
                         : "+v"(e.v0), "+v"(e.v1), "+v"(e.v2), "+v"(e.v3) : "s"(sc));
        if constexpr (EPI_MODE == 1)
            asm volatile("v_mul_f32 %2, %4, %2\n\tv_mul_f32 %3, %4, %3\n\tv_maximum3_f32 %0, %0, 0, 0\n\tv_maximum3_f32 %1, %1, 0, 0\n\tv_maximum3_f32 %2, %2, 0, 0\n\tv_maximum3_f32 %3, %3, 0, 0"
                         : "+v"(e.v0), "+v"(e.v1), "+v"(e.v2), "+v"(e.v3) : "s"(sc));
        if constexpr (EPI_MODE >= 2) {
            f32x2 lo = {e.v0, e.v1}, hi = {e.v2, e.v3}; const f32x2 s2 = {sc, sc};
            asm volatile("v_pk_mul_f32 %0, %2, %0\n\tv_pk_mul_f32 %1, %2, %1" : "+v"(lo), "+v"(hi) : "s"(s2));
            e.v0 = lo[0]; e.v1 = lo[1]; e.v2 = hi[0]; e.v3 = hi[1];
            if constexpr (EPI_MODE == 2) asm volatile("v_maximum3_f32 %0, %0, 0, 0\n\tv_maximum3_f32 %1, %1, 0, 0" : "+v"(e.v0), "+v"(e.v1));
            else asm volatile("v_maximum3_f32 %0, %0, 0, 0\n\tv_maximum3_f32 %1, %1, 0, 0\n\tv_maximum3_f32 %2, %2, 0, 0\n\tv_maximum3_f32 %3, %3, 0, 0" : "+v"(e.v0), "+v"(e.v1), "+v"(e.v2), "+v"(e.v3));
        }
    } else {
        if constexpr (EPI_MODE == 0) asm volatile("v_maximum3_f32 %0, %0, 0, 0\n\tv_maximum3_f32 %1, %1, 0, 0\n\tv_maximum3_f32 %2, %2, 0, 0" : "+v"(e.v1), "+v"(e.v2), "+v"(e.v3));
        if constexpr (EPI_MODE == 2) asm volatile("v_maximum3_f32 %0, %0, 0, 0\n\tv_maximum3_f32 %1, %1, 0, 0" : "+v"(e.v2), "+v"(e.v3));
        const f32x4 v = {e.v0, e.v1, e.v2, e.v3};
        __builtin_amdgcn_raw_buffer_store_b128(__builtin_bit_cast(u32x4, v), ro, ovoff + bj * 256, obase + (unsigned)(ai * 128 + m * 16 + j) * 65536u, AUX_NT);
        asm volatile("" :: "v"(other.v0), "v"(other.v1), "v"(other.v2), "v"(other.v3));
    }
}

__global__ __launch_bounds__(256, 1) void fused_gemm(Args a) {
    __shared__ __attribute__((aligned(16))) char shm[LDS_BYTES];
    LAS char* lds = (LAS char*)shm;
    const int tid = threadIdx.x, wid = __builtin_amdgcn_readfirstlane(tid >> 6), lane = tid & 63;
    const int bid = blockIdx.x;
    const unsigned rank = (unsigned)a.rank & (NR - 1);
    char* const myws = a.myws;
    GAS unsigned* const ctr = (GAS unsigned*)(myws + WS_CTR + bid * 128);
    const unsigned ctr_old = __hip_atomic_load(ctr, __ATOMIC_RELAXED, __HIP_MEMORY_SCOPE_AGENT);
#define CALL_INIT() const unsigned call = __builtin_amdgcn_readfirstlane(ctr_old) + 1u; const size_t doff = WS_DATA + (size_t)(call & 1u) * DATA_BYTES
#define CALL_ADVANCE() do { if (tid == 0) __hip_atomic_store(ctr, call, __ATOMIC_RELAXED, __HIP_MEMORY_SCOPE_AGENT); } while (0)
    const __amdgpu_buffer_rsrc_t rx = __builtin_amdgcn_make_buffer_rsrc((void*)a.x, 0, (unsigned)M * KS, 0x00020000);
    const __amdgpu_buffer_rsrc_t rw = __builtin_amdgcn_make_buffer_rsrc((void*)a.w, 0, (unsigned)KS * N, 0x00020000);
    const unsigned wv = (unsigned)((lane >> 4) * 16) * N + wid * 64 + 4 * (lane & 15);
    if (bid < NCOMM) {
        const unsigned xv = (unsigned)((wid & 1) * 128 + (wid >> 1) * 64 + (lane & 15)) * KS + (lane >> 4) * 16;
        const unsigned ov = (wid & 1) * HB + (wid >> 1) * 8192 + lane * 16;
        unsigned rawA[16], rawB[16], tick = 0; u32x4 xrA[2], xrB[2];
        unsigned u0 = (bid << 16) | a.order[bid >> 2], u1 = ((bid + NCOMM) << 16) | a.order[(bid + NCOMM) >> 2];
        unsigned u2 = ((bid + 2 * NCOMM) << 16) | a.order[(bid + 2 * NCOMM) >> 2], u3 = ((bid + 3 * NCOMM) << 16) | a.order[(bid + 3 * NCOMM) >> 2], pfl = 0;
#define COMM_LOAD(raw, xr, u) do { const unsigned _p = (u) & 255u, _k = ((u) >> 8) & 3u, _h = ((u) >> 16) & 1u, _f = ((u) >> 17) & 1u; \
            const bool _x = _p < 16; \
            const unsigned sox = _x ? (_p * 256u + _f * 32u) * KS + _k * 128u + _h * 64u : 0u, sow = _x ? 0u : (_k * 128u + _h * 64u) * N + (_p - 16u) * 256u; \
            const __amdgpu_buffer_rsrc_t rsx = __builtin_amdgcn_make_buffer_rsrc((void*)a.x, 0, _x ? (unsigned)M * KS : 0u, 0x00020000); \
            const __amdgpu_buffer_rsrc_t rsw = __builtin_amdgcn_make_buffer_rsrc((void*)a.w, 0, _x ? 0u : (unsigned)KS * N, 0x00020000); \
            _Pragma("unroll") for (int i = 0; i < 2; ++i) xr[i] = __builtin_amdgcn_raw_buffer_load_b128(rsx, xv + i * 16 * KS, sox, 0); \
            _Pragma("unroll") for (int i = 0; i < 16; ++i) raw[i] = __builtin_amdgcn_raw_buffer_load_b32(rsw, wv, sow + i * N, 0); } while (0)
        COMM_LOAD(rawA, xrA, u0); COMM_LOAD(rawB, xrB, u1);
        { const unsigned e0 = a.order[tid], e1 = a.order[256 + tid < NBLOCK ? 256 + tid : NBLOCK - 1];
          *(LAS unsigned*)(lds + tid * 4) = e0; *(LAS unsigned*)(lds + 1024 + tid * 4) = e1; }
        CALL_INIT();
        __amdgpu_buffer_rsrc_t rd[NR - 1];
#pragma unroll
        for (int i = 0; i < NR - 1; ++i) rd[i] = __builtin_amdgcn_make_buffer_rsrc(a.peer[(unsigned)(i + 2 * wid + bid) % (NR - 1)] + doff, 0, DATA_BYTES, 0x00020000);
        GAS unsigned* fl = (GAS unsigned*)(a.peer[lane < NR - 1 ? lane : 0] + WS_FLAGS) + rank * 4;
        GAS unsigned* const tmo = (GAS unsigned*)(myws + WS_TMO);
        char* const tickp = myws + WS_TICKET;
        const unsigned tbase = (call - 1u) * NUNIT - 4u * NCOMM, one = 1u;
        bool bad = false;
#define SPIN_UNTIL(cond) do { const unsigned long long _t0 = __builtin_amdgcn_s_memrealtime(); \
            while (__builtin_amdgcn_ballot_w64(!(cond)) != 0ull) { \
                if (__builtin_amdgcn_readfirstlane(__hip_atomic_load(tmo, __ATOMIC_RELAXED, __HIP_MEMORY_SCOPE_AGENT)) == call \
                    || __builtin_amdgcn_s_memrealtime() - _t0 > DEADLINE_TICKS) { \
                    __hip_atomic_store(tmo, call, __ATOMIC_RELAXED, __HIP_MEMORY_SCOPE_AGENT); bad = true; break; } \
                __builtin_amdgcn_s_sleep(8); } } while (0)
#define CT(i) do {} while (0)
#define CTRACE() do {} while (0)
#define COMM_ST(o, vo, so) _Pragma("unroll") for (int d = 0; d < NR - 1; ++d) __builtin_amdgcn_raw_buffer_store_b128(o, rd[d], vo, so, AUX_SYS)
#define COMM_STEP(raw, xr, first, word) do { const unsigned _p0 = u0 & 255u, _k0 = (u0 >> 8) & 3u, _q0 = (u0 >> 16) & 3u; \
            u32x4 o0, o1; \
            { const unsigned sel = (_q0 & 2u) ? 0x07030602u : 0x05010400u; \
              _Pragma("unroll") for (int b = 0; b < 4; ++b) { \
                    const unsigned t0 = __builtin_amdgcn_perm(raw[4 * b + 1], raw[4 * b], sel), t2 = __builtin_amdgcn_perm(raw[4 * b + 3], raw[4 * b + 2], sel); \
                    o0[b] = __builtin_amdgcn_perm(t2, t0, 0x05040100u) | xr[0][b]; o1[b] = __builtin_amdgcn_perm(t2, t0, 0x07060302u) | xr[1][b]; } } \
            asm volatile("" : "+v"(o0), "+v"(o1)); CT(0); \
            const unsigned sd = (_k0 * NR + rank) * SLAB + _p0 * BLK + (_q0 & 2u) * 2048u + (_q0 & 1u) * 1024u; \
            COMM_LOAD(raw, xr, u2); SCHED; CT(1); COMM_ST(o0, ov, sd); CT(2); WAIT_V(25); CT(3); \
            if (!(first) && tid == 0) { asm volatile("" : "+v"(tick)); const unsigned _t = tick - tbase; \
                *(volatile LAS unsigned*)(lds + LDS_MISC + (word)) = (_t << 16) | *(volatile LAS unsigned*)(lds + ((_t & 2047u) & ~3u)); } \
            WAIT_L(0); BAR; CT(4); \
            unsigned un = 0; \
            if (!(first)) { un = __builtin_amdgcn_readfirstlane(*(volatile LAS unsigned*)(lds + LDS_MISC + (word))); \
                if (tid < NR - 1) __hip_atomic_store(fl + pfl, call, __ATOMIC_RELAXED, __HIP_MEMORY_SCOPE_SYSTEM); } \
            else CALL_ADVANCE(); \
            if (tid == 0) asm volatile("global_atomic_add %0, %1, %2, off sc0" : "=&v"(tick) : "v"(tickp), "v"(one) : "memory"); \
            COMM_ST(o1, ov + 2048, sd); CT(5); \
            CTRACE(); \
            pfl = _p0 * 128u + _k0 * (NR * 4u) + _q0; \
            u0 = u1; u1 = u2; if (first) { u2 = u3; } else { u2 = un; } } while (0)
        COMM_STEP(rawA, xrA, true, 8);
        for (;;) {
            COMM_STEP(rawB, xrB, false, 12); if ((u0 >> 16) >= NUNIT) break;
            COMM_STEP(rawA, xrA, false, 8); if ((u0 >> 16) >= NUNIT) break;
        }
        WAIT_V(0); BAR;
        if (tid < NR - 1) __hip_atomic_store(fl + pfl, call, __ATOMIC_RELAXED, __HIP_MEMORY_SCOPE_SYSTEM);
        SPIN_UNTIL(__hip_atomic_load((GAS unsigned*)(myws + WS_TDONE) + (lane < NSHARE ? lane : 0), __ATOMIC_RELAXED, __HIP_MEMORY_SCOPE_AGENT) >= call);
        __builtin_amdgcn_fence(__ATOMIC_ACQUIRE, "agent");
        const __amdgpu_buffer_rsrc_t ro = __builtin_amdgcn_make_buffer_rsrc((void*)a.out, 0, (unsigned)M * N * 4u, 0x00020000);
        const unsigned fv = (unsigned)((bid * 4 + wid) * 4) * 65536u + lane * 16;
        u32x4 fw[NSHARE][4];
#pragma unroll
        for (int t = 0; t < NSHARE; ++t)
#pragma unroll
            for (int i = 0; i < 4; ++i) fw[t][i] = __builtin_amdgcn_raw_buffer_load_b128(ro, fv, a.shared[t] + i * 65536u, AUX_SYS);
        if (bad) fw[0][0] = (u32x4){0x7fc00000u, 0x7fc00000u, 0x7fc00000u, 0x7fc00000u};
#pragma unroll
        for (int i = 0; i < NR - 1; ++i) rd[i] = __builtin_amdgcn_make_buffer_rsrc(a.peer_out[(unsigned)(i + 2 * wid + bid) % (NR - 1)], 0, (unsigned)M * N * 4u, 0x00020000);
#pragma unroll
        for (int t = 0; t < NSHARE; ++t)
#pragma unroll
            for (int i = 0; i < 4; ++i) { COMM_ST(fw[t][i], fv, a.shared[t] + i * 65536u); }
        WAIT_V(0); BAR;
        if (wid == 0) {
            if (lane < NR - 1) __hip_atomic_store((GAS unsigned*)(a.peer[lane] + WS_FWD) + rank * NCOMM + bid, call, __ATOMIC_RELAXED, __HIP_MEMORY_SCOPE_SYSTEM);
            SPIN_UNTIL(__hip_atomic_load((GAS unsigned*)(myws + WS_FWD) + ((rank + 1u + (lane < NR - 1 ? lane : 0)) & (NR - 1)) * NCOMM + bid, __ATOMIC_RELAXED, __HIP_MEMORY_SCOPE_SYSTEM) >= call);
            if (bad && lane == 0) a.out[(a.shared[0] + fv) / 4] = __builtin_nanf("");
        }
        return;
    }

    const int gw = bid - NCOMM;
    const unsigned long long tl = a.tiles[gw];
    const int ntiles = (int)(tl >> 50) & 7;
    const unsigned koff0 = (unsigned)(tl >> 56) & 31u;
    const int wr = wid >> 1, wc = wid & 1, fr = lane & 15, fq = lane >> 4;
    constexpr int NLOC = 2 * NPANEL;
    const int lw = gw - (NWG - NCOMM - NLOC);
    unsigned raw[4][16];
    if (lw >= 0) {
        const unsigned h = lw & 1, p = lw >> 1;
        if (p < 16) {
            const unsigned xv = (unsigned)(wid * 64 + (lane & 15)) * KS + h * 64 + (lane >> 4) * 16;
#pragma unroll
            for (int i = 0; i < 4; ++i)
#pragma unroll
                for (int j = 0; j < 4; ++j) { const u32x4 v = __builtin_amdgcn_raw_buffer_load_b128(rx, xv + j * 16 * KS, p * (256u * KS) + i * 128u, 0);
                    raw[i][4 * j] = v[0]; raw[i][4 * j + 1] = v[1]; raw[i][4 * j + 2] = v[2]; raw[i][4 * j + 3] = v[3]; }
        } else {
#pragma unroll
            for (int i = 0; i < 4; ++i)
#pragma unroll
                for (int j = 0; j < 16; ++j) raw[i][j] = __builtin_amdgcn_raw_buffer_load_b32(rw, wv, (i * 128u + h * 64u + j) * N + (p - 16u) * 256u, 0);
        }
    }
    if (tid == 0) *(LAS unsigned*)(lds + LDS_MISC + 4) = 0u;
    CALL_INIT();
    const __amdgpu_buffer_rsrc_t rin = __builtin_amdgcn_make_buffer_rsrc(myws + doff, 0, DATA_BYTES, 0x00020000);
    const __amdgpu_buffer_rsrc_t rfl = __builtin_amdgcn_make_buffer_rsrc(myws + WS_FLAGS, 0, FLAG_BYTES, 0x00020000);
    GAS unsigned* tmo = (GAS unsigned*)(myws + WS_TMO);
    if (lw >= 0) {
        const unsigned h = lw & 1, p = lw >> 1;
#pragma unroll
        for (int i = 0; i < 4; ++i) {
            const unsigned so = (i * NR + rank) * SLAB + p * BLK + h * 1024;
            if (p < 16) {
#pragma unroll
                for (int j = 0; j < 4; ++j) __builtin_amdgcn_raw_buffer_store_b128((u32x4){raw[i][4 * j], raw[i][4 * j + 1], raw[i][4 * j + 2], raw[i][4 * j + 3]}, rin, lane * 16, so + (wid * 4 + j) * 2048, AUX_SYS);
            } else {
                u32x4 o[4];
#pragma unroll
                for (int b = 0; b < 4; ++b) {
                    const unsigned r0 = raw[i][4 * b], r1 = raw[i][4 * b + 1], r2 = raw[i][4 * b + 2], r3 = raw[i][4 * b + 3];
                    const unsigned t0 = __builtin_amdgcn_perm(r1, r0, 0x05010400), t1 = __builtin_amdgcn_perm(r1, r0, 0x07030602);
                    const unsigned t2 = __builtin_amdgcn_perm(r3, r2, 0x05010400), t3 = __builtin_amdgcn_perm(r3, r2, 0x07030602);
                    o[0][b] = __builtin_amdgcn_perm(t2, t0, 0x05040100); o[1][b] = __builtin_amdgcn_perm(t2, t0, 0x07060302);
                    o[2][b] = __builtin_amdgcn_perm(t3, t1, 0x05040100); o[3][b] = __builtin_amdgcn_perm(t3, t1, 0x07060302);
                }
#pragma unroll
                for (int c = 0; c < 4; ++c) __builtin_amdgcn_raw_buffer_store_b128(o[c], rin, lane * 16, so + (wid & 1) * HB + (wid >> 1) * 8192 + c * 2048, AUX_SYS);
            }
        }
    }
    asm volatile("s_waitcnt vmcnt(0) lgkmcnt(0)" ::: "memory"); BAR;
    CALL_ADVANCE();
    if (lw >= 0 && tid < 8)
        __hip_atomic_store((GAS unsigned*)(myws + WS_FLAGS) + (lw >> 1) * 128u + ((tid >> 1) * NR + rank) * 4u + (tid & 1) * 2 + (lw & 1), call, __ATOMIC_RELAXED, __HIP_MEMORY_SCOPE_SYSTEM);
    const float scale = __builtin_bit_cast(float, __builtin_amdgcn_readfirstlane(__builtin_bit_cast(int, a.sx[0] * a.sw[0])));
    const unsigned voff = wid * 4096 + lane * 16, ldsw = wid * 4096;
    const unsigned ldsb = (unsigned)(unsigned long)lds;
    unsigned aoff = ldsb + wr * 8192 + lane * 16, boff = ldsb + 4 * HB + wc * 8192 + lane * 16;
    asm volatile("" : "+v"(aoff), "+v"(boff));
    const unsigned ovoff = (unsigned)(wr * 64 + 4 * fq) * 65536u + wc * 512 + fr * 16;
#define TILE(i) ((unsigned)(tl >> (10 * (i))) & 0x3ffu)
#define SA(b, h) (((b) * 2 + (h)) * HB)
#define SB(b, h) ((4 + (b) * 2 + (h)) * HB)
#define G1(bufoff, soff, i) __builtin_amdgcn_raw_ptr_buffer_load_lds(rin, (LAS void*)(lds + (bufoff) + ldsw), 16, voff, (soff), (i) * 1024, AUX_SYS)
#define STAGE4(bufoff, soff) do { G1(bufoff, soff, 0); G1(bufoff, soff, 1); G1(bufoff, soff, 2); G1(bufoff, soff, 3); } while (0)
#define RD(dst, base, off) asm volatile("ds_read_b128 %0, %1 offset:%c2" : "+v"(dst) : "v"(base), "i"(off) : "memory")
#define CAT(lo, hi) __builtin_shufflevector(lo, hi, 0, 1, 2, 3, 4, 5, 6, 7)
#define MF(Z, ai, bj, m, n, RA, RB) mfma<acc_idx(ai, bj, m, n), Z>(CAT(RA[2 * (m)], RA[2 * (m) + 1]), CAT(RB[2 * (n)], RB[2 * (n) + 1]))
#define EPG(eq, g) do { if constexpr ((eq) >= 0) { if constexpr ((((g) / 3) & 1) == 0) epi_gap<(eq) < 0 ? 0 : (eq), (g)>(ev0, ev1, esc, erout, ovoff, eobase); \
                                                   else epi_gap<(eq) < 0 ? 0 : (eq), (g)>(ev1, ev0, esc, erout, ovoff, eobase); } } while (0)
#define QUAD(Z, ai, bj, RA, RB, RN, rdb, rdoff, stoff, soff, vm, eq, g0, hook) do { u32x4 _fv; unsigned _pi; \
        if constexpr ((vm) >= 0) { asm volatile("s_waitcnt vmcnt(%0) lgkmcnt(0)" :: "n"((vm) < 0 ? 0 : (vm)) : "memory"); BAR; } else WAIT_L(0); SCHED; \
        MF(Z, ai, bj, 0, 0, RA, RB); RD(RN[0], rdb, (rdoff)); EPG(eq, g0 + 0); SCHED; \
        MF(Z, ai, bj, 0, 1, RA, RB); RD(RN[1], rdb, (rdoff) + 1024); EPG(eq, g0 + 1); SCHED; \
        MF(Z, ai, bj, 0, 2, RA, RB); RD(RN[2], rdb, (rdoff) + 2048); EPG(eq, g0 + 2); if constexpr ((hook) >= 3) PICK_A(_pi); SCHED; \
        MF(Z, ai, bj, 0, 3, RA, RB); RD(RN[3], rdb, (rdoff) + 3072); EPG(eq, g0 + 3); SCHED; \
        MF(Z, ai, bj, 1, 0, RA, RB); RD(RN[4], rdb, (rdoff) + 4096); EPG(eq, g0 + 4); if constexpr ((hook) == 3) PICK_B(_pi, kaA, kaB); if constexpr ((hook) == 4) PICK_B(_pi, kbA, kbB); SCHED; \
        MF(Z, ai, bj, 1, 1, RA, RB); RD(RN[5], rdb, (rdoff) + 5120); EPG(eq, g0 + 5); SCHED; \
        MF(Z, ai, bj, 1, 2, RA, RB); RD(RN[6], rdb, (rdoff) + 6144); EPG(eq, g0 + 6); SCHED; \
        MF(Z, ai, bj, 1, 3, RA, RB); RD(RN[7], rdb, (rdoff) + 7168); EPG(eq, g0 + 7); SCHED; \
        MF(Z, ai, bj, 2, 0, RA, RB); G1(stoff, soff, 0); EPG(eq, g0 + 8); SCHED; \
        MF(Z, ai, bj, 2, 1, RA, RB); EPG(eq, g0 + 9); if constexpr ((hook) == 2) _fv = FLAG_LOAD(0); SCHED; \
        MF(Z, ai, bj, 2, 2, RA, RB); G1(stoff, soff, 1); EPG(eq, g0 + 10); SCHED; \
        MF(Z, ai, bj, 2, 3, RA, RB); EPG(eq, g0 + 11); SCHED; \
        MF(Z, ai, bj, 3, 0, RA, RB); G1(stoff, soff, 2); EPG(eq, g0 + 12); SCHED; \
        MF(Z, ai, bj, 3, 1, RA, RB); EPG(eq, g0 + 13); if constexpr ((hook) == 2) FLAG_MASK(_fv); SCHED; \
        MF(Z, ai, bj, 3, 2, RA, RB); G1(stoff, soff, 3); EPG(eq, g0 + 14); SCHED; \
        MF(Z, ai, bj, 3, 3, RA, RB); EPG(eq, g0 + 15); if constexpr ((hook) == 1) FLAG_FETCH(0); SCHED; } while (0)

    unsigned rmask = 0, done = 0, kaA, kaB, kbA, kbB;
#define BASE_A(tile) (((tile) & 15u) * BLK)
#define BASE_B(tile) ((16u + ((tile) >> 4)) * BLK)
#define FLAG_SOFF(tile) ((wid < 2 ? ((tile) & 15u) : 16u + ((tile) >> 4)) * 512u + (wid & 1) * 256u)
    unsigned pA = BASE_A(TILE(0)), pB = BASE_B(TILE(0)), psoff = FLAG_SOFF(TILE(0)), fsoff = psoff;
    const unsigned fvoff = lane * 4;
    int froff = LDS_FLAGS + ((lane & 32) + ((lane + koff0) & 31)) * 16; asm volatile("" : "+v"(froff));
#define FLAG_FETCH_AT(slot, so) __builtin_amdgcn_raw_ptr_buffer_load_lds(rfl, (LAS void*)(lds + LDS_FLAGS + (slot) * 1024 + wid * 256), 4, fvoff, so, 0, AUX_SYS)
#define FLAG_FETCH(slot) FLAG_FETCH_AT(slot, fsoff)
#define FLAG_LOAD(slot) (*(volatile LAS u32x4*)(lds + froff + (slot) * 1024))
#define FLAG_MASK(_v) do { const unsigned _a = _v[0] < _v[1] ? _v[0] : _v[1], _b = _v[2] < _v[3] ? _v[2] : _v[3]; \
            const unsigned long long _m = __builtin_amdgcn_ballot_w64((_a < _b ? _a : _b) >= call); \
            rmask |= (unsigned)_m & (unsigned)(_m >> 32); } while (0)
#define TWAIT(t0) do {} while (0)
#define PICK_A(_i) do { \
            if (__builtin_expect((rmask & ~done) == 0u, 0)) {        \
                const unsigned long long _t0 = __builtin_amdgcn_s_memrealtime(); \
                for (;;) { \
                    FLAG_FETCH_AT(1, psoff); \
                    if (wid == 0 && __builtin_amdgcn_s_memrealtime() - _t0 > 10000ull        \
                        && (__builtin_amdgcn_readfirstlane(__hip_atomic_load(tmo, __ATOMIC_RELAXED, __HIP_MEMORY_SCOPE_AGENT)) == call \
                            || __builtin_amdgcn_s_memrealtime() - _t0 > DEADLINE_TICKS)) { \
                        __hip_atomic_store(tmo, call, __ATOMIC_RELAXED, __HIP_MEMORY_SCOPE_AGENT); \
                        *(volatile LAS unsigned*)(lds + LDS_MISC + 4) = 1u; } \
                    asm volatile("s_waitcnt vmcnt(0) lgkmcnt(0)" ::: "memory"); BAR; \
                    const u32x4 _pv = FLAG_LOAD(1); FLAG_MASK(_pv); \
                    if (__builtin_amdgcn_readfirstlane(*(volatile LAS unsigned*)(lds + LDS_MISC + 4)) != 0u) rmask = ~0u; \
                    WAIT_L(0); BAR; \
                    if ((rmask & ~done) != 0u) break; \
                    __builtin_amdgcn_s_sleep(POLL_SLEEP); \
                } \
                TWAIT(_t0); \
            } \
            _i = __builtin_ctz(rmask & ~done); done |= 1u << _i; } while (0)
#define PICK_B(_i, oA, oB) do { const unsigned _q = ((_i + koff0) & 31u) * SLAB; oA = _q + pA; oB = _q + pB; asm volatile("" : "+s"(oA), "+s"(oB)); } while (0)
#define PICK(oA, oB) do { unsigned _pi; PICK_A(_pi); PICK_B(_pi, oA, oB); } while (0)
#define ITER(KIND, sw, S1, S2, S3, S4) do { \
        if (sw) fsoff = nsoff; \
        QUAD(KIND == 0, 0, 0, S1, B0a, B0b, boff, SA(1, 0), SA(1, 0), kbA, (KIND == 0 ? 38 : KIND == 3 ? 27 : 16), (KIND == 0 ? 3 : -1), 0, 4); \
        QUAD(false, 0, 0, S2, B0b, S3, boff, SA(0, 1), SB(0, 0), kaB, -1, (KIND == 0 ? 3 : -1), 16, 1); \
        if constexpr (KIND == 2) { \
            esc = *(volatile LAS unsigned*)(lds + LDS_MISC + 4) != 0u ? __builtin_nanf("") : scale; \
            esc = __builtin_bit_cast(float, __builtin_amdgcn_readfirstlane(__builtin_bit_cast(int, esc))); } \
        QUAD(KIND == 0, 0, 1, S1, S3, S4, boff, SA(1, 1), SB(1, 0), kbB, (KIND == 0 ? 38 : 17), (KIND == 0 ? 3 : KIND == 2 ? 0 : -1), (KIND == 0 ? 32 : 0), 0); \
        QUAD(false, 0, 1, S2, S4, S1, aoff, SA(0, 1), SB(0, 1), kaB + HB, -1, (KIND == 0 ? 2 : KIND == 2 ? 0 : -1), (KIND == 0 ? 0 : 16), 0); \
        QUAD(KIND == 0, 1, 1, S1, S3, S2, aoff, SA(1, 1), SB(1, 1), kbB + HB, (KIND == 0 ? 38 : KIND == 2 ? 27 : 17), (KIND == 0 ? 2 : KIND == 2 ? 0 : -1), (KIND == 0 ? 16 : 32), 0); \
        QUAD(false, 1, 1, S2, S4, S3, aoff, SA(0, 0), SA(0, 1), kaA + HB, -1, (KIND == 0 ? 2 : KIND == 2 ? 1 : -1), (KIND == 0 ? 32 : 0), 0); \
        if (sw) { done = 0; rmask = nmask; pA = nA; pB = nB; psoff = nsoff; } \
        QUAD(KIND == 0, 1, 0, S1, B0a, S4, aoff, SA(1, 0), SA(1, 1), kbA + HB, (KIND == 0 ? 38 : KIND == 2 ? 37 : 16), (KIND == 2 ? 1 : -1), 16, 2); \
        QUAD(false, 1, 0, S2, B0b, B0a, boff, SA(0, 0), SA(0, 0), kaA, -1, (KIND == 2 ? 1 : -1), 32, 3); } while (0)

#define STAMP(ui, it) do {} while (0)
    i32x4 R1[8], R2[8], R3[8], R4[8], B0a[8], B0b[8];
    PICK(kaA, kaB);
    STAGE4(SA(0, 0), kaA); STAGE4(SB(0, 0), kaB); STAGE4(SB(0, 1), kaB + HB); STAGE4(SA(0, 1), kaA + HB);
    PICK(kbA, kbB);
    STAGE4(SA(1, 0), kbA); STAGE4(SB(1, 0), kbB); STAGE4(SB(1, 1), kbB + HB); STAGE4(SA(1, 1), kbA + HB);
    WAIT_V(12); BAR;
#pragma unroll
    for (int i = 0; i < 8; ++i) asm volatile("" : "=v"(R1[i]), "=v"(R2[i]), "=v"(R3[i]), "=v"(R4[i]), "=v"(B0a[i]), "=v"(B0b[i]));
#define RD8(U, base, off) RD(U[0], base, off); RD(U[1], base, off + 1024); RD(U[2], base, off + 2048); RD(U[3], base, off + 3072); \
                          RD(U[4], base, off + 4096); RD(U[5], base, off + 5120); RD(U[6], base, off + 6144); RD(U[7], base, off + 7168)
    RD8(R1, aoff, SA(0, 0)); RD8(B0a, boff, SA(0, 0)); WAIT_L(0); RD8(R2, aoff, SA(1, 0));
    WAIT_L(0); BAR;
    PICK(kaA, kaB);
    STAGE4(SA(0, 0), kaA);
    WAIT_V(0);
    __amdgpu_buffer_rsrc_t erout = __builtin_amdgcn_make_buffer_rsrc((void*)a.out, 0, 0, 0x00020000);
    unsigned eobase = 0; float esc = 0.f;
    Ev ev0 = {0.f, 0.f, 0.f, 0.f}, ev1 = {0.f, 0.f, 0.f, 0.f};
    int ui = 0;
    do {
        const bool more = ui + 1 < ntiles; const unsigned nt = TILE(more ? ui + 1 : ui);
        const unsigned nA = BASE_A(nt), nB = BASE_B(nt), nsoff = FLAG_SOFF(nt), nmask = more ? 0u : ~0u;
        STAMP(ui, 0);
        ITER(0, false, R1, R2, R3, R4);
        STAMP(ui, 1);
        ITER(3, false, R3, R4, R1, R2);
        if (__builtin_expect(ui == (int)((tl >> 53) & 7u) && ui > 0, 0)) {
            WAIT_V(0); BAR;
            if (tid == 0) {
                __builtin_amdgcn_fence(__ATOMIC_RELEASE, "agent"); WAIT_V(0);
                __hip_atomic_store((GAS unsigned*)(myws + WS_TDONE) + (unsigned)(tl >> 61), call, __ATOMIC_RELAXED, __HIP_MEMORY_SCOPE_AGENT);
            }
        }
        for (int it = 2; it < 14; it += 2) { STAMP(ui, it); ITER(1, false, R1, R2, R3, R4); STAMP(ui, it + 1); ITER(1, false, R3, R4, R1, R2); }
        STAMP(ui, 14);
        ITER(1, true, R1, R2, R3, R4);
        erout = __builtin_amdgcn_make_buffer_rsrc((void*)a.out, 0, (unsigned)M * N * 4u, 0x00020000);
        eobase = (TILE(ui) & 15u) * (256u * 65536u) + (TILE(ui) >> 4) * 1024u;
        STAMP(ui, 15);
        ITER(2, false, R3, R4, R1, R2);
    } while (++ui < ntiles);
    STAMP(ntiles, 0);
#define EP3(q, g) EPG(q, g); EPG(q, g + 1); EPG(q, g + 2)
#define EP12(q, g) EP3(q, g); EP3(q, g + 3); EP3(q, g + 6); EP3(q, g + 9)
    EP12(3, 0); EP12(3, 12); EP12(3, 24); EP12(3, 36); EP12(2, 0); EP12(2, 12); EP12(2, 24); EP12(2, 36);
    WAIT_V(0);
}

#define P_KS0 2.0
#define P_SX -3.0
#define P_RX 1.18
#define P_SPS 8.0
#define P_ZOFF -4.0
#define P_KZ 28
#define P_RH 1
#define P_LEN2 38.0
#define P_S3 44.0
#define P_G1X -2.0
void make_tables(Args& a, int rank) {
    double key[320]; int n = 0;
    constexpr int PZ = 15 - NZ;
    auto koff = [](int p) { return p < 4 ? 0 : p < PZ ? (int)(P_KS0 + (p - 4) * P_SPS / (PZ - 5) / 1.25 + 0.5) : P_KZ; };
    auto phase = [](int p) { return p < PZ ? p * P_SPS / (PZ - 5) : P_ZOFF; };
    auto add = [&](int panel, double start, double rate, int rot, int off) {
        for (int kt = 0; kt < 4; ++kt) {
            const double k = start + ((kt * NR + ((rank + rot) & (NR - 1)) - off) & 31) * rate + panel * 1e-6;
            int i = n++;
            for (; i > 0 && key[i - 1] > k; --i) { key[i] = key[i - 1]; a.order[i] = a.order[i - 1]; }
            key[i] = k; a.order[i] = (unsigned)((kt << 8) | panel);
        }
    };
    for (int i = 0; i < 8; ++i) { add(i, -1000.0 + P_G1X, 1.0, i, 0); add(16 + i, -1000.0, 1.0, i, 0); add(8 + i, P_SX, P_RX, i, 0); }
    for (int p = 4; p < PZ; ++p) add(16 + 8 + (p - 4), phase(p) - phase(4), P_LEN2 / 32.0, p, koff(p));
    for (int g = 0; g < 3; ++g)
        for (int p = 0; p < 15; ++p) add(16 + 19 + g * 15 + p, P_S3 + g * 38.0 + phase(p), 1.18, p, koff(p));
    for (int i = 0; i < NZ; ++i) add(16 + 19 - NZ + i, P_S3 + P_RH * 38.0 + phase(PZ), 1.18, 4 * i + 1, koff(PZ));
    const int H = 19 + P_RH * 15 + PZ;
    int smp[NSHARE], snp[NSHARE];
    for (int i = 0; i < 2 * NZ; ++i) { smp[i] = (2 * NZ * rank + i) & 15; snp[i] = 19 - NZ + ((2 * NZ * rank) >> 4); }
    smp[2 * NZ] = rank; snp[2 * NZ] = H;
    for (int i = 0; i < NSHARE; ++i) a.shared[i] = (unsigned)smp[i] * (256u * 65536u) + (unsigned)snp[i] * 1024u;
    for (int g = 0; g < NWG - NCOMM; ++g) {
        const int xcd = g & 7, r = (xcd >> 2) * 30 + (g >> 3), p = (2 * (r >> 2)) % 15, m = (xcd & 3) * 4 + (r & 3);
        int mp[5], np[5], nt = 0; unsigned long long tl = 0;
        if (p < 4) { mp[0] = m & 7; mp[1] = 8 + (m & 7); np[0] = np[1] = 2 * p + (m >> 3); nt = 2; }
        else if (p < PZ) { mp[0] = m; np[0] = 8 + p - 4; nt = 1; }
        for (int k = 0; k < 3; ++k) {
            if (19 + k * 15 + p == H && m < 8) {
                if (m < NSHARE) { mp[nt] = smp[m]; np[nt] = snp[m]; ++nt; tl |= ((unsigned long long)nt << 53) | ((unsigned long long)m << 61); }
            } else { mp[nt] = m; np[nt] = 19 + k * 15 + p; ++nt; }
        }
        tl |= ((unsigned long long)nt << 50) | ((unsigned long long)koff(p) << 56);
        for (int i = 0; i < nt; ++i) tl |= (unsigned long long)((np[i] << 4) | mp[i]) << (10 * i);
        a.tiles[g] = tl;
    }
}
}

extern "C" void kernel_launch(int rank, int nranks, ncclComm_t comm,
                              hipStream_t stream,
                              void* const* d_in, const int* in_sizes, int n_in,
                              void* d_out, int out_size,
                              const int* in_dtypes, int out_dtype,
                              void* const* d_out_all,
                              void* const* d_ws_all, size_t ws_bytes) {
    (void)comm; (void)in_dtypes; (void)out_dtype;
    if (nranks != NR || (unsigned)rank >= (unsigned)NR || n_in != 4 || in_sizes[0] != M * KS || in_sizes[1] != KS * N ||
        out_size != M * N || ws_bytes < WS_NEED) {
        fprintf(stderr, "kernel_launch: unexpected shapes\n");
        return;
    }
    Args a;
    a.x = (const unsigned char*)d_in[0]; a.w = (const unsigned char*)d_in[1];
    a.sx = (const float*)d_in[2]; a.sw = (const float*)d_in[3];
    a.out = (float*)d_out;
    a.myws = (char*)d_ws_all[rank];
    for (int i = 0; i < NR - 1; ++i) { a.peer[i] = (char*)d_ws_all[(rank + 1 + i) % NR]; a.peer_out[i] = (char*)d_out_all[(rank + 1 + i) % NR]; }
    a.rank = rank;
    make_tables(a, rank);
    hipLaunchKernelGGL(fused_gemm, dim3(NWG), dim3(256), 0, stream, a);
    HIP_CHECK(hipGetLastError());
}
```

```cpp
#include <hip/hip_runtime.h>
#include <rccl/rccl.h>
#include <cstdio>

#define HIP_CHECK(expr) do { hipError_t _e = (expr); if (_e != hipSuccess) { \
    fprintf(stderr, "HIP error %s at %s:%d\n", hipGetErrorString(_e), __FILE__, __LINE__); \
    return; } } while (0)

#define LAS __attribute__((address_space(3)))
#define GAS __attribute__((address_space(1)))
typedef int i32x4 __attribute__((ext_vector_type(4)));
typedef unsigned u32x4 __attribute__((ext_vector_type(4)));
typedef int i32x8 __attribute__((ext_vector_type(8)));
typedef float f32x4 __attribute__((ext_vector_type(4)));

namespace {
constexpr int NR = 8;
constexpr int M = 4096, N = 16384, KS = 512;
constexpr unsigned BLK = 32768, HB = 16384;
constexpr unsigned NPANEL = 80;
constexpr unsigned SLAB = NPANEL * BLK;
constexpr unsigned DATA_BYTES = 32 * SLAB;
constexpr unsigned WS_FLAGS = 0;
constexpr unsigned FLAG_BYTES = NPANEL * 512;
constexpr unsigned WS_CTR = 65536;
constexpr unsigned WS_TMO = WS_CTR + 256 * 128;
constexpr unsigned WS_TICKET = WS_TMO + 128;
constexpr unsigned WS_TDONE = WS_TICKET + 128;
#define NZ 2
constexpr int NSHARE = 2 * NZ + 1;
constexpr unsigned WS_RETILE = WS_TDONE + 128;
constexpr unsigned WS_FWD = WS_RETILE + 128;
constexpr unsigned NBLOCK = 320, NUNIT = 4 * NBLOCK;
constexpr size_t WS_DATA = 1u << 20;
constexpr size_t WS_NEED = WS_DATA + 2 * (size_t)DATA_BYTES;
constexpr int NCOMM = 16, NWG = 256;
constexpr unsigned long long DEADLINE_TICKS = 50000000ull;
constexpr int AUX_SYS = 17;
#define RAW_UNITS 64u
#define RAW_STEP 8u
#define POLL_SLEEP 8
constexpr unsigned LDS_FLAGS = 8 * HB;
constexpr unsigned LDS_MISC = LDS_FLAGS + 2 * 1024;
constexpr unsigned LDS_BYTES = LDS_MISC + 64;

struct Args {
    const unsigned char* x; const unsigned char* w; const float* sx; const float* sw; float* out;
    char* myws; char* peer[NR - 1];
    char* peer_out[NR - 1];
    int rank;
    unsigned shared[NSHARE];
    unsigned order[NBLOCK];
    unsigned long long tiles[NWG - NCOMM];
};

#define WAIT_V(n) asm volatile("s_waitcnt vmcnt(" #n ")" ::: "memory")
#define WAIT_L(n) asm volatile("s_waitcnt lgkmcnt(" #n ")" ::: "memory")
#define BAR __builtin_amdgcn_s_barrier()
#define SCHED __builtin_amdgcn_sched_barrier(0)
#define ST_PAD() do { SCHED; asm volatile("s_nop 1"); SCHED; } while (0)

#define RCLOB "a0","a1","a2","a3","a4","a5","a6","a7","a8","a9","a10","a11","a12","a13","a14","a15","a16","a17","a18","a19","a20","a21","a22","a23","a24","a25","a26","a27","a28","a29","a30","a31","a32","a33","a34","a35","a36","a37","a38","a39","a40","a41","a42","a43","a44","a45","a46","a47","a48","a49","a50","a51","a52","a53","a54","a55","a56","a57","a58","a59","a60","a61","a62","a63","a64","a65","a66","a67","a68","a69","a70","a71","a72","a73","a74","a75","a76","a77","a78","a79","a80","a81","a82","a83","a84","a85","a86","a87","a88","a89","a90","a91","a92","a93","a94","a95","a96","a97","a98","a99","a100","a101","a102","a103","a104","a105","a106","a107","a108","a109","a110","a111","a112","a113","a114","a115","a116","a117","a118","a119","a120","a121","a122","a123","a124","a125","a126","a127","a128","a129","a130","a131","a132","a133","a134","a135","a136","a137","a138","a139","a140","a141","a142","a143","a144","a145","a146","a147","a148","a149","a150","a151","a152","a153","a154","a155","a156","a157","a158","a159","a160","a161","a162","a163","a164","a165","a166","a167","a168","a169","a170","a171","a172","a173","a174","a175","a176","a177","a178","a179","a180","a181","a182","a183","a184","a185","a186","a187","a188","a189","a190","a191","a192","a193","a194","a195","a196","a197","a198","a199","a200","a201","a202","a203","a204","a205","a206","a207","a208","a209","a210","a211","a212","a213","a214","a215","a216","a217","a218","a219","a220","a221","a222","a223","a224","a225","a226","a227","a228","a229","a230","a231","a232","a233","a234","a235","a236","a237","a238","a239","a240","a241","a242","a243","a244","a245","a246","a247","a248","a249","a250","a251","a252","a253","a254","a255","v32","v33","v34","v35","v36","v37","v38","v39","v40","v41","v42","v43","v44","v45","v46","v47","v48","v49","v50","v51","v52","v53","v54","v55","v56","v57","v58","v59","v60","v61","v62","v63","v64","v65","v66","v67","v68","v69","v70","v71","v72","v73","v74","v75","v76","v77","v78","v79","v80","v81","v82","v83","v84","v85","v86","v87","v88","v89","v90","v91","v92","v93","v94","v95","v96","v97","v98","v99","v100","v101","v102","v103","v104","v105","v106","v107","v108","v109","v110","v111","v112","v113","v114","v115","v116","v117","v118","v119","v120","v121","v122","v123","v124","v125","v126","v127","v128","v129","v130","v131","v132","v133","v134","v135","v136","v137","v138","v139","v140","v141","v142","v143","v144","v145","v146","v147","v148","v149","v150","v151","v152","v153","v154","v155","v156","v157","v158","v159","v160","v161","v162","v163","v164","v165","v166","v167","v168","v169","v170","v171","v172","v173","v174","v175","v176","v177","v178","v179","v180","v181","v182","v183","v184","v185","v186","v187","v188","v189","v190","v191","v192","v193","v194","v195","v196","v197","v198","v199","v200","v201","v202","v203","v204","v205","v206","v207","v208","v209","v210","v211","v212","v213","v214","v215","v216","v217","v218","v219","v220","v221","v222","v223","v224","v225","v226","v227","v228","v229","v230","v231","v232","v233","v234","v235","v236","v237","v238","v239","v240","v241","v242","v243","v244","v245","v246","v247","v248","v249","v250","v251","v252","v253","v254","v255"
constexpr int V_EV = 32, V_R1 = 64, V_R2 = 96, V_R3 = 128, V_R4 = 160, V_B0A = 192, V_B0B = 224;
constexpr int acc_idx(int ai, int bj, int m, int n) { return (((ai * 2 + bj) * 4 + m) * 4 + n) * 4; }
template <int IDX, bool ZERO, int X, int Y> __device__ __forceinline__ void mfma() {
    if constexpr ((IDX & 63) == 0) {
        if constexpr (ZERO) asm volatile("v_mfma_f32_16x16x128_f8f6f4 a[%c0:%c1], v[%c2:%c3], v[%c4:%c5], 0 cbsz:1 blgp:1" :: "i"(IDX), "i"(IDX + 3), "i"(X), "i"(X + 7), "i"(Y), "i"(Y + 7) : RCLOB);
        else asm volatile("v_mfma_f32_16x16x128_f8f6f4 a[%c0:%c1], v[%c2:%c3], v[%c4:%c5], a[%c0:%c1] cbsz:1 blgp:1" :: "i"(IDX), "i"(IDX + 3), "i"(X), "i"(X + 7), "i"(Y), "i"(Y + 7) : RCLOB);
    } else {
        if constexpr (ZERO) asm volatile("v_mfma_f32_16x16x128_f8f6f4 a[%c0:%c1], v[%c2:%c3], v[%c4:%c5], 0 cbsz:1 blgp:1" :: "i"(IDX), "i"(IDX + 3), "i"(X), "i"(X + 7), "i"(Y), "i"(Y + 7));
        else asm volatile("v_mfma_f32_16x16x128_f8f6f4 a[%c0:%c1], v[%c2:%c3], v[%c4:%c5], a[%c0:%c1] cbsz:1 blgp:1" :: "i"(IDX), "i"(IDX + 3), "i"(X), "i"(X + 7), "i"(Y), "i"(Y + 7));
    }
}
template <int Q, int G> __device__ __forceinline__ void epi_gap(float sc, __amdgpu_buffer_rsrc_t ro, unsigned ovoff, unsigned obase) {
    constexpr int E = G / 3, part = G % 3, m = E >> 2, j = E & 3, ai = Q >> 1, bj = Q & 1, V = V_EV + (E & 1) * 4;
    if constexpr (part == 0)
        asm volatile("v_accvgpr_read_b32 v%c1, a%c5\n\tv_accvgpr_read_b32 v%c2, a%c6\n\tv_accvgpr_read_b32 v%c3, a%c7\n\tv_accvgpr_read_b32 v%c4, a%c8\n\t"
                     "v_mul_f32 v%c1, %0, v%c1\n\tv_mul_f32 v%c2, %0, v%c2"
                     :: "s"(sc), "i"(V), "i"(V + 1), "i"(V + 2), "i"(V + 3),
                        "i"(acc_idx(ai, bj, m, 0) + j), "i"(acc_idx(ai, bj, m, 1) + j), "i"(acc_idx(ai, bj, m, 2) + j), "i"(acc_idx(ai, bj, m, 3) + j));
    else if constexpr (part == 1)
        asm volatile("v_mul_f32 v%c4, %0, v%c4\n\tv_mul_f32 v%c5, %0, v%c5\n\tv_maximum3_f32 v%c2, v%c2, 0, 0\n\tv_maximum3_f32 v%c3, v%c3, 0, 0\n\t"
                     "v_maximum3_f32 v%c4, v%c4, 0, 0\n\tv_maximum3_f32 v%c5, v%c5, 0, 0"
                     :: "s"(sc), "i"(0), "i"(V), "i"(V + 1), "i"(V + 2), "i"(V + 3));
    else
        asm volatile("buffer_store_dwordx4 v[%c3:%c4], %0, %1, %2 offen offset:%c5 nt"
                     :: "v"(ovoff), "s"(ro), "s"(obase + (unsigned)(ai * 128 + m * 16 + j) * 65536u), "i"(V), "i"(V + 3), "i"(bj * 512) : "memory");
}

__global__ __launch_bounds__(256, 1) void fused_gemm(Args a) {
    __shared__ __attribute__((aligned(16))) char shm[LDS_BYTES];
    LAS char* lds = (LAS char*)shm;
    const int wid = __builtin_amdgcn_readfirstlane(threadIdx.x >> 6);
#define LANE_INIT() int tid = threadIdx.x; asm volatile("" : "+v"(tid)); const int lane = tid & 63; \
    const unsigned wv = (unsigned)((lane >> 4) * 16) * N + wid * 64 + 4 * (lane & 15)
    const int bid = blockIdx.x;
    const unsigned rank = (unsigned)a.rank & (NR - 1);
    char* const myws = a.myws;
    GAS unsigned* const ctr = (GAS unsigned*)(myws + WS_CTR + bid * 128);
#define CALL_LOAD() asm volatile("" ::: "memory"); const unsigned ctr_old = __hip_atomic_load(ctr, __ATOMIC_RELAXED, __HIP_MEMORY_SCOPE_AGENT)
#define CALL_INIT() const unsigned call = __builtin_amdgcn_readfirstlane(ctr_old) + 1u; const size_t doff = WS_DATA + (size_t)(call & 1u) * DATA_BYTES
#define CALL_ADVANCE() do { if (tid == 0) __hip_atomic_store(ctr, call, __ATOMIC_RELAXED, __HIP_MEMORY_SCOPE_AGENT); } while (0)
    const __amdgpu_buffer_rsrc_t rx = __builtin_amdgcn_make_buffer_rsrc((void*)a.x, 0, (unsigned)M * KS, 0x00020000);
    const __amdgpu_buffer_rsrc_t rw = __builtin_amdgcn_make_buffer_rsrc((void*)a.w, 0, (unsigned)KS * N, 0x00020000);
    if (bid < NCOMM) {
        LANE_INIT(); CALL_LOAD();
        const unsigned xv = (unsigned)(wid * 64 + (lane & 15)) * KS + (lane >> 4) * 16;
        const unsigned ov = wid * 8192 + lane * 16;
        unsigned rawA[16], rawB[16]; u32x4 xrA[2], xrB[2];
        unsigned u0 = (bid << 16) | a.order[bid >> 2], u1 = ((bid + NCOMM) << 16) | a.order[(bid + NCOMM) >> 2];
        unsigned u2 = ((bid + 2 * NCOMM) << 16) | a.order[(bid + 2 * NCOMM) >> 2], u3 = ((bid + 3 * NCOMM) << 16) | a.order[(bid + 3 * NCOMM) >> 2], pfl = 0;
#define COMM_LOAD(raw, xr, u) do { const unsigned _p = (u) & 255u, _k = ((u) >> 8) & 3u, _h = ((u) >> 16) & 1u, _f = ((u) >> 17) & 1u; \
            const bool _x = _p < 16; \
            const unsigned sox = _x ? (_p * 256u + _f * 32u) * KS + _k * 128u + _h * 64u : 0u, sow = _x ? 0u : (_k * 128u + _h * 64u) * N + (_p - 16u) * 256u; \
            const __amdgpu_buffer_rsrc_t rsx = __builtin_amdgcn_make_buffer_rsrc((void*)a.x, 0, _x ? (unsigned)M * KS : 0u, 0x00020000); \
            const __amdgpu_buffer_rsrc_t rsw = __builtin_amdgcn_make_buffer_rsrc((void*)a.w, 0, _x ? 0u : (unsigned)KS * N, 0x00020000); \
            _Pragma("unroll") for (int i = 0; i < 2; ++i) xr[i] = __builtin_amdgcn_raw_buffer_load_b128(rsx, xv + i * 16 * KS, sox, 0); \
            _Pragma("unroll") for (int i = 0; i < 16; ++i) raw[i] = __builtin_amdgcn_raw_buffer_load_b32(rsw, wv, sow + i * N, 0); } while (0)
        COMM_LOAD(rawA, xrA, u0); COMM_LOAD(rawB, xrB, u1);
        { const unsigned e0 = a.order[tid], e1 = a.order[256 + tid < NBLOCK ? 256 + tid : NBLOCK - 1];
          *(LAS unsigned*)(lds + tid * 4) = e0; *(LAS unsigned*)(lds + 1024 + tid * 4) = e1; }
        CALL_INIT();
        __amdgpu_buffer_rsrc_t rd[NR - 1];
#pragma unroll
        for (int i = 0; i < NR - 1; ++i) rd[i] = __builtin_amdgcn_make_buffer_rsrc(a.peer[(unsigned)(i + 2 * wid + bid) % (NR - 1)] + doff, 0, DATA_BYTES, 0x00020000);
        GAS unsigned* fl = (GAS unsigned*)(a.peer[lane < NR - 1 ? lane : 0] + WS_FLAGS) + rank * 4;
        GAS unsigned* const tmo = (GAS unsigned*)(myws + WS_TMO);
        char* const tickp = myws + WS_TICKET;
        const unsigned tbase = (call - 1u) * NUNIT - 4u * NCOMM, one = 1u;
        bool bad = false;
#define SPIN_UNTIL(cond) do { const unsigned long long _t0 = __builtin_amdgcn_s_memrealtime(); \
            while (__builtin_amdgcn_ballot_w64(!(cond)) != 0ull) { \
                if (__builtin_amdgcn_readfirstlane(__hip_atomic_load(tmo, __ATOMIC_RELAXED, __HIP_MEMORY_SCOPE_AGENT)) == call \
                    || __builtin_amdgcn_s_memrealtime() - _t0 > DEADLINE_TICKS) { \
                    __hip_atomic_store(tmo, call, __ATOMIC_RELAXED, __HIP_MEMORY_SCOPE_AGENT); bad = true; break; } \
                __builtin_amdgcn_s_sleep(8); } } while (0)
#define CT(i) do {} while (0)
#define CTRACE() do {} while (0)
#define COMM_ST(o, vo, so) do { _Pragma("unroll") for (int d = 0; d < NR - 1; ++d) __builtin_amdgcn_raw_buffer_store_b128(o, rd[d], vo, so, AUX_SYS); ST_PAD(); } while (0)
#define UNIT_OFF(u) (((((u) >> 8) & 3u) * NR + rank) * SLAB + ((u) & 255u) * BLK + (((u) >> 16) & 2u) * 2048u + (((u) >> 16) & 1u) * 1024u)
        const __amdgpu_buffer_rsrc_t rl = __builtin_amdgcn_make_buffer_rsrc(myws + doff, 0, DATA_BYTES, 0x00020000);
#define COMM_STEP(raw, xr, first, word, craw, lraw) do { \
            u32x4 o0 = xr[0], o1 = xr[1]; \
            if (craw) { const unsigned sel = (u0 & 0x20000u) ? 0x07030602u : 0x05010400u; \
              _Pragma("unroll") for (int b = 0; b < 4; ++b) { \
                    const unsigned t0 = __builtin_amdgcn_perm(raw[4 * b + 1], raw[4 * b], sel), t2 = __builtin_amdgcn_perm(raw[4 * b + 3], raw[4 * b + 2], sel); \
                    o0[b] |= __builtin_amdgcn_perm(t2, t0, 0x05040100u); o1[b] |= __builtin_amdgcn_perm(t2, t0, 0x07060302u); } } \
            asm volatile("" : "+v"(o0), "+v"(o1)); CT(0); \
            const unsigned sd = UNIT_OFF(u0); \
            if (lraw) COMM_LOAD(raw, xr, u2); \
            else { xr[0] = __builtin_amdgcn_raw_buffer_load_b128(rl, ov, UNIT_OFF(u2), AUX_SYS); xr[1] = __builtin_amdgcn_raw_buffer_load_b128(rl, ov + 2048, UNIT_OFF(u2), AUX_SYS); } \
            SCHED; CT(1); COMM_ST(o0, ov, sd); CT(2); if (lraw) WAIT_V(25); else WAIT_V(9); CT(3); \
            if (!(first) && tid == 0) { unsigned _t; asm volatile("v_subrev_u32 %0, %1, v255" : "=v"(_t) : "s"(tbase)); \
                *(volatile LAS unsigned*)(lds + LDS_MISC + (word)) = (_t << 16) | *(volatile LAS unsigned*)(lds + ((_t & 2047u) & ~3u)); } \
            WAIT_L(0); BAR; CT(4); \
            unsigned un = 0; \
            if (!(first)) { un = __builtin_amdgcn_readfirstlane(*(volatile LAS unsigned*)(lds + LDS_MISC + (word))); \
                if (tid < NR - 1) __hip_atomic_store(fl + pfl, call, __ATOMIC_RELAXED, __HIP_MEMORY_SCOPE_SYSTEM); } \
            else CALL_ADVANCE(); \
              \
            if (tid == 0) asm volatile("global_atomic_add v255, %0, %1, off sc0" :: "v"(tickp), "v"(one) : "memory", "v255"); \
            COMM_ST(o1, ov + 2048, sd); CT(5); \
            CTRACE(); \
            pfl = (u0 & 255u) * 128u + ((u0 >> 8) & 3u) * (NR * 4u) + ((u0 >> 16) & 3u); \
            u0 = u1; u1 = u2; if (first) { u2 = u3; } else { u2 = un; } } while (0)
#define COMM_END ((u0 >> 16) >= NUNIT)
        COMM_STEP(rawA, xrA, true, 8, true, true);
        char* dout[NR - 1]; unsigned shoff[NSHARE]; float* outp = a.out;
#pragma unroll
        for (int i = 0; i < NR - 1; ++i) { dout[i] = a.peer_out[(unsigned)(i + 2 * wid + bid) % (NR - 1)]; asm volatile("" : "+s"(dout[i])); }
#pragma unroll
        for (int i = 0; i < NSHARE; ++i) { shoff[i] = a.shared[i]; asm volatile("" : "+s"(shoff[i])); }
        asm volatile("" : "+s"(outp));
        const __amdgpu_buffer_rsrc_t ro = __builtin_amdgcn_make_buffer_rsrc((void*)outp, 0, (unsigned)M * N * 4u, 0x00020000);
        do {
            for (;;) {
                COMM_STEP(rawB, xrB, false, 12, true, true); if (COMM_END) break;
                COMM_STEP(rawA, xrA, false, 8, true, true); if ((u0 >> 16) >= RAW_UNITS + RAW_STEP * bid) break;
            }
            if (COMM_END) break;
            SPIN_UNTIL(__hip_atomic_load((GAS unsigned*)(myws + WS_RETILE), __ATOMIC_RELAXED, __HIP_MEMORY_SCOPE_AGENT) >= call * (2u * NPANEL));
            COMM_STEP(rawB, xrB, false, 12, true, false); if (COMM_END) break;
            COMM_STEP(rawA, xrA, false, 8, true, false); if (COMM_END) break;
            for (;;) {
                COMM_STEP(rawB, xrB, false, 12, false, false); if (COMM_END) break;
                COMM_STEP(rawA, xrA, false, 8, false, false); if (COMM_END) break;
            }
        } while (0);
#define TDONE() (__hip_atomic_load((GAS unsigned*)(myws + WS_TDONE) + (lane < NSHARE ? lane : 0), __ATOMIC_RELAXED, __HIP_MEMORY_SCOPE_AGENT) >= call)
#define PUBLISH() do { BAR; if (tid < NR - 1) __hip_atomic_store(fl + pfl, call, __ATOMIC_RELAXED, __HIP_MEMORY_SCOPE_SYSTEM); } while (0)
        const bool ready = __builtin_amdgcn_ballot_w64(!TDONE()) == 0ull;
        WAIT_V(0);
        if (!ready) { PUBLISH(); SPIN_UNTIL(TDONE()); }
        __builtin_amdgcn_fence(__ATOMIC_ACQUIRE, "agent");
        const unsigned fv = (unsigned)((bid * 4 + wid) * 4) * 65536u + lane * 16;
        u32x4 fw[NSHARE][4];
#pragma unroll
        for (int t = 0; t < NSHARE; ++t)
#pragma unroll
            for (int i = 0; i < 4; ++i) { fw[t][i] = __builtin_amdgcn_raw_buffer_load_b128(ro, fv, shoff[t] + i * 65536u, AUX_SYS); SCHED; }
        if (ready) PUBLISH();
        if (bad) fw[0][0] = (u32x4){0x7fc00000u, 0x7fc00000u, 0x7fc00000u, 0x7fc00000u};
#pragma unroll
        for (int i = 0; i < NR - 1; ++i) rd[i] = __builtin_amdgcn_make_buffer_rsrc(dout[i], 0, (unsigned)M * N * 4u, 0x00020000);
#pragma unroll
        for (int t = 0; t < NSHARE; ++t)
#pragma unroll
            for (int i = 0; i < 4; ++i) { COMM_ST(fw[t][i], fv, shoff[t] + i * 65536u); }
        WAIT_V(0); BAR;
        if (wid == 0) {
            if (lane < NR - 1) __hip_atomic_store((GAS unsigned*)(a.peer[lane] + WS_FWD) + rank * NCOMM + bid, call, __ATOMIC_RELAXED, __HIP_MEMORY_SCOPE_SYSTEM);
            SPIN_UNTIL(__hip_atomic_load((GAS unsigned*)(myws + WS_FWD) + ((rank + 1u + (lane < NR - 1 ? lane : 0)) & (NR - 1)) * NCOMM + bid, __ATOMIC_RELAXED, __HIP_MEMORY_SCOPE_SYSTEM) >= call);
            if (bad && lane == 0) a.out[(shoff[0] + fv) / 4] = __builtin_nanf("");
        }
        return;
    }

    LANE_INIT(); CALL_LOAD();
    const int gw = bid - NCOMM;
    const unsigned long long tl = a.tiles[gw];
    const int ntiles = (int)(tl >> 50) & 7;
    const unsigned koff0 = (unsigned)(tl >> 56) & 31u;
    const int wr = wid >> 1, wc = wid & 1, fr = lane & 15, fq = lane >> 4;
    constexpr int NLOC = 2 * NPANEL;
    const int lw = gw - (NWG - NCOMM - NLOC);
    unsigned raw[4][16];
    if (lw >= 0) {
        const unsigned h = lw & 1, p = lw >> 1;
        if (p < 16) {
            const unsigned xv = (unsigned)(wid * 64 + (lane & 15)) * KS + h * 64 + (lane >> 4) * 16;
#pragma unroll
            for (int i = 0; i < 4; ++i)
#pragma unroll
                for (int j = 0; j < 4; ++j) { const u32x4 v = __builtin_amdgcn_raw_buffer_load_b128(rx, xv + j * 16 * KS, p * (256u * KS) + i * 128u, 0);
                    raw[i][4 * j] = v[0]; raw[i][4 * j + 1] = v[1]; raw[i][4 * j + 2] = v[2]; raw[i][4 * j + 3] = v[3]; }
        } else {
#pragma unroll
            for (int i = 0; i < 4; ++i)
#pragma unroll
                for (int j = 0; j < 16; ++j) raw[i][j] = __builtin_amdgcn_raw_buffer_load_b32(rw, wv, (i * 128u + h * 64u + j) * N + (p - 16u) * 256u, 0);
        }
    }
    if (tid == 0) *(LAS unsigned*)(lds + LDS_MISC + 4) = 0u;
    CALL_INIT();
    const __amdgpu_buffer_rsrc_t rin = __builtin_amdgcn_make_buffer_rsrc(myws + doff, 0, DATA_BYTES, 0x00020000);
    const __amdgpu_buffer_rsrc_t rfl = __builtin_amdgcn_make_buffer_rsrc(myws + WS_FLAGS, 0, FLAG_BYTES, 0x00020000);
    GAS unsigned* tmo = (GAS unsigned*)(myws + WS_TMO);
    if (lw >= 0) {
        const unsigned h = lw & 1, p = lw >> 1;
#pragma unroll
        for (int i = 0; i < 4; ++i) {
            const unsigned so = (i * NR + rank) * SLAB + p * BLK + h * 1024;
            if (p < 16) {
#pragma unroll
                for (int j = 0; j < 4; ++j) __builtin_amdgcn_raw_buffer_store_b128((u32x4){raw[i][4 * j], raw[i][4 * j + 1], raw[i][4 * j + 2], raw[i][4 * j + 3]}, rin, lane * 16, so + (wid * 4 + j) * 2048, AUX_SYS);
                ST_PAD();
            } else {
                u32x4 o[4];
#pragma unroll
                for (int b = 0; b < 4; ++b) {
                    const unsigned r0 = raw[i][4 * b], r1 = raw[i][4 * b + 1], r2 = raw[i][4 * b + 2], r3 = raw[i][4 * b + 3];
                    const unsigned t0 = __builtin_amdgcn_perm(r1, r0, 0x05010400), t1 = __builtin_amdgcn_perm(r1, r0, 0x07030602);
                    const unsigned t2 = __builtin_amdgcn_perm(r3, r2, 0x05010400), t3 = __builtin_amdgcn_perm(r3, r2, 0x07030602);
                    o[0][b] = __builtin_amdgcn_perm(t2, t0, 0x05040100); o[1][b] = __builtin_amdgcn_perm(t2, t0, 0x07060302);
                    o[2][b] = __builtin_amdgcn_perm(t3, t1, 0x05040100); o[3][b] = __builtin_amdgcn_perm(t3, t1, 0x07060302);
                }
#pragma unroll
                for (int c = 0; c < 4; ++c) __builtin_amdgcn_raw_buffer_store_b128(o[c], rin, lane * 16, so + wid * 8192 + c * 2048, AUX_SYS);
                ST_PAD();
            }
        }
    }
    asm volatile("s_waitcnt vmcnt(0) lgkmcnt(0)" ::: "memory"); BAR;
    CALL_ADVANCE();
    if (lw >= 0 && tid < 8)
        __hip_atomic_store((GAS unsigned*)(myws + WS_FLAGS) + (lw >> 1) * 128u + ((tid >> 1) * NR + rank) * 4u + (tid & 1) * 2 + (lw & 1), call, __ATOMIC_RELAXED, __HIP_MEMORY_SCOPE_SYSTEM);
    if (lw >= 0 && tid == 0) __hip_atomic_fetch_add((GAS unsigned*)(myws + WS_RETILE), 1u, __ATOMIC_RELAXED, __HIP_MEMORY_SCOPE_AGENT);
    const float scale = __builtin_bit_cast(float, __builtin_amdgcn_readfirstlane(__builtin_bit_cast(int, a.sx[0] * a.sw[0])));
    const unsigned voff = wid * 4096 + lane * 16, ldsw = wid * 4096;
    const unsigned ldsb = (unsigned)(unsigned long)lds;
    unsigned aoff = ldsb + wr * 8192 + lane * 16, boff = ldsb + 4 * HB + wc * 8192 + lane * 16;
    asm volatile("" : "+v"(aoff), "+v"(boff));
    const unsigned ovoff = (unsigned)(wr * 64 + 4 * fq) * 65536u + wc * 256 + fr * 16;
#define TILE(i) ((unsigned)(tl >> (10 * (i))) & 0x3ffu)
#define SA(b, h) (((b) * 2 + (h)) * HB)
#define SB(b, h) ((4 + (b) * 2 + (h)) * HB)
#define G1(bufoff, soff, i) __builtin_amdgcn_raw_ptr_buffer_load_lds(rin, (LAS void*)(lds + (bufoff) + ldsw), 16, voff, (soff), (i) * 1024, AUX_SYS)
#define STAGE4(bufoff, soff) do { G1(bufoff, soff, 0); G1(bufoff, soff, 1); G1(bufoff, soff, 2); G1(bufoff, soff, 3); } while (0)
#define RD(U, i, base, off) asm volatile("ds_read_b128 v[%c1:%c2], %0 offset:%c3" :: "v"(base), "i"((U) + 4 * (i)), "i"((U) + 4 * (i) + 3), "i"((off) + (i) * 1024) : "memory")
#define MF(Z, ai, bj, m, n, RA, RB) mfma<acc_idx(ai, bj, m, n), Z, (RA) + 8 * (m), (RB) + 8 * (n)>()
#define EPG(eq, g) do { if constexpr ((eq) >= 0) epi_gap<(eq) < 0 ? 0 : (eq), (g)>(esc, erout, ovoff, eobase); } while (0)
#define QUAD(Z, ai, bj, RA, RB, RN, rdb, rdoff, stoff, soff, vm, eq, g0, hook) do { u32x4 _fv; unsigned _pi; \
        if constexpr ((vm) >= 0) { asm volatile("s_waitcnt vmcnt(%0) lgkmcnt(0)" :: "n"((vm) < 0 ? 0 : (vm)) : "memory"); BAR; } else WAIT_L(0); SCHED; \
        MF(Z, ai, bj, 0, 0, RA, RB); RD(RN, 0, rdb, rdoff); EPG(eq, g0 + 0); SCHED; \
        MF(Z, ai, bj, 0, 1, RA, RB); RD(RN, 1, rdb, rdoff); EPG(eq, g0 + 1); SCHED; \
        MF(Z, ai, bj, 0, 2, RA, RB); RD(RN, 2, rdb, rdoff); EPG(eq, g0 + 2); if constexpr ((hook) >= 3) PICK_A(_pi); SCHED; \
        MF(Z, ai, bj, 0, 3, RA, RB); RD(RN, 3, rdb, rdoff); EPG(eq, g0 + 3); SCHED; \
        MF(Z, ai, bj, 1, 0, RA, RB); RD(RN, 4, rdb, rdoff); EPG(eq, g0 + 4); if constexpr ((hook) == 3) PICK_B(_pi, kaA, kaB); if constexpr ((hook) == 4) PICK_B(_pi, kbA, kbB); SCHED; \
        MF(Z, ai, bj, 1, 1, RA, RB); RD(RN, 5, rdb, rdoff); EPG(eq, g0 + 5); SCHED; \
        MF(Z, ai, bj, 1, 2, RA, RB); RD(RN, 6, rdb, rdoff); EPG(eq, g0 + 6); SCHED; \
        MF(Z, ai, bj, 1, 3, RA, RB); RD(RN, 7, rdb, rdoff); EPG(eq, g0 + 7); SCHED; \
        MF(Z, ai, bj, 2, 0, RA, RB); G1(stoff, soff, 0); EPG(eq, g0 + 8); SCHED; \
        MF(Z, ai, bj, 2, 1, RA, RB); EPG(eq, g0 + 9); if constexpr ((hook) == 2) _fv = FLAG_LOAD(0); SCHED; \
        MF(Z, ai, bj, 2, 2, RA, RB); G1(stoff, soff, 1); EPG(eq, g0 + 10); SCHED; \
        MF(Z, ai, bj, 2, 3, RA, RB); EPG(eq, g0 + 11); SCHED; \
        MF(Z, ai, bj, 3, 0, RA, RB); G1(stoff, soff, 2); EPG(eq, g0 + 12); SCHED; \
        MF(Z, ai, bj, 3, 1, RA, RB); EPG(eq, g0 + 13); if constexpr ((hook) == 2) FLAG_MASK(_fv); SCHED; \
        MF(Z, ai, bj, 3, 2, RA, RB); G1(stoff, soff, 3); EPG(eq, g0 + 14); SCHED; \
        MF(Z, ai, bj, 3, 3, RA, RB); EPG(eq, g0 + 15); if constexpr ((hook) == 1) FLAG_FETCH(0); SCHED; } while (0)

    unsigned rmask = 0, done = 0, kaA, kaB, kbA, kbB;
#define BASE_A(tile) (((tile) & 15u) * BLK)
#define BASE_B(tile) ((16u + ((tile) >> 4)) * BLK)
#define FLAG_SOFF(tile) ((wid < 2 ? ((tile) & 15u) : 16u + ((tile) >> 4)) * 512u + (wid & 1) * 256u)
    unsigned pA = BASE_A(TILE(0)), pB = BASE_B(TILE(0)), psoff = FLAG_SOFF(TILE(0)), fsoff = psoff;
    const unsigned fvoff = lane * 4;
    int froff = LDS_FLAGS + ((lane & 32) + ((lane + koff0) & 31)) * 16; asm volatile("" : "+v"(froff));
#define FLAG_FETCH_AT(slot, so) __builtin_amdgcn_raw_ptr_buffer_load_lds(rfl, (LAS void*)(lds + LDS_FLAGS + (slot) * 1024 + wid * 256), 4, fvoff, so, 0, AUX_SYS)
#define FLAG_FETCH(slot) FLAG_FETCH_AT(slot, fsoff)
#define FLAG_LOAD(slot) (*(volatile LAS u32x4*)(lds + froff + (slot) * 1024))
#define FLAG_MASK(_v) do { const unsigned _a = _v[0] < _v[1] ? _v[0] : _v[1], _b = _v[2] < _v[3] ? _v[2] : _v[3]; \
            const unsigned long long _m = __builtin_amdgcn_ballot_w64((_a < _b ? _a : _b) >= call); \
            rmask |= (unsigned)_m & (unsigned)(_m >> 32); } while (0)
#define TWAIT(t0) do {} while (0)
#define PICK_A(_i) do { \
            if (__builtin_expect((rmask & ~done) == 0u, 0)) {        \
                const unsigned long long _t0 = __builtin_amdgcn_s_memrealtime(); \
                for (;;) { \
                    FLAG_FETCH_AT(1, psoff); \
                    if (wid == 0 && __builtin_amdgcn_s_memrealtime() - _t0 > 10000ull        \
                        && (__builtin_amdgcn_readfirstlane(__hip_atomic_load(tmo, __ATOMIC_RELAXED, __HIP_MEMORY_SCOPE_AGENT)) == call \
                            || __builtin_amdgcn_s_memrealtime() - _t0 > DEADLINE_TICKS)) { \
                        __hip_atomic_store(tmo, call, __ATOMIC_RELAXED, __HIP_MEMORY_SCOPE_AGENT); \
                        *(volatile LAS unsigned*)(lds + LDS_MISC + 4) = 1u; } \
                    asm volatile("s_waitcnt vmcnt(0) lgkmcnt(0)" ::: "memory"); BAR; \
                    const u32x4 _pv = FLAG_LOAD(1); FLAG_MASK(_pv); \
                    if (__builtin_amdgcn_readfirstlane(*(volatile LAS unsigned*)(lds + LDS_MISC + 4)) != 0u) rmask = ~0u; \
                    WAIT_L(0); BAR; \
                    if ((rmask & ~done) != 0u) break; \
                    __builtin_amdgcn_s_sleep(POLL_SLEEP); \
                } \
                TWAIT(_t0); \
            } \
            _i = __builtin_ctz(rmask & ~done); done |= 1u << _i; } while (0)
#define PICK_B(_i, oA, oB) do { const unsigned _q = ((_i + koff0) & 31u) * SLAB; oA = _q + pA; oB = _q + pB; asm volatile("" : "+s"(oA), "+s"(oB)); } while (0)
#define PICK(oA, oB) do { unsigned _pi; PICK_A(_pi); PICK_B(_pi, oA, oB); } while (0)
#define ITER(KIND, sw, S1, S2, S3, S4) do { \
        if (sw) fsoff = nsoff; \
        QUAD(KIND == 0, 0, 0, S1, V_B0A, V_B0B, boff, SA(1, 0), SA(1, 0), kbA, (KIND == 0 ? 38 : KIND == 3 ? 27 : 16), (KIND == 0 ? 3 : -1), 0, 4); \
        QUAD(false, 0, 0, S2, V_B0B, S3, boff, SA(0, 1), SB(0, 0), kaB, -1, (KIND == 0 ? 3 : -1), 16, 1); \
        if constexpr (KIND == 2) { \
            esc = *(volatile LAS unsigned*)(lds + LDS_MISC + 4) != 0u ? __builtin_nanf("") : scale; \
            esc = __builtin_bit_cast(float, __builtin_amdgcn_readfirstlane(__builtin_bit_cast(int, esc))); } \
        QUAD(KIND == 0, 0, 1, S1, S3, S4, boff, SA(1, 1), SB(1, 0), kbB, (KIND == 0 ? 38 : 17), (KIND == 0 ? 3 : KIND == 2 ? 0 : -1), (KIND == 0 ? 32 : 0), 0); \
        QUAD(false, 0, 1, S2, S4, S1, aoff, SA(0, 1), SB(0, 1), kaB + HB, -1, (KIND == 0 ? 2 : KIND == 2 ? 0 : -1), (KIND == 0 ? 0 : 16), 0); \
        QUAD(KIND == 0, 1, 1, S1, S3, S2, aoff, SA(1, 1), SB(1, 1), kbB + HB, (KIND == 0 ? 38 : KIND == 2 ? 27 : 17), (KIND == 0 ? 2 : KIND == 2 ? 0 : -1), (KIND == 0 ? 16 : 32), 0); \
        QUAD(false, 1, 1, S2, S4, S3, aoff, SA(0, 0), SA(0, 1), kaA + HB, -1, (KIND == 0 ? 2 : KIND == 2 ? 1 : -1), (KIND == 0 ? 32 : 0), 0); \
        if (sw) { done = 0; rmask = nmask; pA = nA; pB = nB; psoff = nsoff; } \
        QUAD(KIND == 0, 1, 0, S1, V_B0A, S4, aoff, SA(1, 0), SA(1, 1), kbA + HB, (KIND == 0 ? 38 : KIND == 2 ? 37 : 16), (KIND == 2 ? 1 : -1), 16, 2); \
        QUAD(false, 1, 0, S2, V_B0B, V_B0A, boff, SA(0, 0), SA(0, 0), kaA, -1, (KIND == 2 ? 1 : -1), 32, 3); } while (0)

#define STAMP(ui, it) do {} while (0)
    PICK(kaA, kaB);
    STAGE4(SA(0, 0), kaA); STAGE4(SB(0, 0), kaB); STAGE4(SB(0, 1), kaB + HB); STAGE4(SA(0, 1), kaA + HB);
    PICK(kbA, kbB);
    STAGE4(SA(1, 0), kbA); STAGE4(SB(1, 0), kbB); STAGE4(SB(1, 1), kbB + HB); STAGE4(SA(1, 1), kbA + HB);
    PICK(kaA, kaB);
    WAIT_V(12); BAR;
    asm volatile("; GEMM_REGS_BEGIN" ::: "memory", RCLOB);
#define RD8(U, base, off) RD(U, 0, base, off); RD(U, 1, base, off); RD(U, 2, base, off); RD(U, 3, base, off); RD(U, 4, base, off); RD(U, 5, base, off); RD(U, 6, base, off); RD(U, 7, base, off)
    RD8(V_R1, aoff, SA(0, 0)); RD8(V_B0A, boff, SA(0, 0)); WAIT_L(0); RD8(V_R2, aoff, SA(1, 0));
    WAIT_L(0); BAR;
    STAGE4(SA(0, 0), kaA);
    WAIT_V(0);
    __amdgpu_buffer_rsrc_t erout = __builtin_amdgcn_make_buffer_rsrc((void*)a.out, 0, 0, 0x00020000);
    unsigned eobase = 0; float esc = 0.f;
    int ui = 0;
    do {
        const bool more = ui + 1 < ntiles; const unsigned nt = TILE(more ? ui + 1 : ui);
        const unsigned nA = BASE_A(nt), nB = BASE_B(nt), nsoff = FLAG_SOFF(nt), nmask = more ? 0u : ~0u;
        STAMP(ui, 0);
        ITER(0, false, V_R1, V_R2, V_R3, V_R4);
        STAMP(ui, 1);
        ITER(3, false, V_R3, V_R4, V_R1, V_R2);
        if (__builtin_expect(ui == (int)((tl >> 53) & 7u) && ui > 0, 0)) {
            WAIT_V(0); BAR;
            if (tid == 0) {
                __builtin_amdgcn_fence(__ATOMIC_RELEASE, "agent"); WAIT_V(0);
                __hip_atomic_store((GAS unsigned*)(myws + WS_TDONE) + (unsigned)(tl >> 61), call, __ATOMIC_RELAXED, __HIP_MEMORY_SCOPE_AGENT);
            }
        }
        for (int it = 2; it < 14; it += 2) { STAMP(ui, it); ITER(1, false, V_R1, V_R2, V_R3, V_R4); STAMP(ui, it + 1); ITER(1, false, V_R3, V_R4, V_R1, V_R2); }
        STAMP(ui, 14);
        ITER(1, true, V_R1, V_R2, V_R3, V_R4);
        erout = __builtin_amdgcn_make_buffer_rsrc((void*)a.out, 0, (unsigned)M * N * 4u, 0x00020000);
        eobase = (TILE(ui) & 15u) * (256u * 65536u) + (TILE(ui) >> 4) * 1024u;
        STAMP(ui, 15);
        ITER(2, false, V_R3, V_R4, V_R1, V_R2);
    } while (++ui < ntiles);
    STAMP(ntiles, 0);
#define EP3(q, g) EPG(q, g); EPG(q, g + 1); EPG(q, g + 2)
#define EP12(q, g) EP3(q, g); EP3(q, g + 3); EP3(q, g + 6); EP3(q, g + 9)
    EP12(3, 0); EP12(3, 12); EP12(3, 24); EP12(3, 36); EP12(2, 0); EP12(2, 12); EP12(2, 24); EP12(2, 36);
    WAIT_V(0);
    asm volatile("; GEMM_REGS_END" ::: "memory");
}

#define P_KS0 2.0
#define P_SX -3.0
#define P_RX 1.18
#define P_SPS 8.0
#define P_TAU 38.0
#define P_ZOFF -8.0
#define P_KZ 25
#define P_RH 1
#define P_LEN2 38.0
#define P_S3 44.0
#define P_G1X -2.0
void make_tables(Args& a, int rank) {
    double key[320]; int n = 0;
    constexpr int PZ = 15 - NZ;
    auto koff = [](int p) { return p < 4 ? 0 : p < PZ ? (int)(P_KS0 + (p - 4) * P_SPS / (PZ - 5) / 1.25 + 0.5) : P_KZ; };
    auto phase = [](int p) { return p < PZ ? p * P_SPS / (PZ - 5) : P_ZOFF; };
    auto add = [&](int panel, double start, double rate, int rot, int off) {
        for (int kt = 0; kt < 4; ++kt) {
            const double k = start + ((kt * NR + ((rank + rot) & (NR - 1)) - off) & 31) * rate + panel * 1e-6;
            int i = n++;
            for (; i > 0 && key[i - 1] > k; --i) { key[i] = key[i - 1]; a.order[i] = a.order[i - 1]; }
            key[i] = k; a.order[i] = (unsigned)((kt << 8) | panel);
        }
    };
    for (int i = 0; i < 8; ++i) { add(i, -1000.0 + P_G1X, 1.0, i, 0); add(16 + i, -1000.0, 1.0, i, 0); add(8 + i, P_SX, P_RX, i, 0); }
    for (int p = 4; p < PZ; ++p) add(16 + 8 + (p - 4), phase(p) - phase(4), P_LEN2 / 32.0, p, koff(p));
    for (int g = 0; g < 3; ++g)
        for (int p = 0; p < 15; ++p) add(16 + 19 + g * 15 + p, P_S3 + g * P_TAU + phase(p), 1.18, p, koff(p));
    for (int i = 0; i < NZ; ++i) add(16 + 19 - NZ + i, P_S3 + P_RH * P_TAU + phase(PZ), 1.18, 4 * i + 1, koff(PZ));
    const int H = 19 + P_RH * 15 + PZ;
    int smp[NSHARE], snp[NSHARE];
    for (int i = 0; i < 2 * NZ; ++i) { smp[i] = (2 * NZ * rank + i) & 15; snp[i] = 19 - NZ + ((2 * NZ * rank) >> 4); }
    smp[2 * NZ] = rank; snp[2 * NZ] = H;
    for (int i = 0; i < NSHARE; ++i) a.shared[i] = (unsigned)smp[i] * (256u * 65536u) + (unsigned)snp[i] * 1024u;
    for (int g = 0; g < NWG - NCOMM; ++g) {
        const int xcd = g & 7, r = (xcd >> 2) * 30 + (g >> 3), p = (2 * (r >> 2)) % 15, m = (xcd & 3) * 4 + (r & 3);
        int mp[5], np[5], nt = 0; unsigned long long tl = 0;
        if (p < 4) { mp[0] = m & 7; mp[1] = 8 + (m & 7); np[0] = np[1] = 2 * p + (m >> 3); nt = 2; }
        else if (p < PZ) { mp[0] = m; np[0] = 8 + p - 4; nt = 1; }
        for (int k = 0; k < 3; ++k) {
            if (19 + k * 15 + p == H && m < 8) {
                if (m < NSHARE) { mp[nt] = smp[m]; np[nt] = snp[m]; ++nt; tl |= ((unsigned long long)nt << 53) | ((unsigned long long)m << 61); }
            } else { mp[nt] = m; np[nt] = 19 + k * 15 + p; ++nt; }
        }
        tl |= ((unsigned long long)nt << 50) | ((unsigned long long)koff(p) << 56);
        for (int i = 0; i < nt; ++i) tl |= (unsigned long long)((np[i] << 4) | mp[i]) << (10 * i);
        a.tiles[g] = tl;
    }
}
}

extern "C" void kernel_launch(int rank, int nranks, ncclComm_t comm,
                              hipStream_t stream,
                              void* const* d_in, const int* in_sizes, int n_in,
                              void* d_out, int out_size,
                              const int* in_dtypes, int out_dtype,
                              void* const* d_out_all,
                              void* const* d_ws_all, size_t ws_bytes) {
    (void)comm; (void)in_dtypes; (void)out_dtype;
    if (nranks != NR || (unsigned)rank >= (unsigned)NR || n_in != 4 || in_sizes[0] != M * KS || in_sizes[1] != KS * N ||
        out_size != M * N || ws_bytes < WS_NEED) {
        fprintf(stderr, "kernel_launch: unexpected shapes\n");
        return;
    }
    Args a;
    a.x = (const unsigned char*)d_in[0]; a.w = (const unsigned char*)d_in[1];
    a.sx = (const float*)d_in[2]; a.sw = (const float*)d_in[3];
    a.out = (float*)d_out;
    a.myws = (char*)d_ws_all[rank];
    for (int i = 0; i < NR - 1; ++i) { a.peer[i] = (char*)d_ws_all[(rank + 1 + i) % NR]; a.peer_out[i] = (char*)d_out_all[(rank + 1 + i) % NR]; }
    a.rank = rank;
    make_tables(a, rank);
    hipLaunchKernelGGL(fused_gemm, dim3(NWG), dim3(256), 0, stream, a);
    HIP_CHECK(hipGetLastError());
}
```

```cpp
#include <hip/hip_runtime.h>
#include <rccl/rccl.h>
#include <cstdio>

#define HIP_CHECK(expr) do { hipError_t _e = (expr); if (_e != hipSuccess) { \
    fprintf(stderr, "HIP error %s at %s:%d\n", hipGetErrorString(_e), __FILE__, __LINE__); \
    return; } } while (0)

#define LAS __attribute__((address_space(3)))
#define GAS __attribute__((address_space(1)))
typedef unsigned u32x4 __attribute__((ext_vector_type(4)));

namespace {
constexpr int NR = 8;
constexpr int M = 4096, N = 16384, KS = 512;
constexpr unsigned BLK = 32768, HB = 16384;
constexpr unsigned NPANEL = 80;
constexpr unsigned SLAB = NPANEL * BLK;
constexpr unsigned DATA_BYTES = 32 * SLAB;
constexpr unsigned WS_FLAGS = 0;
constexpr unsigned FLAG_BYTES = NPANEL * 512;
constexpr unsigned WS_CTR = 65536;
constexpr unsigned WS_TMO = WS_CTR + 256 * 128;
constexpr unsigned WS_TICKET = WS_TMO + 128;
constexpr unsigned WS_TDONE = WS_TICKET + 128;
constexpr int NZ = 2;
constexpr int NSHARE = 2 * NZ + 1;
constexpr unsigned WS_RETILE = WS_TDONE + 128;
constexpr unsigned WS_FLIP = WS_RETILE + 128;
constexpr unsigned WS_FWD = WS_FLIP + 128;
constexpr unsigned NBLOCK = 320, NUNIT = 4 * NBLOCK;
constexpr size_t WS_DATA = 1u << 20;
constexpr size_t WS_NEED = WS_DATA + 2 * (size_t)DATA_BYTES;
constexpr int NCOMM = 16, NWG = 256;
constexpr unsigned long long DEADLINE_TICKS = 50000000ull;
constexpr int AUX_SYS = 17;
constexpr unsigned RAW_UNITS = 64u, RAW_STEP = 8u;
constexpr unsigned LDS_FLAGS = 8 * HB;
constexpr unsigned LDS_MISC = LDS_FLAGS + 2 * 1024;
constexpr unsigned LDS_BYTES = LDS_MISC + 64;

struct Args {
    const unsigned char* x; const unsigned char* w; const float* sx; const float* sw; float* out;
    char* myws; char* peer[NR - 1];
    char* peer_out[NR - 1];
    int rank;
    unsigned shared[NSHARE];
    unsigned order[NBLOCK];
    unsigned long long tiles[NWG - NCOMM];
};

#define WAIT_V(n) asm volatile("s_waitcnt vmcnt(" #n ")" ::: "memory")
#define WAIT_L(n) asm volatile("s_waitcnt lgkmcnt(" #n ")" ::: "memory")
#define BAR __builtin_amdgcn_s_barrier()
#define SCHED __builtin_amdgcn_sched_barrier(0)
#define ST_PAD() do { SCHED; asm volatile("s_nop 1"); SCHED; } while (0)

#define RCLOB "a0","a1","a2","a3","a4","a5","a6","a7","a8","a9","a10","a11","a12","a13","a14","a15","a16","a17","a18","a19","a20","a21","a22","a23","a24","a25","a26","a27","a28","a29","a30","a31","a32","a33","a34","a35","a36","a37","a38","a39","a40","a41","a42","a43","a44","a45","a46","a47","a48","a49","a50","a51","a52","a53","a54","a55","a56","a57","a58","a59","a60","a61","a62","a63","a64","a65","a66","a67","a68","a69","a70","a71","a72","a73","a74","a75","a76","a77","a78","a79","a80","a81","a82","a83","a84","a85","a86","a87","a88","a89","a90","a91","a92","a93","a94","a95","a96","a97","a98","a99","a100","a101","a102","a103","a104","a105","a106","a107","a108","a109","a110","a111","a112","a113","a114","a115","a116","a117","a118","a119","a120","a121","a122","a123","a124","a125","a126","a127","a128","a129","a130","a131","a132","a133","a134","a135","a136","a137","a138","a139","a140","a141","a142","a143","a144","a145","a146","a147","a148","a149","a150","a151","a152","a153","a154","a155","a156","a157","a158","a159","a160","a161","a162","a163","a164","a165","a166","a167","a168","a169","a170","a171","a172","a173","a174","a175","a176","a177","a178","a179","a180","a181","a182","a183","a184","a185","a186","a187","a188","a189","a190","a191","a192","a193","a194","a195","a196","a197","a198","a199","a200","a201","a202","a203","a204","a205","a206","a207","a208","a209","a210","a211","a212","a213","a214","a215","a216","a217","a218","a219","a220","a221","a222","a223","v24","v25","v26","v27","v28","v29","v30","v31","v32","v33","v34","v35","v36","v37","v38","v39","v40","v41","v42","v43","v44","v45","v46","v47","v48","v49","v50","v51","v52","v53","v54","v55","v56","v57","v58","v59","v60","v61","v62","v63","v64","v65","v66","v67","v68","v69","v70","v71","v72","v73","v74","v75","v76","v77","v78","v79","v80","v81","v82","v83","v84","v85","v86","v87","v88","v89","v90","v91","v92","v93","v94","v95","v96","v97","v98","v99","v100","v101","v102","v103","v104","v105","v106","v107","v108","v109","v110","v111","v112","v113","v114","v115","v116","v117","v118","v119","v120","v121","v122","v123","v124","v125","v126","v127","v128","v129","v130","v131","v132","v133","v134","v135","v136","v137","v138","v139","v140","v141","v142","v143","v144","v145","v146","v147","v148","v149","v150","v151","v152","v153","v154","v155","v156","v157","v158","v159","v160","v161","v162","v163","v164","v165","v166","v167","v168","v169","v170","v171","v172","v173","v174","v175","v176","v177","v178","v179","v180","v181","v182","v183","v184","v185","v186","v187","v188","v189","v190","v191","v192","v193","v194","v195","v196","v197","v198","v199","v200","v201","v202","v203","v204","v205","v206","v207","v208","v209","v210","v211","v212","v213","v214","v215","v216","v217","v218","v219","v220","v221","v222","v223","v224","v225","v226","v227","v228","v229","v230","v231","v232","v233","v234","v235","v236","v237","v238","v239","v240","v241","v242","v243","v244","v245","v246","v247","v248","v249","v250","v251","v252","v253","v254","v255"
constexpr int V_EV = 24, V_R1 = 0, V_R2 = 32, V_R3 = 64, V_R4 = 96, V_B0A = 128, V_B0B = 160;
constexpr int acc_idx(int ai, int bj, int m, int n) { return ((ai * 2 + bj) * 4 + m) * 4 + n; }
constexpr bool acc_in_v(int f) { return f < 56; }
constexpr int acc_reg(int f) { return f < 56 ? 32 + 4 * f : 192 + 4 * (f - 56); }
#define MFMA_ASM(D, C, ...) asm volatile("v_mfma_f32_16x16x128_f8f6f4 " D "[%c0:%c1], a[%c2:%c3], a[%c4:%c5], " C " cbsz:1 blgp:1" \
                                         :: "i"(R), "i"(R + 3), "i"(X), "i"(X + 7), "i"(Y), "i"(Y + 7) __VA_ARGS__)
template <int F, bool ZERO, int X, int Y> __device__ __forceinline__ void mfma() {
    constexpr int R = acc_reg(F);
    if constexpr ((F & 15) == 0) { if constexpr (ZERO) MFMA_ASM("v", "0", : RCLOB); else MFMA_ASM("v", "v[%c0:%c1]", : RCLOB); }
    else if constexpr (acc_in_v(F)) { if constexpr (ZERO) MFMA_ASM("v", "0"); else MFMA_ASM("v", "v[%c0:%c1]"); }
    else { if constexpr (ZERO) MFMA_ASM("a", "0"); else MFMA_ASM("a", "a[%c0:%c1]"); }
}
template <int Q, int G> __device__ __forceinline__ void epi_gap(float sc, __amdgpu_buffer_rsrc_t ro, unsigned ovoff, unsigned obase) {
    constexpr int E = G / 3, part = G % 3, m = E >> 2, j = E & 3, ai = Q >> 1, bj = Q & 1, V = V_EV + (E & 1) * 4, R = acc_reg(acc_idx(ai, bj, m, 0)) + j;
    constexpr bool inv = acc_in_v(acc_idx(ai, bj, m, 0));
    if constexpr (part == 0 && inv)
        asm volatile("v_mul_f32 v%c1, %0, v%c5\n\tv_mul_f32 v%c2, %0, v%c6\n\tv_mul_f32 v%c3, %0, v%c7\n\tv_mul_f32 v%c4, %0, v%c8"
                     :: "s"(sc), "i"(V), "i"(V + 1), "i"(V + 2), "i"(V + 3), "i"(R), "i"(R + 4), "i"(R + 8), "i"(R + 12));
    else if constexpr (part == 0)
        asm volatile("v_accvgpr_read_b32 v%c1, a%c5\n\tv_accvgpr_read_b32 v%c2, a%c6\n\tv_accvgpr_read_b32 v%c3, a%c7\n\tv_accvgpr_read_b32 v%c4, a%c8\n\t"
                     "v_mul_f32 v%c1, %0, v%c1\n\tv_mul_f32 v%c2, %0, v%c2"
                     :: "s"(sc), "i"(V), "i"(V + 1), "i"(V + 2), "i"(V + 3), "i"(R), "i"(R + 4), "i"(R + 8), "i"(R + 12));
    else if constexpr (part == 1) {
        if constexpr (!inv) asm volatile("v_mul_f32 v%c1, %0, v%c1\n\tv_mul_f32 v%c2, %0, v%c2" :: "s"(sc), "i"(V + 2), "i"(V + 3));
        asm volatile("v_maximum3_f32 v%c0, v%c0, 0, 0\n\tv_maximum3_f32 v%c1, v%c1, 0, 0\n\tv_maximum3_f32 v%c2, v%c2, 0, 0\n\tv_maximum3_f32 v%c3, v%c3, 0, 0"
                     :: "i"(V), "i"(V + 1), "i"(V + 2), "i"(V + 3));
    } else
        asm volatile("buffer_store_dwordx4 v[%c3:%c4], %0, %1, %2 offen offset:%c5 nt"
                     :: "v"(ovoff), "s"(ro), "s"(obase + (unsigned)(ai * 128 + m * 16 + j) * 65536u), "i"(V), "i"(V + 3), "i"(bj * 512) : "memory");
}

__global__ __launch_bounds__(256, 1) void fused_gemm(Args a) {
    __shared__ __attribute__((aligned(16))) char shm[LDS_BYTES];
    LAS char* lds = (LAS char*)shm;
    const int wid = __builtin_amdgcn_readfirstlane(threadIdx.x >> 6);
#define LANE_INIT() int tid = threadIdx.x; asm volatile("" : "+v"(tid)); const int lane = tid & 63; \
    const unsigned wv = (unsigned)((lane >> 4) * 16) * N + wid * 64 + 4 * (lane & 15)
    const int bid = blockIdx.x;
    const unsigned rank = (unsigned)a.rank & (NR - 1);
    char* const myws = a.myws;
    GAS unsigned* const ctr = (GAS unsigned*)(myws + WS_CTR + bid * 128);
#define CALL_LOAD() asm volatile("" ::: "memory"); const unsigned ctr_old = __hip_atomic_load(ctr, __ATOMIC_RELAXED, __HIP_MEMORY_SCOPE_AGENT)
#define CALL_INIT() const unsigned call = __builtin_amdgcn_readfirstlane(ctr_old) + 1u; const size_t doff = WS_DATA + (size_t)(call & 1u) * DATA_BYTES
#define CALL_ADVANCE() do { if (tid == 0) __hip_atomic_store(ctr, call, __ATOMIC_RELAXED, __HIP_MEMORY_SCOPE_AGENT); } while (0)
    const __amdgpu_buffer_rsrc_t rx = __builtin_amdgcn_make_buffer_rsrc((void*)a.x, 0, (unsigned)M * KS, 0x00020000);
    const __amdgpu_buffer_rsrc_t rw = __builtin_amdgcn_make_buffer_rsrc((void*)a.w, 0, (unsigned)KS * N, 0x00020000);
    if (bid < NCOMM) {
        LANE_INIT(); CALL_LOAD();
        const unsigned xv = (unsigned)(wid * 64 + (lane & 15)) * KS + (lane >> 4) * 16;
        const unsigned ov = wid * 8192 + lane * 16;
        unsigned rawA[16], rawB[16]; u32x4 xrA[2], xrB[2];
        unsigned u0 = (bid << 16) | a.order[bid >> 2], u1 = ((bid + NCOMM) << 16) | a.order[(bid + NCOMM) >> 2];
        unsigned u2 = ((bid + 2 * NCOMM) << 16) | a.order[(bid + 2 * NCOMM) >> 2], u3 = ((bid + 3 * NCOMM) << 16) | a.order[(bid + 3 * NCOMM) >> 2], pfl = 0;
#define COMM_LOAD(raw, xr, u) do { const unsigned _p = (u) & 255u, _k = ((u) >> 8) & 3u, _h = ((u) >> 16) & 1u, _f = ((u) >> 17) & 1u; \
            const bool _x = _p < 16; \
            const unsigned sox = _x ? (_p * 256u + _f * 32u) * KS + _k * 128u + _h * 64u : 0u, sow = _x ? 0u : (_k * 128u + _h * 64u) * N + (_p - 16u) * 256u; \
            const __amdgpu_buffer_rsrc_t rsx = __builtin_amdgcn_make_buffer_rsrc((void*)a.x, 0, _x ? (unsigned)M * KS : 0u, 0x00020000); \
            const __amdgpu_buffer_rsrc_t rsw = __builtin_amdgcn_make_buffer_rsrc((void*)a.w, 0, _x ? 0u : (unsigned)KS * N, 0x00020000); \
            _Pragma("unroll") for (int i = 0; i < 2; ++i) xr[i] = __builtin_amdgcn_raw_buffer_load_b128(rsx, xv + i * 16 * KS, sox, 0); \
            _Pragma("unroll") for (int i = 0; i < 16; ++i) raw[i] = __builtin_amdgcn_raw_buffer_load_b32(rsw, wv, sow + i * N, 0); } while (0)
        COMM_LOAD(rawA, xrA, u0); COMM_LOAD(rawB, xrB, u1);
        { const unsigned e0 = a.order[tid], e1 = a.order[256 + tid < NBLOCK ? 256 + tid : NBLOCK - 1];
          *(LAS unsigned*)(lds + tid * 4) = e0; *(LAS unsigned*)(lds + 1024 + tid * 4) = e1; }
        CALL_INIT();
        __amdgpu_buffer_rsrc_t rd[NR - 1];
#pragma unroll
        for (int i = 0; i < NR - 1; ++i) rd[i] = __builtin_amdgcn_make_buffer_rsrc(a.peer[(unsigned)(i + 2 * wid + bid) % (NR - 1)] + doff, 0, DATA_BYTES, 0x00020000);
        GAS unsigned* fl = (GAS unsigned*)(a.peer[lane < NR - 1 ? lane : 0] + WS_FLAGS) + rank * 4;
        GAS unsigned* const tmo = (GAS unsigned*)(myws + WS_TMO);
        char* const tickp = myws + WS_TICKET;
        const unsigned tbase = (call - 1u) * NUNIT - 4u * NCOMM, one = 1u;
        bool bad = false;
#define SPIN_UNTIL(cond) do { const unsigned long long _t0 = __builtin_amdgcn_s_memrealtime(); \
            while (__builtin_amdgcn_ballot_w64(!(cond)) != 0ull) { \
                if (__builtin_amdgcn_readfirstlane(__hip_atomic_load(tmo, __ATOMIC_RELAXED, __HIP_MEMORY_SCOPE_AGENT)) == call \
                    || __builtin_amdgcn_s_memrealtime() - _t0 > DEADLINE_TICKS) { \
                    __hip_atomic_store(tmo, call, __ATOMIC_RELAXED, __HIP_MEMORY_SCOPE_AGENT); bad = true; break; } \
                __builtin_amdgcn_s_sleep(8); } } while (0)
#define COMM_ST(o, vo, so) do { _Pragma("unroll") for (int d = 0; d < NR - 1; ++d) __builtin_amdgcn_raw_buffer_store_b128(o, rd[d], vo, so, AUX_SYS); ST_PAD(); } while (0)
#define UNIT_OFF(u) (((((u) >> 8) & 3u) * NR + rank) * SLAB + ((u) & 255u) * BLK + (((u) >> 16) & 2u) * 2048u + (((u) >> 16) & 1u) * 1024u)
        const __amdgpu_buffer_rsrc_t rl = __builtin_amdgcn_make_buffer_rsrc(myws + doff, 0, DATA_BYTES, 0x00020000);
#define COMM_STEP(raw, xr, first, word, craw, lraw) do { \
            u32x4 o0 = xr[0], o1 = xr[1]; \
            if (craw) { const unsigned sel = (u0 & 0x20000u) ? 0x07030602u : 0x05010400u; \
              _Pragma("unroll") for (int b = 0; b < 4; ++b) { \
                    const unsigned t0 = __builtin_amdgcn_perm(raw[4 * b + 1], raw[4 * b], sel), t2 = __builtin_amdgcn_perm(raw[4 * b + 3], raw[4 * b + 2], sel); \
                    o0[b] |= __builtin_amdgcn_perm(t2, t0, 0x05040100u); o1[b] |= __builtin_amdgcn_perm(t2, t0, 0x07060302u); } } \
            asm volatile("" : "+v"(o0), "+v"(o1)); \
            const unsigned sd = UNIT_OFF(u0); \
            if (lraw) COMM_LOAD(raw, xr, u2); \
            else { xr[0] = __builtin_amdgcn_raw_buffer_load_b128(rl, ov, UNIT_OFF(u2), AUX_SYS); xr[1] = __builtin_amdgcn_raw_buffer_load_b128(rl, ov + 2048, UNIT_OFF(u2), AUX_SYS); } \
            SCHED; COMM_ST(o0, ov, sd); if (lraw) WAIT_V(25); else WAIT_V(9); \
            if (!(first) && tid == 0) { unsigned _t; asm volatile("v_subrev_u32 %0, %1, v255" : "=v"(_t) : "s"(tbase)); \
                *(volatile LAS unsigned*)(lds + LDS_MISC + (word)) = (_t << 16) | *(volatile LAS unsigned*)(lds + ((_t & 2047u) & ~3u)); } \
            WAIT_L(0); BAR; \
            unsigned un = 0; \
            if (!(first)) { un = __builtin_amdgcn_readfirstlane(*(volatile LAS unsigned*)(lds + LDS_MISC + (word))); \
                if (tid < NR - 1) __hip_atomic_store(fl + pfl, call, __ATOMIC_RELAXED, __HIP_MEMORY_SCOPE_SYSTEM); } \
            else CALL_ADVANCE(); \
              \
            if (tid == 0) asm volatile("global_atomic_add v255, %0, %1, off sc0" :: "v"(tickp), "v"(one) : "memory", "v255"); \
            COMM_ST(o1, ov + 2048, sd); \
            pfl = (u0 & 255u) * 128u + ((u0 >> 8) & 3u) * (NR * 4u) + ((u0 >> 16) & 3u); \
            u0 = u1; u1 = u2; if (first) { u2 = u3; } else { u2 = un; } } while (0)
#define COMM_END ((u0 >> 16) >= NUNIT)
        COMM_STEP(rawA, xrA, true, 8, true, true);
        char* dout[NR - 1]; unsigned shoff[NSHARE]; float* outp = a.out;
#pragma unroll
        for (int i = 0; i < NR - 1; ++i) { dout[i] = a.peer_out[(unsigned)(i + 2 * wid + bid) % (NR - 1)]; asm volatile("" : "+s"(dout[i])); }
#pragma unroll
        for (int i = 0; i < NSHARE; ++i) { shoff[i] = a.shared[i]; asm volatile("" : "+s"(shoff[i])); }
        asm volatile("" : "+s"(outp));
        const __amdgpu_buffer_rsrc_t ro = __builtin_amdgcn_make_buffer_rsrc((void*)outp, 0, (unsigned)M * N * 4u, 0x00020000);
        do {
            for (;;) {
                COMM_STEP(rawB, xrB, false, 12, true, true); if (COMM_END) break;
                COMM_STEP(rawA, xrA, false, 8, true, true); if ((u0 >> 16) >= RAW_UNITS + RAW_STEP * bid) break;
            }
            if (COMM_END) break;
            SPIN_UNTIL(__hip_atomic_load((GAS unsigned*)(myws + WS_RETILE), __ATOMIC_RELAXED, __HIP_MEMORY_SCOPE_AGENT) >= call * (2u * NPANEL));
            COMM_STEP(rawB, xrB, false, 12, true, false); if (COMM_END) break;
            COMM_STEP(rawA, xrA, false, 8, true, false); if (COMM_END) break;
            for (;;) {
                COMM_STEP(rawB, xrB, false, 12, false, false); if (COMM_END) break;
                COMM_STEP(rawA, xrA, false, 8, false, false); if (COMM_END) break;
            }
        } while (0);
#define TDONE() (__hip_atomic_load((GAS unsigned*)(myws + WS_TDONE) + (lane < NSHARE ? lane : 0), __ATOMIC_RELAXED, __HIP_MEMORY_SCOPE_AGENT) >= call)
#define PUBLISH() do { BAR; if (tid < NR - 1) __hip_atomic_store(fl + pfl, call, __ATOMIC_RELAXED, __HIP_MEMORY_SCOPE_SYSTEM); } while (0)
        const bool ready = __builtin_amdgcn_ballot_w64(!TDONE()) == 0ull;
        WAIT_V(0);
        if (!ready) { PUBLISH(); SPIN_UNTIL(TDONE()); }
        __builtin_amdgcn_fence(__ATOMIC_ACQUIRE, "agent");
        const unsigned fv = (unsigned)((bid * 4 + wid) * 4) * 65536u + lane * 16;
        u32x4 fw[NSHARE][4];
#pragma unroll
        for (int t = 0; t < NSHARE; ++t)
#pragma unroll
            for (int i = 0; i < 4; ++i) { fw[t][i] = __builtin_amdgcn_raw_buffer_load_b128(ro, fv, shoff[t] + i * 65536u, AUX_SYS); SCHED; }
        if (ready) PUBLISH();
        if (bad) fw[0][0] = (u32x4){0x7fc00000u, 0x7fc00000u, 0x7fc00000u, 0x7fc00000u};
#pragma unroll
        for (int i = 0; i < NR - 1; ++i) rd[i] = __builtin_amdgcn_make_buffer_rsrc(dout[i], 0, (unsigned)M * N * 4u, 0x00020000);
#pragma unroll
        for (int t = 0; t < NSHARE; ++t)
#pragma unroll
            for (int i = 0; i < 4; ++i) { COMM_ST(fw[t][i], fv, shoff[t] + i * 65536u); }
        WAIT_V(0); BAR;
        if (wid == 0) {
            if (lane < NR - 1) __hip_atomic_store((GAS unsigned*)(a.peer[lane] + WS_FWD) + rank * NCOMM + bid, call, __ATOMIC_RELAXED, __HIP_MEMORY_SCOPE_SYSTEM);
            SPIN_UNTIL(__hip_atomic_load((GAS unsigned*)(myws + WS_FWD) + ((rank + 1u + (lane < NR - 1 ? lane : 0)) & (NR - 1)) * NCOMM + bid, __ATOMIC_RELAXED, __HIP_MEMORY_SCOPE_SYSTEM) >= call);
            if (bad && lane == 0) a.out[(shoff[0] + fv) / 4] = __builtin_nanf("");
        }
        return;
    }

    LANE_INIT(); CALL_LOAD();
    const int gw = bid - NCOMM;
    const int wr = wid >> 1, wc = wid & 1, fr = lane & 15, fq = lane >> 4;
    constexpr int NLOC = 2 * NPANEL;
    const int lw = gw - (NWG - NCOMM - NLOC);
    unsigned raw[4][16];
    if (lw >= 0) {
        const unsigned h = lw & 1, p = lw >> 1;
        if (p < 16) {
            const unsigned xv = (unsigned)(wid * 64 + (lane & 15)) * KS + h * 64 + (lane >> 4) * 16;
#pragma unroll
            for (int i = 0; i < 4; ++i)
#pragma unroll
                for (int j = 0; j < 4; ++j) { const u32x4 v = __builtin_amdgcn_raw_buffer_load_b128(rx, xv + j * 16 * KS, p * (256u * KS) + i * 128u, 0);
                    raw[i][4 * j] = v[0]; raw[i][4 * j + 1] = v[1]; raw[i][4 * j + 2] = v[2]; raw[i][4 * j + 3] = v[3]; }
        } else {
#pragma unroll
            for (int i = 0; i < 4; ++i)
#pragma unroll
                for (int j = 0; j < 16; ++j) raw[i][j] = __builtin_amdgcn_raw_buffer_load_b32(rw, wv, (i * 128u + h * 64u + j) * N + (p - 16u) * 256u, 0);
        }
    }
    if (tid == 0) *(LAS unsigned*)(lds + LDS_MISC + 4) = 0u;
    CALL_INIT();
    const __amdgpu_buffer_rsrc_t rin = __builtin_amdgcn_make_buffer_rsrc(myws + doff, 0, DATA_BYTES, 0x00020000);
    const __amdgpu_buffer_rsrc_t rfl = __builtin_amdgcn_make_buffer_rsrc(myws + WS_FLAGS, 0, FLAG_BYTES, 0x00020000);
    GAS unsigned* tmo = (GAS unsigned*)(myws + WS_TMO);
    if (lw >= 0) {
        const unsigned h = lw & 1, p = lw >> 1;
#pragma unroll
        for (int i = 0; i < 4; ++i) {
            const unsigned so = (i * NR + rank) * SLAB + p * BLK + h * 1024;
            if (p < 16) {
#pragma unroll
                for (int j = 0; j < 4; ++j) __builtin_amdgcn_raw_buffer_store_b128((u32x4){raw[i][4 * j], raw[i][4 * j + 1], raw[i][4 * j + 2], raw[i][4 * j + 3]}, rin, lane * 16, so + (wid * 4 + j) * 2048, AUX_SYS);
                ST_PAD();
            } else {
                u32x4 o[4];
#pragma unroll
                for (int b = 0; b < 4; ++b) {
                    const unsigned r0 = raw[i][4 * b], r1 = raw[i][4 * b + 1], r2 = raw[i][4 * b + 2], r3 = raw[i][4 * b + 3];
                    const unsigned t0 = __builtin_amdgcn_perm(r1, r0, 0x05010400), t1 = __builtin_amdgcn_perm(r1, r0, 0x07030602);
                    const unsigned t2 = __builtin_amdgcn_perm(r3, r2, 0x05010400), t3 = __builtin_amdgcn_perm(r3, r2, 0x07030602);
                    o[0][b] = __builtin_amdgcn_perm(t2, t0, 0x05040100); o[1][b] = __builtin_amdgcn_perm(t2, t0, 0x07060302);
                    o[2][b] = __builtin_amdgcn_perm(t3, t1, 0x05040100); o[3][b] = __builtin_amdgcn_perm(t3, t1, 0x07060302);
                }
#pragma unroll
                for (int c = 0; c < 4; ++c) __builtin_amdgcn_raw_buffer_store_b128(o[c], rin, lane * 16, so + wid * 8192 + c * 2048, AUX_SYS);
                ST_PAD();
            }
        }
    }
    asm volatile("s_waitcnt vmcnt(0) lgkmcnt(0)" ::: "memory"); BAR;
    CALL_ADVANCE();
    if (lw >= 0 && tid < 8)
        __hip_atomic_store((GAS unsigned*)(myws + WS_FLAGS) + (lw >> 1) * 128u + ((tid >> 1) * NR + rank) * 4u + (tid & 1) * 2 + (lw & 1), call, __ATOMIC_RELAXED, __HIP_MEMORY_SCOPE_SYSTEM);
    if (lw >= 0 && tid == 0) __hip_atomic_fetch_add((GAS unsigned*)(myws + WS_RETILE), 1u, __ATOMIC_RELAXED, __HIP_MEMORY_SCOPE_AGENT);
    unsigned flip = 0;
    if (gw == 0) {
        unsigned xcc; asm volatile("s_getreg_b32 %0, hwreg(HW_REG_XCC_ID)" : "=s"(xcc));
        if (tid == 0) __hip_atomic_store((GAS unsigned*)(myws + WS_FLIP), (call << 1) | (xcc & 1u), __ATOMIC_RELAXED, __HIP_MEMORY_SCOPE_AGENT);
        flip = (xcc & 1u);
    } else {
        const unsigned long long t0 = __builtin_amdgcn_s_memrealtime();
        for (;;) {
            flip = __builtin_amdgcn_readfirstlane(__hip_atomic_load((GAS unsigned*)(myws + WS_FLIP), __ATOMIC_RELAXED, __HIP_MEMORY_SCOPE_AGENT));
            if ((flip >> 1) == (call & 0x7fffffffu)) break;
            if (__builtin_amdgcn_readfirstlane(__hip_atomic_load(tmo, __ATOMIC_RELAXED, __HIP_MEMORY_SCOPE_AGENT)) == call || __builtin_amdgcn_s_memrealtime() - t0 > DEADLINE_TICKS) {
                __hip_atomic_store(tmo, call, __ATOMIC_RELAXED, __HIP_MEMORY_SCOPE_AGENT); *(volatile LAS unsigned*)(lds + LDS_MISC + 4) = 1u; break; }
            __builtin_amdgcn_s_sleep(8);
        }
        WAIT_L(0); BAR;
        flip = __builtin_amdgcn_readfirstlane(*(volatile LAS unsigned*)(lds + LDS_MISC + 4)) != 0u ? 0u : (flip & 1u);
    }
    const unsigned long long tl = a.tiles[gw ^ __builtin_amdgcn_readfirstlane(flip)];
    const int ntiles = (int)(tl >> 50) & 7;
    const unsigned koff0 = (unsigned)(tl >> 56) & 31u;
    const float scale = __builtin_bit_cast(float, __builtin_amdgcn_readfirstlane(__builtin_bit_cast(int, a.sx[0] * a.sw[0])));
    const unsigned voff = wid * 4096 + lane * 16, ldsw = wid * 4096;
    const unsigned ldsb = (unsigned)(unsigned long)lds;
    unsigned aoff = ldsb + wr * 8192 + lane * 16, boff = ldsb + 4 * HB + wc * 8192 + lane * 16;
    asm volatile("" : "+v"(aoff), "+v"(boff));
    const unsigned ovoff = (unsigned)(wr * 64 + 4 * fq) * 65536u + wc * 256 + fr * 16;
#define TILE(i) ((unsigned)(tl >> (10 * (i))) & 0x3ffu)
#define SA(b, h) (((b) * 2 + (h)) * HB)
#define SB(b, h) ((4 + (b) * 2 + (h)) * HB)
#define G1(bufoff, soff, i) __builtin_amdgcn_raw_ptr_buffer_load_lds(rin, (LAS void*)(lds + (bufoff) + ldsw), 16, voff, (soff), (i) * 1024, AUX_SYS)
#define STAGE4(bufoff, soff) do { G1(bufoff, soff, 0); G1(bufoff, soff, 1); G1(bufoff, soff, 2); G1(bufoff, soff, 3); } while (0)
#define RD(U, i, base, off) asm volatile("ds_read_b128 a[%c1:%c2], %0 offset:%c3" :: "v"(base), "i"((U) + 4 * (i)), "i"((U) + 4 * (i) + 3), "i"((off) + (i) * 1024) : "memory")
#define MF(Z, ai, bj, m, n, RA, RB) mfma<acc_idx(ai, bj, m, n), Z, (RA) + 8 * (m), (RB) + 8 * (n)>()
#define EPG(eq, g) do { if constexpr ((eq) >= 0) epi_gap<(eq) < 0 ? 0 : (eq), (g)>(esc, erout, ovoff, eobase); } while (0)
#define QUAD(Z, ai, bj, RA, RB, RN, rdb, rdoff, stoff, soff, vm, eq, g0, hook) do { u32x4 _fv; unsigned _pi; \
        if constexpr ((vm) >= 0) { asm volatile("s_waitcnt vmcnt(%0) lgkmcnt(0)" :: "n"((vm) < 0 ? 0 : (vm)) : "memory"); BAR; } else WAIT_L(0); SCHED; \
        MF(Z, ai, bj, 0, 0, RA, RB); RD(RN, 0, rdb, rdoff); EPG(eq, g0 + 0); SCHED; \
        MF(Z, ai, bj, 0, 1, RA, RB); RD(RN, 1, rdb, rdoff); EPG(eq, g0 + 1); SCHED; \
        MF(Z, ai, bj, 0, 2, RA, RB); RD(RN, 2, rdb, rdoff); EPG(eq, g0 + 2); if constexpr ((hook) >= 3) PICK_A(_pi); SCHED; \
        MF(Z, ai, bj, 0, 3, RA, RB); RD(RN, 3, rdb, rdoff); EPG(eq, g0 + 3); SCHED; \
        MF(Z, ai, bj, 1, 0, RA, RB); RD(RN, 4, rdb, rdoff); EPG(eq, g0 + 4); if constexpr ((hook) == 3) PICK_B(_pi, kaA, kaB); if constexpr ((hook) == 4) PICK_B(_pi, kbA, kbB); SCHED; \
        MF(Z, ai, bj, 1, 1, RA, RB); RD(RN, 5, rdb, rdoff); EPG(eq, g0 + 5); SCHED; \
        MF(Z, ai, bj, 1, 2, RA, RB); RD(RN, 6, rdb, rdoff); EPG(eq, g0 + 6); SCHED; \
        MF(Z, ai, bj, 1, 3, RA, RB); RD(RN, 7, rdb, rdoff); EPG(eq, g0 + 7); SCHED; \
        MF(Z, ai, bj, 2, 0, RA, RB); G1(stoff, soff, 0); EPG(eq, g0 + 8); SCHED; \
        MF(Z, ai, bj, 2, 1, RA, RB); EPG(eq, g0 + 9); if constexpr ((hook) == 2) _fv = FLAG_LOAD(0); SCHED; \
        MF(Z, ai, bj, 2, 2, RA, RB); G1(stoff, soff, 1); EPG(eq, g0 + 10); SCHED; \
        MF(Z, ai, bj, 2, 3, RA, RB); EPG(eq, g0 + 11); SCHED; \
        MF(Z, ai, bj, 3, 0, RA, RB); G1(stoff, soff, 2); EPG(eq, g0 + 12); SCHED; \
        MF(Z, ai, bj, 3, 1, RA, RB); EPG(eq, g0 + 13); if constexpr ((hook) == 2) FLAG_MASK(_fv); SCHED; \
        MF(Z, ai, bj, 3, 2, RA, RB); G1(stoff, soff, 3); EPG(eq, g0 + 14); SCHED; \
        MF(Z, ai, bj, 3, 3, RA, RB); EPG(eq, g0 + 15); if constexpr ((hook) == 1) FLAG_FETCH(0); SCHED; } while (0)

    unsigned rmask = 0, done = 0, kaA, kaB, kbA, kbB;
#define BASE_A(tile) (((tile) & 15u) * BLK)
#define BASE_B(tile) ((16u + ((tile) >> 4)) * BLK)
#define FLAG_SOFF(tile) ((wid < 2 ? ((tile) & 15u) : 16u + ((tile) >> 4)) * 512u + (wid & 1) * 256u)
    unsigned pA = BASE_A(TILE(0)), pB = BASE_B(TILE(0)), psoff = FLAG_SOFF(TILE(0)), fsoff = psoff;
    const unsigned fvoff = lane * 4;
    int froff = LDS_FLAGS + ((lane & 32) + ((lane + koff0) & 31)) * 16; asm volatile("" : "+v"(froff));
#define FLAG_FETCH_AT(slot, so) __builtin_amdgcn_raw_ptr_buffer_load_lds(rfl, (LAS void*)(lds + LDS_FLAGS + (slot) * 1024 + wid * 256), 4, fvoff, so, 0, AUX_SYS)
#define FLAG_FETCH(slot) FLAG_FETCH_AT(slot, fsoff)
#define FLAG_LOAD(slot) (*(volatile LAS u32x4*)(lds + froff + (slot) * 1024))
#define FLAG_MASK(_v) do { const unsigned _a = _v[0] < _v[1] ? _v[0] : _v[1], _b = _v[2] < _v[3] ? _v[2] : _v[3]; \
            const unsigned long long _m = __builtin_amdgcn_ballot_w64((_a < _b ? _a : _b) >= call); \
            rmask |= (unsigned)_m & (unsigned)(_m >> 32); } while (0)
#define PICK_A(_i) do { \
            if (__builtin_expect((rmask & ~done) == 0u, 0)) {        \
                const unsigned long long _t0 = __builtin_amdgcn_s_memrealtime(); \
                for (;;) { \
                    FLAG_FETCH_AT(1, psoff); \
                    if (wid == 0 && __builtin_amdgcn_s_memrealtime() - _t0 > 10000ull        \
                        && (__builtin_amdgcn_readfirstlane(__hip_atomic_load(tmo, __ATOMIC_RELAXED, __HIP_MEMORY_SCOPE_AGENT)) == call \
                            || __builtin_amdgcn_s_memrealtime() - _t0 > DEADLINE_TICKS)) { \
                        __hip_atomic_store(tmo, call, __ATOMIC_RELAXED, __HIP_MEMORY_SCOPE_AGENT); \
                        *(volatile LAS unsigned*)(lds + LDS_MISC + 4) = 1u; } \
                    asm volatile("s_waitcnt vmcnt(0) lgkmcnt(0)" ::: "memory"); BAR; \
                    const u32x4 _pv = FLAG_LOAD(1); FLAG_MASK(_pv); \
                    if (__builtin_amdgcn_readfirstlane(*(volatile LAS unsigned*)(lds + LDS_MISC + 4)) != 0u) rmask = ~0u; \
                    WAIT_L(0); BAR; \
                    if ((rmask & ~done) != 0u) break; \
                    __builtin_amdgcn_s_sleep(8); \
                } \
            } \
            _i = __builtin_ctz(rmask & ~done); done |= 1u << _i; } while (0)
#define PICK_B(_i, oA, oB) do { const unsigned _q = ((_i + koff0) & 31u) * SLAB; oA = _q + pA; oB = _q + pB; asm volatile("" : "+s"(oA), "+s"(oB)); } while (0)
#define PICK(oA, oB) do { unsigned _pi; PICK_A(_pi); PICK_B(_pi, oA, oB); } while (0)
#define ITER(KIND, sw, S1, S2, S3, S4) do { \
        if (sw) fsoff = nsoff; \
        QUAD(KIND == 0, 0, 0, S1, V_B0A, V_B0B, boff, SA(1, 0), SA(1, 0), kbA, (KIND == 0 ? 38 : KIND == 3 ? 27 : 16), (KIND == 0 ? 3 : -1), 0, 4); \
        QUAD(false, 0, 0, S2, V_B0B, S3, boff, SA(0, 1), SB(0, 0), kaB, -1, (KIND == 0 ? 3 : -1), 16, 1); \
        if constexpr (KIND == 2) { \
            esc = *(volatile LAS unsigned*)(lds + LDS_MISC + 4) != 0u ? __builtin_nanf("") : scale; \
            esc = __builtin_bit_cast(float, __builtin_amdgcn_readfirstlane(__builtin_bit_cast(int, esc))); } \
        QUAD(KIND == 0, 0, 1, S1, S3, S4, boff, SA(1, 1), SB(1, 0), kbB, (KIND == 0 ? 38 : 17), (KIND == 0 ? 3 : KIND == 2 ? 0 : -1), (KIND == 0 ? 32 : 0), 0); \
        QUAD(false, 0, 1, S2, S4, S1, aoff, SA(0, 1), SB(0, 1), kaB + HB, -1, (KIND == 0 ? 2 : KIND == 2 ? 0 : -1), (KIND == 0 ? 0 : 16), 0); \
        QUAD(KIND == 0, 1, 1, S1, S3, S2, aoff, SA(1, 1), SB(1, 1), kbB + HB, (KIND == 0 ? 38 : KIND == 2 ? 27 : 17), (KIND == 0 ? 2 : KIND == 2 ? 0 : -1), (KIND == 0 ? 16 : 32), 0); \
        QUAD(false, 1, 1, S2, S4, S3, aoff, SA(0, 0), SA(0, 1), kaA + HB, -1, (KIND == 0 ? 2 : KIND == 2 ? 1 : -1), (KIND == 0 ? 32 : 0), 0); \
        if (sw) { done = 0; rmask = nmask; pA = nA; pB = nB; psoff = nsoff; } \
        QUAD(KIND == 0, 1, 0, S1, V_B0A, S4, aoff, SA(1, 0), SA(1, 1), kbA + HB, (KIND == 0 ? 38 : KIND == 2 ? 37 : 16), (KIND == 2 ? 1 : -1), 16, 2); \
        QUAD(false, 1, 0, S2, V_B0B, V_B0A, boff, SA(0, 0), SA(0, 0), kaA, -1, (KIND == 2 ? 1 : -1), 32, 3); } while (0)

    PICK(kaA, kaB);
    STAGE4(SA(0, 0), kaA); STAGE4(SB(0, 0), kaB); STAGE4(SB(0, 1), kaB + HB); STAGE4(SA(0, 1), kaA + HB);
    PICK(kbA, kbB);
    STAGE4(SA(1, 0), kbA); STAGE4(SB(1, 0), kbB); STAGE4(SB(1, 1), kbB + HB); STAGE4(SA(1, 1), kbA + HB);
    PICK(kaA, kaB);
    WAIT_V(12); BAR;
    asm volatile("; GEMM_REGS_BEGIN" ::: "memory", RCLOB);
#define RD8(U, base, off) RD(U, 0, base, off); RD(U, 1, base, off); RD(U, 2, base, off); RD(U, 3, base, off); RD(U, 4, base, off); RD(U, 5, base, off); RD(U, 6, base, off); RD(U, 7, base, off)
    RD8(V_R1, aoff, SA(0, 0)); RD8(V_B0A, boff, SA(0, 0)); WAIT_L(0); RD8(V_R2, aoff, SA(1, 0));
    WAIT_L(0); BAR;
    STAGE4(SA(0, 0), kaA);
    WAIT_V(0);
    __amdgpu_buffer_rsrc_t erout = __builtin_amdgcn_make_buffer_rsrc((void*)a.out, 0, 0, 0x00020000);
    unsigned eobase = 0; float esc = 0.f;
    int ui = 0;
    do {
        const bool more = ui + 1 < ntiles; const unsigned nt = TILE(more ? ui + 1 : ui);
        const unsigned nA = BASE_A(nt), nB = BASE_B(nt), nsoff = FLAG_SOFF(nt), nmask = more ? 0u : ~0u;
        ITER(0, false, V_R1, V_R2, V_R3, V_R4);
        ITER(3, false, V_R3, V_R4, V_R1, V_R2);
        if (__builtin_expect(ui == (int)((tl >> 53) & 7u) && ui > 0, 0)) {
            WAIT_V(0); BAR;
            if (tid == 0) {
                __builtin_amdgcn_fence(__ATOMIC_RELEASE, "agent"); WAIT_V(0);
                __hip_atomic_store((GAS unsigned*)(myws + WS_TDONE) + (unsigned)(tl >> 61), call, __ATOMIC_RELAXED, __HIP_MEMORY_SCOPE_AGENT);
            }
        }
        for (int it = 2; it < 14; it += 2) { ITER(1, false, V_R1, V_R2, V_R3, V_R4); ITER(1, false, V_R3, V_R4, V_R1, V_R2); }
        ITER(1, true, V_R1, V_R2, V_R3, V_R4);
        erout = __builtin_amdgcn_make_buffer_rsrc((void*)a.out, 0, (unsigned)M * N * 4u, 0x00020000);
        eobase = (TILE(ui) & 15u) * (256u * 65536u) + (TILE(ui) >> 4) * 1024u;
        ITER(2, false, V_R3, V_R4, V_R1, V_R2);
    } while (++ui < ntiles);
#define EP3(q, g) EPG(q, g); EPG(q, g + 1); EPG(q, g + 2)
#define EP12(q, g) EP3(q, g); EP3(q, g + 3); EP3(q, g + 6); EP3(q, g + 9)
    EP12(3, 0); EP12(3, 12); EP12(3, 24); EP12(3, 36); EP12(2, 0); EP12(2, 12); EP12(2, 24); EP12(2, 36);
    WAIT_V(0);
    asm volatile("; GEMM_REGS_END" ::: "memory");
}

void make_tables(Args& a, int rank) {
    constexpr double P_G1X = -2.0, P_SX = -3.0, P_RX = 1.18, P_SPS = 10.0, P_LEN2 = 38.0, P_S3 = 44.0, P_TAU = 38.0, P_ZOFF = -8.0, P_KS0 = 2.0;
    constexpr int P_KZ = 25, P_RH = 1;
    double key[320]; int n = 0;
    constexpr int PZ = 15 - NZ;
    auto koff = [](int p) { return p < 4 ? 0 : p < PZ ? (int)(P_KS0 + (p - 4) * P_SPS / (PZ - 5) / 1.25 + 0.5) : P_KZ; };
    auto phase = [](int p) { return p < PZ ? p * P_SPS / (PZ - 5) : P_ZOFF; };
    auto add = [&](int panel, double start, double rate, int rot, int off) {
        for (int kt = 0; kt < 4; ++kt) {
            const double k = start + ((kt * NR + ((rank + rot) & (NR - 1)) - off) & 31) * rate + panel * 1e-6;
            int i = n++;
            for (; i > 0 && key[i - 1] > k; --i) { key[i] = key[i - 1]; a.order[i] = a.order[i - 1]; }
            key[i] = k; a.order[i] = (unsigned)((kt << 8) | panel);
        }
    };
    for (int i = 0; i < 8; ++i) { add(i, -1000.0 + P_G1X, 1.0, i, 0); add(16 + i, -1000.0, 1.0, i, 0); add(8 + i, P_SX, P_RX, i, 0); }
    for (int p = 4; p < PZ; ++p) add(16 + 8 + (p - 4), phase(p) - phase(4), P_LEN2 / 32.0, p, koff(p));
    for (int g = 0; g < 3; ++g)
        for (int p = 0; p < 15; ++p) add(16 + 19 + g * 15 + p, P_S3 + g * P_TAU + phase(p), 1.18, p, koff(p));
    for (int i = 0; i < NZ; ++i) add(16 + 19 - NZ + i, P_S3 + P_RH * P_TAU + phase(PZ), 1.18, 4 * i + 1, koff(PZ));
    const int H = 19 + P_RH * 15 + PZ;
    int smp[NSHARE], snp[NSHARE];
    for (int i = 0; i < 2 * NZ; ++i) { smp[i] = (2 * NZ * rank + i) & 15; snp[i] = 19 - NZ + ((2 * NZ * rank) >> 4); }
    smp[2 * NZ] = rank; snp[2 * NZ] = H;
    for (int i = 0; i < NSHARE; ++i) a.shared[i] = (unsigned)smp[i] * (256u * 65536u) + (unsigned)snp[i] * 1024u;
    for (int g = 0; g < NWG - NCOMM; ++g) {
        static const int seq[15] = {0, 2, 12, 11, 10, 9, 7, 5, 8, 6, 4, 14, 13, 3, 1};
        const int r = (g & 1) ? 59 - (g >> 3) : (g >> 3), p = seq[r >> 2], m = ((g & 7) >> 1) * 4 + (r & 3);
        int mp[5], np[5], nt = 0; unsigned long long tl = 0;
        if (p < 4) { mp[0] = m & 7; mp[1] = 8 + (m & 7); np[0] = np[1] = 2 * p + (m >> 3); nt = 2; }
        else if (p < PZ) { mp[0] = m; np[0] = 8 + p - 4; nt = 1; }
        for (int k = 0; k < 3; ++k) {
            if (19 + k * 15 + p == H && m < 8) {
                if (m < NSHARE) { mp[nt] = smp[m]; np[nt] = snp[m]; ++nt; tl |= ((unsigned long long)nt << 53) | ((unsigned long long)m << 61); }
            } else { mp[nt] = m; np[nt] = 19 + k * 15 + p; ++nt; }
        }
        tl |= ((unsigned long long)nt << 50) | ((unsigned long long)koff(p) << 56);
        for (int i = 0; i < nt; ++i) tl |= (unsigned long long)((np[i] << 4) | mp[i]) << (10 * i);
        a.tiles[g] = tl;
    }
}
}

extern "C" void kernel_launch(int rank, int nranks, ncclComm_t comm,
                              hipStream_t stream,
                              void* const* d_in, const int* in_sizes, int n_in,
                              void* d_out, int out_size,
                              const int* in_dtypes, int out_dtype,
                              void* const* d_out_all,
                              void* const* d_ws_all, size_t ws_bytes) {
    (void)comm; (void)in_dtypes; (void)out_dtype;
    if (nranks != NR || (unsigned)rank >= (unsigned)NR || n_in != 4 || in_sizes[0] != M * KS || in_sizes[1] != KS * N ||
        out_size != M * N || ws_bytes < WS_NEED) {
        fprintf(stderr, "kernel_launch: unexpected shapes\n");
        return;
    }
    Args a;
    a.x = (const unsigned char*)d_in[0]; a.w = (const unsigned char*)d_in[1];
    a.sx = (const float*)d_in[2]; a.sw = (const float*)d_in[3];
    a.out = (float*)d_out;
    a.myws = (char*)d_ws_all[rank];
    for (int i = 0; i < NR - 1; ++i) { a.peer[i] = (char*)d_ws_all[(rank + 1 + i) % NR]; a.peer_out[i] = (char*)d_out_all[(rank + 1 + i) % NR]; }
    a.rank = rank;
    make_tables(a, rank);
    hipLaunchKernelGGL(fused_gemm, dim3(NWG), dim3(256), 0, stream, a);
    HIP_CHECK(hipGetLastError());
}
```

```cpp
#include <hip/hip_runtime.h>
#include <rccl/rccl.h>
#include <cstdio>

#define HIP_CHECK(expr) do { hipError_t _e = (expr); if (_e != hipSuccess) { \
    fprintf(stderr, "HIP error %s at %s:%d\n", hipGetErrorString(_e), __FILE__, __LINE__); \
    return; } } while (0)

#define LAS __attribute__((address_space(3)))
#define GAS __attribute__((address_space(1)))
typedef unsigned u32x4 __attribute__((ext_vector_type(4)));

namespace {
constexpr int NR = 8;
constexpr int M = 4096, N = 16384, KS = 512;
constexpr unsigned BLK = 32768, HB = 16384;
constexpr unsigned NPANEL = 80;
constexpr unsigned SLAB = NPANEL * BLK;
constexpr unsigned DATA_BYTES = 32 * SLAB;
constexpr unsigned WS_FLAGS = 0;
constexpr unsigned FLAG_BYTES = NPANEL * 512;
constexpr unsigned WS_CTR = 65536;
constexpr unsigned WS_TMO = WS_CTR + 256 * 128;
constexpr unsigned WS_TICKET = WS_TMO + 128;
constexpr unsigned WS_TDONE = WS_TICKET + 128;
constexpr int NZ = 2;
static_assert(NZ == 2, "the senders' destination masks name column panels 17 and 18");
constexpr int NSHARE = 2 * NZ + 1;
constexpr unsigned WS_RETILE = WS_TDONE + 128;
constexpr unsigned WS_FLIP = WS_RETILE + 128;
constexpr unsigned WS_FWD = WS_FLIP + 128;
constexpr unsigned NBLOCK = 320, NUNIT = 4 * NBLOCK;
constexpr size_t WS_DATA = 1u << 20;
constexpr size_t WS_NEED = WS_DATA + 2 * (size_t)DATA_BYTES;
constexpr int NCOMM = 16, NWG = 256;
constexpr unsigned long long DEADLINE_TICKS = 50000000ull;
constexpr int AUX_SYS = 17;
constexpr unsigned RAW_UNITS = 64u, RAW_STEP = 8u;
constexpr unsigned LDS_FLAGS = 8 * HB;
constexpr unsigned LDS_MISC = LDS_FLAGS + 2 * 1024;
constexpr unsigned LDS_BYTES = LDS_MISC + 64;

struct Args {
    const unsigned char* x; const unsigned char* w; const float* sx; const float* sw; float* out;
    char* myws; char* peer[NR - 1];
    char* peer_out[NR - 1];
    int rank;
    unsigned shared[NSHARE];
    unsigned order[NBLOCK];
    unsigned long long tiles[NWG - NCOMM];
};

#define WAIT_V(n) asm volatile("s_waitcnt vmcnt(" #n ")" ::: "memory")
#define WAIT_L(n) asm volatile("s_waitcnt lgkmcnt(" #n ")" ::: "memory")
#define BAR __builtin_amdgcn_s_barrier()
#define SCHED __builtin_amdgcn_sched_barrier(0)
#define ST_PAD() do { SCHED; asm volatile("s_nop 1"); SCHED; } while (0)

#define RCLOB "a0","a1","a2","a3","a4","a5","a6","a7","a8","a9","a10","a11","a12","a13","a14","a15","a16","a17","a18","a19","a20","a21","a22","a23","a24","a25","a26","a27","a28","a29","a30","a31","a32","a33","a34","a35","a36","a37","a38","a39","a40","a41","a42","a43","a44","a45","a46","a47","a48","a49","a50","a51","a52","a53","a54","a55","a56","a57","a58","a59","a60","a61","a62","a63","a64","a65","a66","a67","a68","a69","a70","a71","a72","a73","a74","a75","a76","a77","a78","a79","a80","a81","a82","a83","a84","a85","a86","a87","a88","a89","a90","a91","a92","a93","a94","a95","a96","a97","a98","a99","a100","a101","a102","a103","a104","a105","a106","a107","a108","a109","a110","a111","a112","a113","a114","a115","a116","a117","a118","a119","a120","a121","a122","a123","a124","a125","a126","a127","a128","a129","a130","a131","a132","a133","a134","a135","a136","a137","a138","a139","a140","a141","a142","a143","a144","a145","a146","a147","a148","a149","a150","a151","a152","a153","a154","a155","a156","a157","a158","a159","a160","a161","a162","a163","a164","a165","a166","a167","a168","a169","a170","a171","a172","a173","a174","a175","a176","a177","a178","a179","a180","a181","a182","a183","a184","a185","a186","a187","a188","a189","a190","a191","a192","a193","a194","a195","a196","a197","a198","a199","a200","a201","a202","a203","a204","a205","a206","a207","a208","a209","a210","a211","a212","a213","a214","a215","a216","a217","a218","a219","a220","a221","a222","a223","v24","v25","v26","v27","v28","v29","v30","v31","v32","v33","v34","v35","v36","v37","v38","v39","v40","v41","v42","v43","v44","v45","v46","v47","v48","v49","v50","v51","v52","v53","v54","v55","v56","v57","v58","v59","v60","v61","v62","v63","v64","v65","v66","v67","v68","v69","v70","v71","v72","v73","v74","v75","v76","v77","v78","v79","v80","v81","v82","v83","v84","v85","v86","v87","v88","v89","v90","v91","v92","v93","v94","v95","v96","v97","v98","v99","v100","v101","v102","v103","v104","v105","v106","v107","v108","v109","v110","v111","v112","v113","v114","v115","v116","v117","v118","v119","v120","v121","v122","v123","v124","v125","v126","v127","v128","v129","v130","v131","v132","v133","v134","v135","v136","v137","v138","v139","v140","v141","v142","v143","v144","v145","v146","v147","v148","v149","v150","v151","v152","v153","v154","v155","v156","v157","v158","v159","v160","v161","v162","v163","v164","v165","v166","v167","v168","v169","v170","v171","v172","v173","v174","v175","v176","v177","v178","v179","v180","v181","v182","v183","v184","v185","v186","v187","v188","v189","v190","v191","v192","v193","v194","v195","v196","v197","v198","v199","v200","v201","v202","v203","v204","v205","v206","v207","v208","v209","v210","v211","v212","v213","v214","v215","v216","v217","v218","v219","v220","v221","v222","v223","v224","v225","v226","v227","v228","v229","v230","v231","v232","v233","v234","v235","v236","v237","v238","v239","v240","v241","v242","v243","v244","v245","v246","v247","v248","v249","v250","v251","v252","v253","v254","v255"
constexpr int V_EV = 24, V_R1 = 0, V_R2 = 32, V_R3 = 64, V_R4 = 96, V_B0A = 128, V_B0B = 160;
constexpr int acc_idx(int ai, int bj, int m, int n) { return ((ai * 2 + bj) * 4 + m) * 4 + n; }
constexpr bool acc_in_v(int f) { return f < 56; }
constexpr int acc_reg(int f) { return f < 56 ? 32 + 4 * f : 192 + 4 * (f - 56); }
#define MFMA_ASM(D, C, ...) asm volatile("v_mfma_f32_16x16x128_f8f6f4 " D "[%c0:%c1], a[%c2:%c3], a[%c4:%c5], " C " cbsz:1 blgp:1" \
                                         :: "i"(R), "i"(R + 3), "i"(X), "i"(X + 7), "i"(Y), "i"(Y + 7) __VA_ARGS__)
template <int F, bool ZERO, int X, int Y> __device__ __forceinline__ void mfma() {
    constexpr int R = acc_reg(F);
    if constexpr ((F & 15) == 0) { if constexpr (ZERO) MFMA_ASM("v", "0", : RCLOB); else MFMA_ASM("v", "v[%c0:%c1]", : RCLOB); }
    else if constexpr (acc_in_v(F)) { if constexpr (ZERO) MFMA_ASM("v", "0"); else MFMA_ASM("v", "v[%c0:%c1]"); }
    else { if constexpr (ZERO) MFMA_ASM("a", "0"); else MFMA_ASM("a", "a[%c0:%c1]"); }
}
template <int Q, int G> __device__ __forceinline__ void epi_gap(float sc, __amdgpu_buffer_rsrc_t ro, unsigned ovoff, unsigned obase) {
    constexpr int E = G / 3, part = G % 3, m = E >> 2, j = E & 3, ai = Q >> 1, bj = Q & 1, V = V_EV + (E & 1) * 4, R = acc_reg(acc_idx(ai, bj, m, 0)) + j;
    constexpr bool inv = acc_in_v(acc_idx(ai, bj, m, 0));
    if constexpr (part == 0 && inv)
        asm volatile("v_mul_f32 v%c1, %0, v%c5\n\tv_mul_f32 v%c2, %0, v%c6\n\tv_mul_f32 v%c3, %0, v%c7\n\tv_mul_f32 v%c4, %0, v%c8"
                     :: "s"(sc), "i"(V), "i"(V + 1), "i"(V + 2), "i"(V + 3), "i"(R), "i"(R + 4), "i"(R + 8), "i"(R + 12));
    else if constexpr (part == 0)
        asm volatile("v_accvgpr_read_b32 v%c1, a%c5\n\tv_accvgpr_read_b32 v%c2, a%c6\n\tv_accvgpr_read_b32 v%c3, a%c7\n\tv_accvgpr_read_b32 v%c4, a%c8\n\t"
                     "v_mul_f32 v%c1, %0, v%c1\n\tv_mul_f32 v%c2, %0, v%c2"
                     :: "s"(sc), "i"(V), "i"(V + 1), "i"(V + 2), "i"(V + 3), "i"(R), "i"(R + 4), "i"(R + 8), "i"(R + 12));
    else if constexpr (part == 1) {
        if constexpr (!inv) asm volatile("v_mul_f32 v%c1, %0, v%c1\n\tv_mul_f32 v%c2, %0, v%c2" :: "s"(sc), "i"(V + 2), "i"(V + 3));
        asm volatile("v_maximum3_f32 v%c0, v%c0, 0, 0\n\tv_maximum3_f32 v%c1, v%c1, 0, 0\n\tv_maximum3_f32 v%c2, v%c2, 0, 0\n\tv_maximum3_f32 v%c3, v%c3, 0, 0"
                     :: "i"(V), "i"(V + 1), "i"(V + 2), "i"(V + 3));
    } else
        asm volatile("buffer_store_dwordx4 v[%c3:%c4], %0, %1, %2 offen offset:%c5 nt"
                     :: "v"(ovoff), "s"(ro), "s"(obase + (unsigned)(ai * 128 + m * 16 + j) * 65536u), "i"(V), "i"(V + 3), "i"(bj * 512) : "memory");
}

__global__ __launch_bounds__(256, 1) void fused_gemm(Args a) {
    __shared__ __attribute__((aligned(16))) char shm[LDS_BYTES];
    LAS char* lds = (LAS char*)shm;
    const int wid = __builtin_amdgcn_readfirstlane(threadIdx.x >> 6);
#define LANE_INIT() int tid = threadIdx.x; asm volatile("" : "+v"(tid)); const int lane = tid & 63; \
    const unsigned wv = (unsigned)((lane >> 4) * 16) * N + wid * 64 + 4 * (lane & 15)
    const int bid = blockIdx.x;
    const unsigned rank = (unsigned)a.rank & (NR - 1);
    char* const myws = a.myws;
    GAS unsigned* const ctr = (GAS unsigned*)(myws + WS_CTR + bid * 128);
#define CALL_LOAD() asm volatile("" ::: "memory"); const unsigned ctr_old = __hip_atomic_load(ctr, __ATOMIC_RELAXED, __HIP_MEMORY_SCOPE_AGENT)
#define CALL_INIT() const unsigned call = __builtin_amdgcn_readfirstlane(ctr_old) + 1u; const size_t doff = WS_DATA + (size_t)(call & 1u) * DATA_BYTES
#define CALL_ADVANCE() do { if (tid == 0) __hip_atomic_store(ctr, call, __ATOMIC_RELAXED, __HIP_MEMORY_SCOPE_AGENT); } while (0)
    const __amdgpu_buffer_rsrc_t rx = __builtin_amdgcn_make_buffer_rsrc((void*)a.x, 0, (unsigned)M * KS, 0x00020000);
    const __amdgpu_buffer_rsrc_t rw = __builtin_amdgcn_make_buffer_rsrc((void*)a.w, 0, (unsigned)KS * N, 0x00020000);
    if (bid < NCOMM) {
        LANE_INIT(); CALL_LOAD();
        const unsigned xv = (unsigned)(wid * 64 + (lane & 15)) * KS + (lane >> 4) * 16;
        const unsigned ov = wid * 8192 + lane * 16;
        unsigned rawA[16], rawB[16]; u32x4 xrA[2], xrB[2];
        unsigned u0 = (bid << 16) | a.order[bid >> 2], u1 = ((bid + NCOMM) << 16) | a.order[(bid + NCOMM) >> 2];
        unsigned u2 = ((bid + 2 * NCOMM) << 16) | a.order[(bid + 2 * NCOMM) >> 2], u3 = ((bid + 3 * NCOMM) << 16) | a.order[(bid + 3 * NCOMM) >> 2], pfl = 0;
#define COMM_LOAD(raw, xr, u) do { const unsigned _p = (u) & 255u, _k = ((u) >> 8) & 3u, _h = ((u) >> 16) & 1u, _f = ((u) >> 17) & 1u; \
            const bool _x = _p < 16; \
            const unsigned sox = _x ? (_p * 256u + _f * 32u) * KS + _k * 128u + _h * 64u : 0u, sow = _x ? 0u : (_k * 128u + _h * 64u) * N + (_p - 16u) * 256u; \
            const __amdgpu_buffer_rsrc_t rsx = __builtin_amdgcn_make_buffer_rsrc((void*)a.x, 0, _x ? (unsigned)M * KS : 0u, 0x00020000); \
            const __amdgpu_buffer_rsrc_t rsw = __builtin_amdgcn_make_buffer_rsrc((void*)a.w, 0, _x ? 0u : (unsigned)KS * N, 0x00020000); \
            _Pragma("unroll") for (int i = 0; i < 2; ++i) xr[i] = __builtin_amdgcn_raw_buffer_load_b128(rsx, xv + i * 16 * KS, sox, 0); \
            _Pragma("unroll") for (int i = 0; i < 16; ++i) raw[i] = __builtin_amdgcn_raw_buffer_load_b32(rsw, wv, sow + i * N, 0); } while (0)
        COMM_LOAD(rawA, xrA, u0); COMM_LOAD(rawB, xrB, u1);
        { const unsigned e0 = a.order[tid], e1 = a.order[256 + tid < NBLOCK ? 256 + tid : NBLOCK - 1];
          *(LAS unsigned*)(lds + tid * 4) = e0; *(LAS unsigned*)(lds + 1024 + tid * 4) = e1; }
        CALL_INIT();
        char* dst[NR - 1]; unsigned low4 = 0;
#pragma unroll
        for (int i = 0; i < NR - 1; ++i) { const unsigned pi = (unsigned)(i + 2 * wid + bid) % (NR - 1);
            dst[i] = a.peer[pi] + doff; low4 |= (((rank + 1u + pi) & (NR - 1)) < 4u ? 1u : 0u) << i; }
        GAS unsigned* fl = (GAS unsigned*)(a.peer[lane < NR - 1 ? lane : 0] + WS_FLAGS) + rank * 4;
        GAS unsigned* const tmo = (GAS unsigned*)(myws + WS_TMO);
        char* const tickp = myws + WS_TICKET;
        const unsigned tbase = (call - 1u) * NUNIT - 4u * NCOMM, one = 1u;
        bool bad = false;
#define SPIN_UNTIL(cond) do { const unsigned long long _t0 = __builtin_amdgcn_s_memrealtime(); \
            while (__builtin_amdgcn_ballot_w64(!(cond)) != 0ull) { \
                if (__builtin_amdgcn_readfirstlane(__hip_atomic_load(tmo, __ATOMIC_RELAXED, __HIP_MEMORY_SCOPE_AGENT)) == call \
                    || __builtin_amdgcn_s_memrealtime() - _t0 > DEADLINE_TICKS) { \
                    __hip_atomic_store(tmo, call, __ATOMIC_RELAXED, __HIP_MEMORY_SCOPE_AGENT); bad = true; break; } \
                __builtin_amdgcn_s_sleep(8); } } while (0)
#define COMM_ST(o, vo, so) do { _Pragma("unroll") for (int d = 0; d < NR - 1; ++d) __builtin_amdgcn_raw_buffer_store_b128(o, rd[d], vo, so, AUX_SYS); ST_PAD(); } while (0)
#define UNIT_OFF(u) (((((u) >> 8) & 3u) * NR + rank) * SLAB + ((u) & 255u) * BLK + (((u) >> 16) & 2u) * 2048u + (((u) >> 16) & 1u) * 1024u)
        const __amdgpu_buffer_rsrc_t rl = __builtin_amdgcn_make_buffer_rsrc(myws + doff, 0, DATA_BYTES, 0x00020000);
#define COMM_STEP(raw, xr, first, word, craw, lraw) do { \
            u32x4 o0 = xr[0], o1 = xr[1]; \
            if (craw) { const unsigned sel = (u0 & 0x20000u) ? 0x07030602u : 0x05010400u; \
              _Pragma("unroll") for (int b = 0; b < 4; ++b) { \
                    const unsigned t0 = __builtin_amdgcn_perm(raw[4 * b + 1], raw[4 * b], sel), t2 = __builtin_amdgcn_perm(raw[4 * b + 3], raw[4 * b + 2], sel); \
                    o0[b] |= __builtin_amdgcn_perm(t2, t0, 0x05040100u); o1[b] |= __builtin_amdgcn_perm(t2, t0, 0x07060302u); } } \
            asm volatile("" : "+v"(o0), "+v"(o1)); \
            const unsigned sd = UNIT_OFF(u0); \
              \
            const unsigned _to = (craw) ? 127u : (u0 & 255u) == 16u + 17u ? low4 : (u0 & 255u) == 16u + 18u ? ~low4 : 127u; \
            __amdgpu_buffer_rsrc_t rd[NR - 1]; \
            _Pragma("unroll") for (int d = 0; d < NR - 1; ++d) rd[d] = __builtin_amdgcn_make_buffer_rsrc(dst[d], 0, ((_to >> d) & 1u) ? DATA_BYTES : 0u, 0x00020000); \
            if (lraw) COMM_LOAD(raw, xr, u2); \
            else { xr[0] = __builtin_amdgcn_raw_buffer_load_b128(rl, ov, UNIT_OFF(u2), AUX_SYS); xr[1] = __builtin_amdgcn_raw_buffer_load_b128(rl, ov + 2048, UNIT_OFF(u2), AUX_SYS); } \
            SCHED; COMM_ST(o0, ov, sd); if (lraw) WAIT_V(25); else WAIT_V(9); \
            if (!(first) && tid == 0) { unsigned _t; asm volatile("v_subrev_u32 %0, %1, v255" : "=v"(_t) : "s"(tbase)); \
                *(volatile LAS unsigned*)(lds + LDS_MISC + (word)) = (_t << 16) | *(volatile LAS unsigned*)(lds + ((_t & 2047u) & ~3u)); } \
            WAIT_L(0); BAR; \
            unsigned un = 0; \
            if (!(first)) { un = __builtin_amdgcn_readfirstlane(*(volatile LAS unsigned*)(lds + LDS_MISC + (word))); \
                if (tid < NR - 1) __hip_atomic_store(fl + pfl, call, __ATOMIC_RELAXED, __HIP_MEMORY_SCOPE_SYSTEM); } \
            else CALL_ADVANCE(); \
              \
            if (tid == 0) asm volatile("global_atomic_add v255, %0, %1, off sc0" :: "v"(tickp), "v"(one) : "memory", "v255"); \
            COMM_ST(o1, ov + 2048, sd); \
            pfl = (u0 & 255u) * 128u + ((u0 >> 8) & 3u) * (NR * 4u) + ((u0 >> 16) & 3u); \
            u0 = u1; u1 = u2; if (first) { u2 = u3; } else { u2 = un; } } while (0)
#define COMM_END ((u0 >> 16) >= NUNIT)
        COMM_STEP(rawA, xrA, true, 8, true, true);
        char* dout[NR - 1]; unsigned shoff[NSHARE]; float* outp = a.out;
#pragma unroll
        for (int i = 0; i < NR - 1; ++i) { dout[i] = a.peer_out[(unsigned)(i + 2 * wid + bid) % (NR - 1)]; asm volatile("" : "+s"(dout[i])); }
#pragma unroll
        for (int i = 0; i < NSHARE; ++i) { shoff[i] = a.shared[i]; asm volatile("" : "+s"(shoff[i])); }
        asm volatile("" : "+s"(outp));
        const __amdgpu_buffer_rsrc_t ro = __builtin_amdgcn_make_buffer_rsrc((void*)outp, 0, (unsigned)M * N * 4u, 0x00020000);
        do {
            for (;;) {
                COMM_STEP(rawB, xrB, false, 12, true, true); if (COMM_END) break;
                COMM_STEP(rawA, xrA, false, 8, true, true); if ((u0 >> 16) >= RAW_UNITS + RAW_STEP * bid) break;
            }
            if (COMM_END) break;
            SPIN_UNTIL(__hip_atomic_load((GAS unsigned*)(myws + WS_RETILE), __ATOMIC_RELAXED, __HIP_MEMORY_SCOPE_AGENT) >= call * (2u * NPANEL));
            COMM_STEP(rawB, xrB, false, 12, true, false); if (COMM_END) break;
            COMM_STEP(rawA, xrA, false, 8, true, false); if (COMM_END) break;
            for (;;) {
                COMM_STEP(rawB, xrB, false, 12, false, false); if (COMM_END) break;
                COMM_STEP(rawA, xrA, false, 8, false, false); if (COMM_END) break;
            }
        } while (0);
#define TDONE() (__hip_atomic_load((GAS unsigned*)(myws + WS_TDONE) + (lane < NSHARE ? lane : 0), __ATOMIC_RELAXED, __HIP_MEMORY_SCOPE_AGENT) >= call)
#define PUBLISH() do { BAR; if (tid < NR - 1) __hip_atomic_store(fl + pfl, call, __ATOMIC_RELAXED, __HIP_MEMORY_SCOPE_SYSTEM); } while (0)
        const bool ready = __builtin_amdgcn_ballot_w64(!TDONE()) == 0ull;
        WAIT_V(0);
        if (!ready) { PUBLISH(); SPIN_UNTIL(TDONE()); }
        __builtin_amdgcn_fence(__ATOMIC_ACQUIRE, "agent");
        const unsigned fv = (unsigned)((bid * 4 + wid) * 4) * 65536u + lane * 16;
        u32x4 fw[NSHARE][4];
#pragma unroll
        for (int t = 0; t < NSHARE; ++t)
#pragma unroll
            for (int i = 0; i < 4; ++i) { fw[t][i] = __builtin_amdgcn_raw_buffer_load_b128(ro, fv, shoff[t] + i * 65536u, AUX_SYS); SCHED; }
        if (ready) PUBLISH();
        if (bad) fw[0][0] = (u32x4){0x7fc00000u, 0x7fc00000u, 0x7fc00000u, 0x7fc00000u};
        __amdgpu_buffer_rsrc_t rd[NR - 1];
#pragma unroll
        for (int i = 0; i < NR - 1; ++i) rd[i] = __builtin_amdgcn_make_buffer_rsrc(dout[i], 0, (unsigned)M * N * 4u, 0x00020000);
#pragma unroll
        for (int t = 0; t < NSHARE; ++t)
#pragma unroll
            for (int i = 0; i < 4; ++i) { COMM_ST(fw[t][i], fv, shoff[t] + i * 65536u); }
        WAIT_V(0); BAR;
        if (wid == 0) {
            if (lane < NR - 1) __hip_atomic_store(fl + ((WS_FWD - WS_FLAGS) / 4 + rank * (NCOMM - 4) + bid), call, __ATOMIC_RELAXED, __HIP_MEMORY_SCOPE_SYSTEM);
            SPIN_UNTIL(__hip_atomic_load((GAS unsigned*)(myws + WS_FWD) + ((rank + 1u + (lane < NR - 1 ? lane : 0)) & (NR - 1)) * NCOMM + bid, __ATOMIC_RELAXED, __HIP_MEMORY_SCOPE_SYSTEM) >= call);
            if (bad && lane == 0) a.out[(shoff[0] + fv) / 4] = __builtin_nanf("");
        }
        return;
    }

    LANE_INIT(); CALL_LOAD();
    const int gw = bid - NCOMM;
    const int wr = wid >> 1, wc = wid & 1, fr = lane & 15, fq = lane >> 4;
    constexpr int NLOC = 2 * NPANEL;
    const int lw = gw - (NWG - NCOMM - NLOC);
    unsigned raw[4][16];
    if (lw >= 0) {
        const unsigned h = lw & 1, p = lw >> 1;
        if (p < 16) {
            const unsigned xv = (unsigned)(wid * 64 + (lane & 15)) * KS + h * 64 + (lane >> 4) * 16;
#pragma unroll
            for (int i = 0; i < 4; ++i)
#pragma unroll
                for (int j = 0; j < 4; ++j) { const u32x4 v = __builtin_amdgcn_raw_buffer_load_b128(rx, xv + j * 16 * KS, p * (256u * KS) + i * 128u, 0);
                    raw[i][4 * j] = v[0]; raw[i][4 * j + 1] = v[1]; raw[i][4 * j + 2] = v[2]; raw[i][4 * j + 3] = v[3]; }
        } else {
#pragma unroll
            for (int i = 0; i < 4; ++i)
#pragma unroll
                for (int j = 0; j < 16; ++j) raw[i][j] = __builtin_amdgcn_raw_buffer_load_b32(rw, wv, (i * 128u + h * 64u + j) * N + (p - 16u) * 256u, 0);
        }
    }
    if (tid == 0) *(LAS unsigned*)(lds + LDS_MISC + 4) = 0u;
    CALL_INIT();
    const __amdgpu_buffer_rsrc_t rin = __builtin_amdgcn_make_buffer_rsrc(myws + doff, 0, DATA_BYTES, 0x00020000);
    const __amdgpu_buffer_rsrc_t rfl = __builtin_amdgcn_make_buffer_rsrc(myws + WS_FLAGS, 0, FLAG_BYTES, 0x00020000);
    GAS unsigned* tmo = (GAS unsigned*)(myws + WS_TMO);
    if (lw >= 0) {
        const unsigned h = lw & 1, p = lw >> 1;
#pragma unroll
        for (int i = 0; i < 4; ++i) {
            const unsigned so = (i * NR + rank) * SLAB + p * BLK + h * 1024;
            if (p < 16) {
#pragma unroll
                for (int j = 0; j < 4; ++j) __builtin_amdgcn_raw_buffer_store_b128((u32x4){raw[i][4 * j], raw[i][4 * j + 1], raw[i][4 * j + 2], raw[i][4 * j + 3]}, rin, lane * 16, so + (wid * 4 + j) * 2048, AUX_SYS);
                ST_PAD();
            } else {
                u32x4 o[4];
#pragma unroll
                for (int b = 0; b < 4; ++b) {
                    const unsigned r0 = raw[i][4 * b], r1 = raw[i][4 * b + 1], r2 = raw[i][4 * b + 2], r3 = raw[i][4 * b + 3];
                    const unsigned t0 = __builtin_amdgcn_perm(r1, r0, 0x05010400), t1 = __builtin_amdgcn_perm(r1, r0, 0x07030602);
                    const unsigned t2 = __builtin_amdgcn_perm(r3, r2, 0x05010400), t3 = __builtin_amdgcn_perm(r3, r2, 0x07030602);
                    o[0][b] = __builtin_amdgcn_perm(t2, t0, 0x05040100); o[1][b] = __builtin_amdgcn_perm(t2, t0, 0x07060302);
                    o[2][b] = __builtin_amdgcn_perm(t3, t1, 0x05040100); o[3][b] = __builtin_amdgcn_perm(t3, t1, 0x07060302);
                }
#pragma unroll
                for (int c = 0; c < 4; ++c) __builtin_amdgcn_raw_buffer_store_b128(o[c], rin, lane * 16, so + wid * 8192 + c * 2048, AUX_SYS);
                ST_PAD();
            }
        }
    }
    asm volatile("s_waitcnt vmcnt(0) lgkmcnt(0)" ::: "memory"); BAR;
    CALL_ADVANCE();
    if (lw >= 0 && tid < 8)
        __hip_atomic_store((GAS unsigned*)(myws + WS_FLAGS) + (lw >> 1) * 128u + ((tid >> 1) * NR + rank) * 4u + (tid & 1) * 2 + (lw & 1), call, __ATOMIC_RELAXED, __HIP_MEMORY_SCOPE_SYSTEM);
    if (lw >= 0 && tid == 0) __hip_atomic_fetch_add((GAS unsigned*)(myws + WS_RETILE), 1u, __ATOMIC_RELAXED, __HIP_MEMORY_SCOPE_AGENT);
    unsigned flip = 0;
    if (gw == 0) {
        unsigned xcc; asm volatile("s_getreg_b32 %0, hwreg(HW_REG_XCC_ID)" : "=s"(xcc));
        if (tid == 0) __hip_atomic_store((GAS unsigned*)(myws + WS_FLIP), (call << 1) | (xcc & 1u), __ATOMIC_RELAXED, __HIP_MEMORY_SCOPE_AGENT);
        flip = (xcc & 1u);
    } else {
        const unsigned long long t0 = __builtin_amdgcn_s_memrealtime();
        for (;;) {
            flip = __builtin_amdgcn_readfirstlane(__hip_atomic_load((GAS unsigned*)(myws + WS_FLIP), __ATOMIC_RELAXED, __HIP_MEMORY_SCOPE_AGENT));
            if ((flip >> 1) == (call & 0x7fffffffu)) break;
            if (__builtin_amdgcn_readfirstlane(__hip_atomic_load(tmo, __ATOMIC_RELAXED, __HIP_MEMORY_SCOPE_AGENT)) == call || __builtin_amdgcn_s_memrealtime() - t0 > DEADLINE_TICKS) {
                __hip_atomic_store(tmo, call, __ATOMIC_RELAXED, __HIP_MEMORY_SCOPE_AGENT); *(volatile LAS unsigned*)(lds + LDS_MISC + 4) = 1u; break; }
            __builtin_amdgcn_s_sleep(8);
        }
        WAIT_L(0); BAR;
        flip = __builtin_amdgcn_readfirstlane(*(volatile LAS unsigned*)(lds + LDS_MISC + 4)) != 0u ? 0u : (flip & 1u);
    }
    const unsigned long long tl = a.tiles[gw ^ __builtin_amdgcn_readfirstlane(flip)];
    const int ntiles = (int)(tl >> 50) & 7;
    const unsigned koff0 = (unsigned)(tl >> 56) & 31u;
    const float scale = __builtin_bit_cast(float, __builtin_amdgcn_readfirstlane(__builtin_bit_cast(int, a.sx[0] * a.sw[0])));
    const unsigned voff = wid * 4096 + lane * 16, ldsw = wid * 4096;
    const unsigned ldsb = (unsigned)(unsigned long)lds;
    unsigned aoff = ldsb + wr * 8192 + lane * 16, boff = ldsb + 4 * HB + wc * 8192 + lane * 16;
    asm volatile("" : "+v"(aoff), "+v"(boff));
    const unsigned ovoff = (unsigned)(wr * 64 + 4 * fq) * 65536u + wc * 256 + fr * 16;
#define TILE(i) ((unsigned)(tl >> (10 * (i))) & 0x3ffu)
#define SA(b, h) (((b) * 2 + (h)) * HB)
#define SB(b, h) ((4 + (b) * 2 + (h)) * HB)
#define G1(bufoff, soff, i) __builtin_amdgcn_raw_ptr_buffer_load_lds(rin, (LAS void*)(lds + (bufoff) + ldsw), 16, voff, (soff), (i) * 1024, AUX_SYS)
#define STAGE4(bufoff, soff) do { G1(bufoff, soff, 0); G1(bufoff, soff, 1); G1(bufoff, soff, 2); G1(bufoff, soff, 3); } while (0)
#define RD(U, i, base, off) asm volatile("ds_read_b128 a[%c1:%c2], %0 offset:%c3" :: "v"(base), "i"((U) + 4 * (i)), "i"((U) + 4 * (i) + 3), "i"((off) + (i) * 1024) : "memory")
#define MF(Z, ai, bj, m, n, RA, RB) mfma<acc_idx(ai, bj, m, n), Z, (RA) + 8 * (m), (RB) + 8 * (n)>()
#define EPG(eq, g) do { if constexpr ((eq) >= 0) epi_gap<(eq) < 0 ? 0 : (eq), (g)>(esc, erout, ovoff, eobase); } while (0)
#define QUAD(Z, ai, bj, RA, RB, RN, rdb, rdoff, stoff, soff, vm, eq, g0, hook) do { u32x4 _fv; unsigned _pi; \
        if constexpr ((vm) >= 0) { asm volatile("s_waitcnt vmcnt(%0) lgkmcnt(0)" :: "n"((vm) < 0 ? 0 : (vm)) : "memory"); BAR; } else WAIT_L(0); SCHED; \
        MF(Z, ai, bj, 0, 0, RA, RB); RD(RN, 0, rdb, rdoff); EPG(eq, g0 + 0); SCHED; \
        MF(Z, ai, bj, 0, 1, RA, RB); RD(RN, 1, rdb, rdoff); EPG(eq, g0 + 1); SCHED; \
        MF(Z, ai, bj, 0, 2, RA, RB); RD(RN, 2, rdb, rdoff); EPG(eq, g0 + 2); if constexpr ((hook) >= 3) PICK_A(_pi); SCHED; \
        MF(Z, ai, bj, 0, 3, RA, RB); RD(RN, 3, rdb, rdoff); EPG(eq, g0 + 3); SCHED; \
        MF(Z, ai, bj, 1, 0, RA, RB); RD(RN, 4, rdb, rdoff); EPG(eq, g0 + 4); if constexpr ((hook) == 3) PICK_B(_pi, kaA, kaB); if constexpr ((hook) == 4) PICK_B(_pi, kbA, kbB); SCHED; \
        MF(Z, ai, bj, 1, 1, RA, RB); RD(RN, 5, rdb, rdoff); EPG(eq, g0 + 5); SCHED; \
        MF(Z, ai, bj, 1, 2, RA, RB); RD(RN, 6, rdb, rdoff); EPG(eq, g0 + 6); SCHED; \
        MF(Z, ai, bj, 1, 3, RA, RB); RD(RN, 7, rdb, rdoff); EPG(eq, g0 + 7); SCHED; \
        MF(Z, ai, bj, 2, 0, RA, RB); G1(stoff, soff, 0); EPG(eq, g0 + 8); SCHED; \
        MF(Z, ai, bj, 2, 1, RA, RB); EPG(eq, g0 + 9); if constexpr ((hook) == 2) _fv = FLAG_LOAD(0); SCHED; \
        MF(Z, ai, bj, 2, 2, RA, RB); G1(stoff, soff, 1); EPG(eq, g0 + 10); SCHED; \
        MF(Z, ai, bj, 2, 3, RA, RB); EPG(eq, g0 + 11); SCHED; \
        MF(Z, ai, bj, 3, 0, RA, RB); G1(stoff, soff, 2); EPG(eq, g0 + 12); SCHED; \
        MF(Z, ai, bj, 3, 1, RA, RB); EPG(eq, g0 + 13); if constexpr ((hook) == 2) FLAG_MASK(_fv); SCHED; \
        MF(Z, ai, bj, 3, 2, RA, RB); G1(stoff, soff, 3); EPG(eq, g0 + 14); SCHED; \
        MF(Z, ai, bj, 3, 3, RA, RB); EPG(eq, g0 + 15); if constexpr ((hook) == 1) FLAG_FETCH(0); SCHED; } while (0)

    unsigned rmask = 0, done = 0, kaA, kaB, kbA, kbB;
#define BASE_A(tile) (((tile) & 15u) * BLK)
#define BASE_B(tile) ((16u + ((tile) >> 4)) * BLK)
#define FLAG_SOFF(tile) ((wid < 2 ? ((tile) & 15u) : 16u + ((tile) >> 4)) * 512u + (wid & 1) * 256u)
    unsigned pA = BASE_A(TILE(0)), pB = BASE_B(TILE(0)), psoff = FLAG_SOFF(TILE(0)), fsoff = psoff;
    const unsigned fvoff = lane * 4;
    int froff = LDS_FLAGS + ((lane & 32) + ((lane + koff0) & 31)) * 16; asm volatile("" : "+v"(froff));
#define FLAG_FETCH_AT(slot, so) __builtin_amdgcn_raw_ptr_buffer_load_lds(rfl, (LAS void*)(lds + LDS_FLAGS + (slot) * 1024 + wid * 256), 4, fvoff, so, 0, AUX_SYS)
#define FLAG_FETCH(slot) FLAG_FETCH_AT(slot, fsoff)
#define FLAG_LOAD(slot) (*(volatile LAS u32x4*)(lds + froff + (slot) * 1024))
#define FLAG_MASK(_v) do { const unsigned _a = _v[0] < _v[1] ? _v[0] : _v[1], _b = _v[2] < _v[3] ? _v[2] : _v[3]; \
            const unsigned long long _m = __builtin_amdgcn_ballot_w64((_a < _b ? _a : _b) >= call); \
            rmask |= (unsigned)_m & (unsigned)(_m >> 32); } while (0)
#define PICK_A(_i) do { \
            if (__builtin_expect((rmask & ~done) == 0u, 0)) {        \
                const unsigned long long _t0 = __builtin_amdgcn_s_memrealtime(); \
                for (;;) { \
                    FLAG_FETCH_AT(1, psoff); \
                    if (wid == 0 && __builtin_amdgcn_s_memrealtime() - _t0 > 10000ull        \
                        && (__builtin_amdgcn_readfirstlane(__hip_atomic_load(tmo, __ATOMIC_RELAXED, __HIP_MEMORY_SCOPE_AGENT)) == call \
                            || __builtin_amdgcn_s_memrealtime() - _t0 > DEADLINE_TICKS)) { \
                        __hip_atomic_store(tmo, call, __ATOMIC_RELAXED, __HIP_MEMORY_SCOPE_AGENT); \
                        *(volatile LAS unsigned*)(lds + LDS_MISC + 4) = 1u; } \
                    asm volatile("s_waitcnt vmcnt(0) lgkmcnt(0)" ::: "memory"); BAR; \
                    const u32x4 _pv = FLAG_LOAD(1); FLAG_MASK(_pv); \
                    if (__builtin_amdgcn_readfirstlane(*(volatile LAS unsigned*)(lds + LDS_MISC + 4)) != 0u) rmask = ~0u; \
                    WAIT_L(0); BAR; \
                    if ((rmask & ~done) != 0u) break; \
                    __builtin_amdgcn_s_sleep(8); \
                } \
            } \
            _i = __builtin_ctz(rmask & ~done); done |= 1u << _i; } while (0)
#define PICK_B(_i, oA, oB) do { const unsigned _q = ((_i + koff0) & 31u) * SLAB; oA = _q + pA; oB = _q + pB; asm volatile("" : "+s"(oA), "+s"(oB)); } while (0)
#define PICK(oA, oB) do { unsigned _pi; PICK_A(_pi); PICK_B(_pi, oA, oB); } while (0)
#define ITER(KIND, sw, S1, S2, S3, S4) do { \
        if (sw) fsoff = nsoff; \
        QUAD(KIND == 0, 0, 0, S1, V_B0A, V_B0B, boff, SA(1, 0), SA(1, 0), kbA, (KIND == 0 ? 38 : KIND == 3 ? 27 : 16), (KIND == 0 ? 3 : -1), 0, 4); \
        QUAD(false, 0, 0, S2, V_B0B, S3, boff, SA(0, 1), SB(0, 0), kaB, -1, (KIND == 0 ? 3 : -1), 16, 1); \
        if constexpr (KIND == 2) { \
            esc = *(volatile LAS unsigned*)(lds + LDS_MISC + 4) != 0u ? __builtin_nanf("") : scale; \
            esc = __builtin_bit_cast(float, __builtin_amdgcn_readfirstlane(__builtin_bit_cast(int, esc))); } \
        QUAD(KIND == 0, 0, 1, S1, S3, S4, boff, SA(1, 1), SB(1, 0), kbB, (KIND == 0 ? 38 : 17), (KIND == 0 ? 3 : KIND == 2 ? 0 : -1), (KIND == 0 ? 32 : 0), 0); \
        QUAD(false, 0, 1, S2, S4, S1, aoff, SA(0, 1), SB(0, 1), kaB + HB, -1, (KIND == 0 ? 2 : KIND == 2 ? 0 : -1), (KIND == 0 ? 0 : 16), 0); \
        QUAD(KIND == 0, 1, 1, S1, S3, S2, aoff, SA(1, 1), SB(1, 1), kbB + HB, (KIND == 0 ? 38 : KIND == 2 ? 27 : 17), (KIND == 0 ? 2 : KIND == 2 ? 0 : -1), (KIND == 0 ? 16 : 32), 0); \
        QUAD(false, 1, 1, S2, S4, S3, aoff, SA(0, 0), SA(0, 1), kaA + HB, -1, (KIND == 0 ? 2 : KIND == 2 ? 1 : -1), (KIND == 0 ? 32 : 0), 0); \
        if (sw) { done = 0; rmask = nmask; pA = nA; pB = nB; psoff = nsoff; } \
        QUAD(KIND == 0, 1, 0, S1, V_B0A, S4, aoff, SA(1, 0), SA(1, 1), kbA + HB, (KIND == 0 ? 38 : KIND == 2 ? 37 : 16), (KIND == 2 ? 1 : -1), 16, 2); \
        QUAD(false, 1, 0, S2, V_B0B, V_B0A, boff, SA(0, 0), SA(0, 0), kaA, -1, (KIND == 2 ? 1 : -1), 32, 3); } while (0)

    PICK(kaA, kaB);
    STAGE4(SA(0, 0), kaA); STAGE4(SB(0, 0), kaB); STAGE4(SB(0, 1), kaB + HB); STAGE4(SA(0, 1), kaA + HB);
    PICK(kbA, kbB);
    STAGE4(SA(1, 0), kbA); STAGE4(SB(1, 0), kbB); STAGE4(SB(1, 1), kbB + HB); STAGE4(SA(1, 1), kbA + HB);
    PICK(kaA, kaB);
    WAIT_V(12); BAR;
    asm volatile("; GEMM_REGS_BEGIN" ::: "memory", RCLOB);
#define RD8(U, base, off) RD(U, 0, base, off); RD(U, 1, base, off); RD(U, 2, base, off); RD(U, 3, base, off); RD(U, 4, base, off); RD(U, 5, base, off); RD(U, 6, base, off); RD(U, 7, base, off)
    RD8(V_R1, aoff, SA(0, 0)); RD8(V_B0A, boff, SA(0, 0)); WAIT_L(0); RD8(V_R2, aoff, SA(1, 0));
    WAIT_L(0); BAR;
    STAGE4(SA(0, 0), kaA);
    WAIT_V(0);
    __amdgpu_buffer_rsrc_t erout = __builtin_amdgcn_make_buffer_rsrc((void*)a.out, 0, 0, 0x00020000);
    unsigned eobase = 0; float esc = 0.f;
    int ui = 0;
    do {
        const bool more = ui + 1 < ntiles; const unsigned nt = TILE(more ? ui + 1 : ui);
        const unsigned nA = BASE_A(nt), nB = BASE_B(nt), nsoff = FLAG_SOFF(nt), nmask = more ? 0u : ~0u;
        ITER(0, false, V_R1, V_R2, V_R3, V_R4);
        ITER(3, false, V_R3, V_R4, V_R1, V_R2);
        if (__builtin_expect(ui == (int)((tl >> 53) & 7u) && ui > 0, 0)) {
            WAIT_V(0); BAR;
            if (tid == 0) {
                __builtin_amdgcn_fence(__ATOMIC_RELEASE, "agent"); WAIT_V(0);
                __hip_atomic_store((GAS unsigned*)(myws + WS_TDONE) + (unsigned)(tl >> 61), call, __ATOMIC_RELAXED, __HIP_MEMORY_SCOPE_AGENT);
            }
        }
        for (int it = 2; it < 14; it += 2) { ITER(1, false, V_R1, V_R2, V_R3, V_R4); ITER(1, false, V_R3, V_R4, V_R1, V_R2); }
        ITER(1, true, V_R1, V_R2, V_R3, V_R4);
        erout = __builtin_amdgcn_make_buffer_rsrc((void*)a.out, 0, (unsigned)M * N * 4u, 0x00020000);
        eobase = (TILE(ui) & 15u) * (256u * 65536u) + (TILE(ui) >> 4) * 1024u;
        ITER(2, false, V_R3, V_R4, V_R1, V_R2);
    } while (++ui < ntiles);
#define EP3(q, g) EPG(q, g); EPG(q, g + 1); EPG(q, g + 2)
#define EP12(q, g) EP3(q, g); EP3(q, g + 3); EP3(q, g + 6); EP3(q, g + 9)
    EP12(3, 0); EP12(3, 12); EP12(3, 24); EP12(3, 36); EP12(2, 0); EP12(2, 12); EP12(2, 24); EP12(2, 36);
    WAIT_V(0);
    asm volatile("; GEMM_REGS_END" ::: "memory");
}

void make_tables(Args& a, int rank) {
    constexpr double P_G1X = -2.0, P_SX = -3.0, P_RX = 1.18, P_SPS = 10.0, P_LEN2 = 38.0, P_S3 = 44.0, P_TAU = 38.0, P_ZOFF = -8.0, P_KS0 = 2.0;
    constexpr int P_KZ = 25, P_RH = 1;
    double key[320]; int n = 0;
    constexpr int PZ = 15 - NZ;
    auto koff = [](int p) { return p < 4 ? 0 : p < PZ ? (int)(P_KS0 + (p - 4) * P_SPS / (PZ - 5) / 1.25 + 0.5) : P_KZ; };
    auto phase = [](int p) { return p < PZ ? p * P_SPS / (PZ - 5) : P_ZOFF; };
    auto add = [&](int panel, double start, double rate, int rot, int off) {
        for (int kt = 0; kt < 4; ++kt) {
            const double k = start + ((kt * NR + ((rank + rot) & (NR - 1)) - off) & 31) * rate + panel * 1e-6;
            int i = n++;
            for (; i > 0 && key[i - 1] > k; --i) { key[i] = key[i - 1]; a.order[i] = a.order[i - 1]; }
            key[i] = k; a.order[i] = (unsigned)((kt << 8) | panel);
        }
    };
    for (int i = 0; i < 8; ++i) { add(i, -1000.0 + P_G1X, 1.0, i, 0); add(16 + i, -1000.0, 1.0, i, 0); add(8 + i, P_SX, P_RX, i, 0); }
    for (int p = 4; p < PZ; ++p) add(16 + 8 + (p - 4), phase(p) - phase(4), P_LEN2 / 32.0, p, koff(p));
    for (int g = 0; g < 3; ++g)
        for (int p = 0; p < 15; ++p) add(16 + 19 + g * 15 + p, P_S3 + g * P_TAU + phase(p), 1.18, p, koff(p));
    for (int i = 0; i < NZ; ++i) add(16 + 19 - NZ + i, P_S3 + P_RH * P_TAU + phase(PZ), 1.18, 1, koff(PZ));
    const int H = 19 + P_RH * 15 + PZ;
    int smp[NSHARE], snp[NSHARE];
    for (int i = 0; i < 2 * NZ; ++i) { smp[i] = (2 * NZ * rank + i) & 15; snp[i] = 19 - NZ + ((2 * NZ * rank) >> 4); }
    smp[2 * NZ] = rank; snp[2 * NZ] = H;
    for (int i = 0; i < NSHARE; ++i) a.shared[i] = (unsigned)smp[i] * (256u * 65536u) + (unsigned)snp[i] * 1024u;
    for (int g = 0; g < NWG - NCOMM; ++g) {
        constexpr int seq[15] = {0, 2, 12, 11, 10, 9, 7, 5, 8, 6, 4, 14, 13, 3, 1};
        const int r = (g & 1) ? 59 - (g >> 3) : (g >> 3), p = seq[r >> 2], m = ((g & 7) >> 1) * 4 + (r & 3);
        int mp[5], np[5], nt = 0; unsigned long long tl = 0;
        if (p < 4) { mp[0] = m & 7; mp[1] = 8 + (m & 7); np[0] = np[1] = 2 * p + (m >> 3); nt = 2; }
        else if (p < PZ) { mp[0] = m; np[0] = 8 + p - 4; nt = 1; }
        for (int k = 0; k < 3; ++k) {
            if (19 + k * 15 + p == H && m < 8) {
                if (m < NSHARE) { mp[nt] = smp[m]; np[nt] = snp[m]; ++nt; tl |= ((unsigned long long)nt << 53) | ((unsigned long long)m << 61); }
            } else { mp[nt] = m; np[nt] = 19 + k * 15 + p; ++nt; }
        }
        tl |= ((unsigned long long)nt << 50) | ((unsigned long long)koff(p) << 56);
        for (int i = 0; i < nt; ++i) tl |= (unsigned long long)((np[i] << 4) | mp[i]) << (10 * i);
        a.tiles[g] = tl;
    }
}
}

extern "C" void kernel_launch(int rank, int nranks, ncclComm_t comm,
                              hipStream_t stream,
                              void* const* d_in, const int* in_sizes, int n_in,
                              void* d_out, int out_size,
                              const int* in_dtypes, int out_dtype,
                              void* const* d_out_all,
                              void* const* d_ws_all, size_t ws_bytes) {
    (void)comm; (void)in_dtypes; (void)out_dtype;
    if (nranks != NR || (unsigned)rank >= (unsigned)NR || n_in != 4 || in_sizes[0] != M * KS || in_sizes[1] != KS * N ||
        out_size != M * N || ws_bytes < WS_NEED) {
        fprintf(stderr, "kernel_launch: unexpected shapes\n");
        return;
    }
    Args a;
    a.x = (const unsigned char*)d_in[0]; a.w = (const unsigned char*)d_in[1];
    a.sx = (const float*)d_in[2]; a.sw = (const float*)d_in[3];
    a.out = (float*)d_out;
    a.myws = (char*)d_ws_all[rank];
    for (int i = 0; i < NR - 1; ++i) { a.peer[i] = (char*)d_ws_all[(rank + 1 + i) % NR]; a.peer_out[i] = (char*)d_out_all[(rank + 1 + i) % NR]; }
    a.rank = rank;
    make_tables(a, rank);
    hipLaunchKernelGGL(fused_gemm, dim3(NWG), dim3(256), 0, stream, a);
    HIP_CHECK(hipGetLastError());
}
```

```cpp
#include <hip/hip_runtime.h>
#include <rccl/rccl.h>
#include <cstdio>

#define HIP_CHECK(expr) do { hipError_t _e = (expr); if (_e != hipSuccess) { \
    fprintf(stderr, "HIP error %s at %s:%d\n", hipGetErrorString(_e), __FILE__, __LINE__); \
    return; } } while (0)

#define LAS __attribute__((address_space(3)))
#define GAS __attribute__((address_space(1)))
typedef unsigned u32x4 __attribute__((ext_vector_type(4)));

namespace {
constexpr int NR = 8;
constexpr int M = 4096, N = 16384, KS = 512;
constexpr unsigned BLK = 32768, HB = 16384;
constexpr unsigned NPANEL = 80;
constexpr unsigned SLAB = NPANEL * BLK;
constexpr unsigned DATA_BYTES = 32 * SLAB;
constexpr unsigned WS_FLAGS = 0;
constexpr unsigned FLAG_BYTES = NPANEL * 512;
constexpr unsigned WS_CTR = 65536;
constexpr unsigned WS_TMO = WS_CTR + 256 * 128;
constexpr unsigned WS_TICKET = WS_TMO + 128;
constexpr unsigned WS_TDONE = WS_TICKET + 128;
constexpr int NZ = 2;
static_assert(NZ == 2, "the senders' destination masks name column panels 17 and 18");
constexpr int NSHARE = 2 * NZ + 1;
constexpr unsigned WS_RETILE = WS_TDONE + 128;
constexpr unsigned WS_FLIP = WS_RETILE + 128;
constexpr unsigned WS_FWD = WS_FLIP + 128;
constexpr unsigned NBLOCK = 320, NUNIT = 4 * NBLOCK;
constexpr size_t WS_DATA = 1u << 20;
constexpr size_t WS_NEED = WS_DATA + 2 * (size_t)DATA_BYTES;
constexpr int NCOMM = 16, NWG = 256;
constexpr unsigned long long DEADLINE_TICKS = 50000000ull;
constexpr int AUX_SYS = 17;
constexpr unsigned RAW_UNITS = 64u, RAW_STEP = 8u;
constexpr unsigned LDS_FLAGS = 8 * HB;
constexpr unsigned LDS_MISC = LDS_FLAGS + 2 * 1024;
constexpr unsigned LDS_BYTES = LDS_MISC + 64;

struct Args {
    const unsigned char* x; const unsigned char* w; const float* sx; const float* sw; float* out;
    char* myws; char* peer[NR - 1];
    char* peer_out[NR - 1];
    int rank;
    unsigned shared[NSHARE];
    unsigned order[NBLOCK];
    unsigned long long tiles[NWG - NCOMM];
};

#define WAIT_V(n) asm volatile("s_waitcnt vmcnt(" #n ")" ::: "memory")
#define WAIT_L(n) asm volatile("s_waitcnt lgkmcnt(" #n ")" ::: "memory")
#define BAR __builtin_amdgcn_s_barrier()
#define SCHED __builtin_amdgcn_sched_barrier(0)
#define ST_PAD() do { SCHED; asm volatile("s_nop 1"); SCHED; } while (0)

#define RCLOB "a0","a1","a2","a3","a4","a5","a6","a7","a8","a9","a10","a11","a12","a13","a14","a15","a16","a17","a18","a19","a20","a21","a22","a23","a24","a25","a26","a27","a28","a29","a30","a31","a32","a33","a34","a35","a36","a37","a38","a39","a40","a41","a42","a43","a44","a45","a46","a47","a48","a49","a50","a51","a52","a53","a54","a55","a56","a57","a58","a59","a60","a61","a62","a63","a64","a65","a66","a67","a68","a69","a70","a71","a72","a73","a74","a75","a76","a77","a78","a79","a80","a81","a82","a83","a84","a85","a86","a87","a88","a89","a90","a91","a92","a93","a94","a95","a96","a97","a98","a99","a100","a101","a102","a103","a104","a105","a106","a107","a108","a109","a110","a111","a112","a113","a114","a115","a116","a117","a118","a119","a120","a121","a122","a123","a124","a125","a126","a127","a128","a129","a130","a131","a132","a133","a134","a135","a136","a137","a138","a139","a140","a141","a142","a143","a144","a145","a146","a147","a148","a149","a150","a151","a152","a153","a154","a155","a156","a157","a158","a159","a160","a161","a162","a163","a164","a165","a166","a167","a168","a169","a170","a171","a172","a173","a174","a175","a176","a177","a178","a179","a180","a181","a182","a183","a184","a185","a186","a187","a188","a189","a190","a191","a192","a193","a194","a195","a196","a197","a198","a199","a200","a201","a202","a203","a204","a205","a206","a207","a208","a209","a210","a211","a212","a213","a214","a215","a216","a217","a218","a219","a220","a221","a222","a223","a224","a225","a226","a227","a228","a229","a230","a231","a232","a233","a234","a235","a236","a237","a238","a239","v40","v41","v42","v43","v44","v45","v46","v47","v48","v49","v50","v51","v52","v53","v54","v55","v56","v57","v58","v59","v60","v61","v62","v63","v64","v65","v66","v67","v68","v69","v70","v71","v72","v73","v74","v75","v76","v77","v78","v79","v80","v81","v82","v83","v84","v85","v86","v87","v88","v89","v90","v91","v92","v93","v94","v95","v96","v97","v98","v99","v100","v101","v102","v103","v104","v105","v106","v107","v108","v109","v110","v111","v112","v113","v114","v115","v116","v117","v118","v119","v120","v121","v122","v123","v124","v125","v126","v127","v128","v129","v130","v131","v132","v133","v134","v135","v136","v137","v138","v139","v140","v141","v142","v143","v144","v145","v146","v147","v148","v149","v150","v151","v152","v153","v154","v155","v156","v157","v158","v159","v160","v161","v162","v163","v164","v165","v166","v167","v168","v169","v170","v171","v172","v173","v174","v175","v176","v177","v178","v179","v180","v181","v182","v183","v184","v185","v186","v187","v188","v189","v190","v191","v192","v193","v194","v195","v196","v197","v198","v199","v200","v201","v202","v203","v204","v205","v206","v207","v208","v209","v210","v211","v212","v213","v214","v215","v216","v217","v218","v219","v220","v221","v222","v223","v224","v225","v226","v227","v228","v229","v230","v231","v232","v233","v234","v235","v236","v237","v238","v239","v240","v241","v242","v243","v244","v245","v246","v247","v248","v249","v250","v251","v252","v253","v254","v255"
constexpr int V_EV = 40, V_R1 = 0, V_R2 = 32, V_R3 = 64, V_R4 = 96, V_B0A = 128, V_B0B = 160;
constexpr int acc_idx(int ai, int bj, int m, int n) { return ((ai * 2 + bj) * 4 + m) * 4 + n; }
constexpr bool acc_in_v(int f) { return f < 52; }
constexpr int acc_reg(int f) { return f < 52 ? 48 + 4 * f : 192 + 4 * (f - 52); }
#define MFMA_ASM(D, C, ...) asm volatile("v_mfma_f32_16x16x128_f8f6f4 " D "[%c0:%c1], a[%c2:%c3], a[%c4:%c5], " C " cbsz:1 blgp:1" \
                                         :: "i"(R), "i"(R + 3), "i"(X), "i"(X + 7), "i"(Y), "i"(Y + 7) __VA_ARGS__)
template <int F, bool ZERO, int X, int Y> __device__ __forceinline__ void mfma() {
    constexpr int R = acc_reg(F);
    if constexpr ((F & 15) == 0) { if constexpr (ZERO) MFMA_ASM("v", "0", : RCLOB); else MFMA_ASM("v", "v[%c0:%c1]", : RCLOB); }
    else if constexpr (acc_in_v(F)) { if constexpr (ZERO) MFMA_ASM("v", "0"); else MFMA_ASM("v", "v[%c0:%c1]"); }
    else { if constexpr (ZERO) MFMA_ASM("a", "0"); else MFMA_ASM("a", "a[%c0:%c1]"); }
}
template <int Q, int G> __device__ __forceinline__ void epi_gap(float sc, __amdgpu_buffer_rsrc_t ro, unsigned ovoff, unsigned obase) {
    constexpr int E = G / 3, part = G % 3, m = E >> 2, j = E & 3, ai = Q >> 1, bj = Q & 1, V = V_EV + (E & 1) * 4, R = acc_reg(acc_idx(ai, bj, m, 0)) + j;
    constexpr bool inv = acc_in_v(acc_idx(ai, bj, m, 0));
    if constexpr (part == 0 && inv)
        asm volatile("v_mul_f32 v%c1, %0, v%c5\n\tv_mul_f32 v%c2, %0, v%c6\n\tv_mul_f32 v%c3, %0, v%c7\n\tv_mul_f32 v%c4, %0, v%c8"
                     :: "s"(sc), "i"(V), "i"(V + 1), "i"(V + 2), "i"(V + 3), "i"(R), "i"(R + 4), "i"(R + 8), "i"(R + 12));
    else if constexpr (part == 0)
        asm volatile("v_accvgpr_read_b32 v%c1, a%c5\n\tv_accvgpr_read_b32 v%c2, a%c6\n\tv_accvgpr_read_b32 v%c3, a%c7\n\tv_accvgpr_read_b32 v%c4, a%c8\n\t"
                     "v_mul_f32 v%c1, %0, v%c1\n\tv_mul_f32 v%c2, %0, v%c2"
                     :: "s"(sc), "i"(V), "i"(V + 1), "i"(V + 2), "i"(V + 3), "i"(R), "i"(R + 4), "i"(R + 8), "i"(R + 12));
    else if constexpr (part == 1) {
        if constexpr (!inv) asm volatile("v_mul_f32 v%c1, %0, v%c1\n\tv_mul_f32 v%c2, %0, v%c2" :: "s"(sc), "i"(V + 2), "i"(V + 3));
        asm volatile("v_maximum3_f32 v%c0, v%c0, 0, 0\n\tv_maximum3_f32 v%c1, v%c1, 0, 0\n\tv_maximum3_f32 v%c2, v%c2, 0, 0\n\tv_maximum3_f32 v%c3, v%c3, 0, 0"
                     :: "i"(V), "i"(V + 1), "i"(V + 2), "i"(V + 3));
    } else
        asm volatile("buffer_store_dwordx4 v[%c3:%c4], %0, %1, %2 offen offset:%c5 nt"
                     :: "v"(ovoff), "s"(ro), "s"(obase + (unsigned)(ai * 128 + m * 16 + j) * 65536u), "i"(V), "i"(V + 3), "i"(bj * 512) : "memory");
}

__global__ __launch_bounds__(256, 1) void fused_gemm(Args a) {
    __shared__ __attribute__((aligned(16))) char shm[LDS_BYTES];
    LAS char* lds = (LAS char*)shm;
    const int wid = __builtin_amdgcn_readfirstlane(threadIdx.x >> 6);
#define LANE_INIT() int tid = threadIdx.x; asm volatile("" : "+v"(tid)); const int lane = tid & 63; \
    const unsigned wv = (unsigned)((lane >> 4) * 16) * N + wid * 64 + 4 * (lane & 15)
    const int bid = blockIdx.x;
    const unsigned rank = (unsigned)a.rank & (NR - 1);
    char* const myws = a.myws;
    GAS unsigned* const ctr = (GAS unsigned*)(myws + WS_CTR + bid * 128);
#define CALL_LOAD() asm volatile("" ::: "memory"); const unsigned ctr_old = __hip_atomic_load(ctr, __ATOMIC_RELAXED, __HIP_MEMORY_SCOPE_AGENT)
#define CALL_INIT() const unsigned call = __builtin_amdgcn_readfirstlane(ctr_old) + 1u; const size_t doff = WS_DATA + (size_t)(call & 1u) * DATA_BYTES
#define CALL_ADVANCE() do { if (tid == 0) __hip_atomic_store(ctr, call, __ATOMIC_RELAXED, __HIP_MEMORY_SCOPE_AGENT); } while (0)
    const __amdgpu_buffer_rsrc_t rx = __builtin_amdgcn_make_buffer_rsrc((void*)a.x, 0, (unsigned)M * KS, 0x00020000);
    const __amdgpu_buffer_rsrc_t rw = __builtin_amdgcn_make_buffer_rsrc((void*)a.w, 0, (unsigned)KS * N, 0x00020000);
    if (bid < NCOMM) {
        LANE_INIT(); CALL_LOAD();
        const unsigned xv = (unsigned)(wid * 64 + (lane & 15)) * KS + (lane >> 4) * 16;
        const unsigned ov = wid * 8192 + lane * 16;
        unsigned rawA[16], rawB[16]; u32x4 xrA[2], xrB[2];
        unsigned u0 = (bid << 16) | a.order[bid >> 2], u1 = ((bid + NCOMM) << 16) | a.order[(bid + NCOMM) >> 2];
        unsigned u2 = ((bid + 2 * NCOMM) << 16) | a.order[(bid + 2 * NCOMM) >> 2], u3 = ((bid + 3 * NCOMM) << 16) | a.order[(bid + 3 * NCOMM) >> 2], pfl = 0;
#define COMM_LOAD(raw, xr, u) do { const unsigned _p = (u) & 255u, _k = ((u) >> 8) & 3u, _h = ((u) >> 16) & 1u, _f = ((u) >> 17) & 1u; \
            const bool _x = _p < 16; \
            const unsigned sox = _x ? (_p * 256u + _f * 32u) * KS + _k * 128u + _h * 64u : 0u, sow = _x ? 0u : (_k * 128u + _h * 64u) * N + (_p - 16u) * 256u; \
            const __amdgpu_buffer_rsrc_t rsx = __builtin_amdgcn_make_buffer_rsrc((void*)a.x, 0, _x ? (unsigned)M * KS : 0u, 0x00020000); \
            const __amdgpu_buffer_rsrc_t rsw = __builtin_amdgcn_make_buffer_rsrc((void*)a.w, 0, _x ? 0u : (unsigned)KS * N, 0x00020000); \
            _Pragma("unroll") for (int i = 0; i < 2; ++i) xr[i] = __builtin_amdgcn_raw_buffer_load_b128(rsx, xv + i * 16 * KS, sox, 0); \
            _Pragma("unroll") for (int i = 0; i < 16; ++i) raw[i] = __builtin_amdgcn_raw_buffer_load_b32(rsw, wv, sow + i * N, 0); } while (0)
        COMM_LOAD(rawA, xrA, u0); COMM_LOAD(rawB, xrB, u1);
        { const unsigned e0 = a.order[tid], e1 = a.order[256 + tid < NBLOCK ? 256 + tid : NBLOCK - 1];
          *(LAS unsigned*)(lds + tid * 4) = e0; *(LAS unsigned*)(lds + 1024 + tid * 4) = e1; }
        CALL_INIT();
        char* dst[NR - 1]; unsigned low4 = 0;
#pragma unroll
        for (int i = 0; i < NR - 1; ++i) { const unsigned pi = (unsigned)(i + 2 * wid + bid) % (NR - 1);
            dst[i] = a.peer[pi] + doff; low4 |= (((rank + 1u + pi) & (NR - 1)) < 4u ? 1u : 0u) << i; }
        GAS unsigned* fl = (GAS unsigned*)(a.peer[lane < NR - 1 ? lane : 0] + WS_FLAGS) + rank * 4;
        GAS unsigned* const tmo = (GAS unsigned*)(myws + WS_TMO);
        char* const tickp = myws + WS_TICKET;
        const unsigned tbase = (call - 1u) * NUNIT - 4u * NCOMM, one = 1u;
        bool bad = false;
#define SPIN_UNTIL(cond) do { const unsigned long long _t0 = __builtin_amdgcn_s_memrealtime(); \
            while (__builtin_amdgcn_ballot_w64(!(cond)) != 0ull) { \
                if (__builtin_amdgcn_readfirstlane(__hip_atomic_load(tmo, __ATOMIC_RELAXED, __HIP_MEMORY_SCOPE_AGENT)) == call \
                    || __builtin_amdgcn_s_memrealtime() - _t0 > DEADLINE_TICKS) { \
                    __hip_atomic_store(tmo, call, __ATOMIC_RELAXED, __HIP_MEMORY_SCOPE_AGENT); bad = true; break; } \
                __builtin_amdgcn_s_sleep(8); } } while (0)
#define COMM_ST(o, vo, so) do { _Pragma("unroll") for (int d = 0; d < NR - 1; ++d) __builtin_amdgcn_raw_buffer_store_b128(o, rd[d], vo, so, AUX_SYS); ST_PAD(); } while (0)
#define UNIT_OFF(u) (((((u) >> 8) & 3u) * NR + rank) * SLAB + ((u) & 255u) * BLK + (((u) >> 16) & 2u) * 2048u + (((u) >> 16) & 1u) * 1024u)
        const __amdgpu_buffer_rsrc_t rl = __builtin_amdgcn_make_buffer_rsrc(myws + doff, 0, DATA_BYTES, 0x00020000);
#define COMM_STEP(raw, xr, first, word, craw, lraw) do { \
            u32x4 o0 = xr[0], o1 = xr[1]; \
            if (craw) { const unsigned sel = (u0 & 0x20000u) ? 0x07030602u : 0x05010400u; \
              _Pragma("unroll") for (int b = 0; b < 4; ++b) { \
                    const unsigned t0 = __builtin_amdgcn_perm(raw[4 * b + 1], raw[4 * b], sel), t2 = __builtin_amdgcn_perm(raw[4 * b + 3], raw[4 * b + 2], sel); \
                    o0[b] |= __builtin_amdgcn_perm(t2, t0, 0x05040100u); o1[b] |= __builtin_amdgcn_perm(t2, t0, 0x07060302u); } } \
            asm volatile("" : "+v"(o0), "+v"(o1)); \
            const unsigned sd = UNIT_OFF(u0); \
              \
            const unsigned _to = (craw) ? 127u : (u0 & 255u) == 16u + 17u ? low4 : (u0 & 255u) == 16u + 18u ? ~low4 : 127u; \
            __amdgpu_buffer_rsrc_t rd[NR - 1]; \
            _Pragma("unroll") for (int d = 0; d < NR - 1; ++d) rd[d] = __builtin_amdgcn_make_buffer_rsrc(dst[d], 0, ((_to >> d) & 1u) ? DATA_BYTES : 0u, 0x00020000); \
            if (lraw) COMM_LOAD(raw, xr, u2); \
            else { xr[0] = __builtin_amdgcn_raw_buffer_load_b128(rl, ov, UNIT_OFF(u2), AUX_SYS); xr[1] = __builtin_amdgcn_raw_buffer_load_b128(rl, ov + 2048, UNIT_OFF(u2), AUX_SYS); } \
            SCHED; COMM_ST(o0, ov, sd); if (lraw) WAIT_V(25); else WAIT_V(9); \
            if (!(first) && tid == 0) { unsigned _t; asm volatile("v_subrev_u32 %0, %1, v255" : "=v"(_t) : "s"(tbase)); \
                *(volatile LAS unsigned*)(lds + LDS_MISC + (word)) = (_t << 16) | *(volatile LAS unsigned*)(lds + ((_t & 2047u) & ~3u)); } \
            WAIT_L(0); BAR; \
            unsigned un = 0; \
            if (!(first)) { un = __builtin_amdgcn_readfirstlane(*(volatile LAS unsigned*)(lds + LDS_MISC + (word))); \
                if (tid < NR - 1) __hip_atomic_store(fl + pfl, call, __ATOMIC_RELAXED, __HIP_MEMORY_SCOPE_SYSTEM); } \
            else CALL_ADVANCE(); \
              \
            if (tid == 0) asm volatile("global_atomic_add v255, %0, %1, off sc0" :: "v"(tickp), "v"(one) : "memory", "v255"); \
            COMM_ST(o1, ov + 2048, sd); \
            pfl = (u0 & 255u) * 128u + ((u0 >> 8) & 3u) * (NR * 4u) + ((u0 >> 16) & 3u); \
            u0 = u1; u1 = u2; if (first) { u2 = u3; } else { u2 = un; } } while (0)
#define COMM_END ((u0 >> 16) >= NUNIT)
        COMM_STEP(rawA, xrA, true, 8, true, true);
        char* dout[NR - 1]; unsigned shoff[NSHARE]; float* outp = a.out;
#pragma unroll
        for (int i = 0; i < NR - 1; ++i) { dout[i] = a.peer_out[(unsigned)(i + 2 * wid + bid) % (NR - 1)]; asm volatile("" : "+s"(dout[i])); }
#pragma unroll
        for (int i = 0; i < NSHARE; ++i) { shoff[i] = a.shared[i]; asm volatile("" : "+s"(shoff[i])); }
        asm volatile("" : "+s"(outp));
        const __amdgpu_buffer_rsrc_t ro = __builtin_amdgcn_make_buffer_rsrc((void*)outp, 0, (unsigned)M * N * 4u, 0x00020000);
        do {
            for (;;) {
                COMM_STEP(rawB, xrB, false, 12, true, true); if (COMM_END) break;
                COMM_STEP(rawA, xrA, false, 8, true, true); if ((u0 >> 16) >= RAW_UNITS + RAW_STEP * bid) break;
            }
            if (COMM_END) break;
            SPIN_UNTIL(__hip_atomic_load((GAS unsigned*)(myws + WS_RETILE), __ATOMIC_RELAXED, __HIP_MEMORY_SCOPE_AGENT) >= call * (2u * NPANEL));
            COMM_STEP(rawB, xrB, false, 12, true, false); if (COMM_END) break;
            COMM_STEP(rawA, xrA, false, 8, true, false); if (COMM_END) break;
            for (;;) {
                COMM_STEP(rawB, xrB, false, 12, false, false); if (COMM_END) break;
                COMM_STEP(rawA, xrA, false, 8, false, false); if (COMM_END) break;
            }
        } while (0);
#define TDONE() (__hip_atomic_load((GAS unsigned*)(myws + WS_TDONE) + (lane < NSHARE ? lane : 0), __ATOMIC_RELAXED, __HIP_MEMORY_SCOPE_AGENT) >= call)
#define PUBLISH() do { BAR; if (tid < NR - 1) __hip_atomic_store(fl + pfl, call, __ATOMIC_RELAXED, __HIP_MEMORY_SCOPE_SYSTEM); } while (0)
        const bool ready = __builtin_amdgcn_ballot_w64(!TDONE()) == 0ull;
        WAIT_V(0);
        if (!ready) { PUBLISH(); SPIN_UNTIL(TDONE()); }
        __builtin_amdgcn_fence(__ATOMIC_ACQUIRE, "agent");
        const unsigned fv = (unsigned)((bid * 4 + wid) * 4) * 65536u + lane * 16;
        u32x4 fw[NSHARE][4];
#pragma unroll
        for (int t = 0; t < NSHARE; ++t)
#pragma unroll
            for (int i = 0; i < 4; ++i) { fw[t][i] = __builtin_amdgcn_raw_buffer_load_b128(ro, fv, shoff[t] + i * 65536u, AUX_SYS); SCHED; }
        if (ready) PUBLISH();
        if (bad) fw[0][0] = (u32x4){0x7fc00000u, 0x7fc00000u, 0x7fc00000u, 0x7fc00000u};
        __amdgpu_buffer_rsrc_t rd[NR - 1];
#pragma unroll
        for (int i = 0; i < NR - 1; ++i) rd[i] = __builtin_amdgcn_make_buffer_rsrc(dout[i], 0, (unsigned)M * N * 4u, 0x00020000);
#pragma unroll
        for (int t = 0; t < NSHARE; ++t)
#pragma unroll
            for (int i = 0; i < 4; ++i) { COMM_ST(fw[t][i], fv, shoff[t] + i * 65536u); }
        WAIT_V(0); BAR;
        if (wid == 0) {
            if (lane < NR - 1) __hip_atomic_store(fl + ((WS_FWD - WS_FLAGS) / 4 + rank * (NCOMM - 4) + bid), call, __ATOMIC_RELAXED, __HIP_MEMORY_SCOPE_SYSTEM);
            SPIN_UNTIL(__hip_atomic_load((GAS unsigned*)(myws + WS_FWD) + ((rank + 1u + (lane < NR - 1 ? lane : 0)) & (NR - 1)) * NCOMM + bid, __ATOMIC_RELAXED, __HIP_MEMORY_SCOPE_SYSTEM) >= call);
            if (bad && lane == 0) a.out[(shoff[0] + fv) / 4] = __builtin_nanf("");
        }
        return;
    }

    LANE_INIT(); CALL_LOAD();
    const int gw = bid - NCOMM;
    const int wr = wid >> 1, wc = wid & 1, fr = lane & 15, fq = lane >> 4;
    constexpr int NLOC = 2 * NPANEL;
    const int lw = gw - (NWG - NCOMM - NLOC);
    unsigned raw[4][16];
    if (lw >= 0) {
        const unsigned h = lw & 1, p = lw >> 1;
        if (p < 16) {
            const unsigned xv = (unsigned)(wid * 64 + (lane & 15)) * KS + h * 64 + (lane >> 4) * 16;
#pragma unroll
            for (int i = 0; i < 4; ++i)
#pragma unroll
                for (int j = 0; j < 4; ++j) { const u32x4 v = __builtin_amdgcn_raw_buffer_load_b128(rx, xv + j * 16 * KS, p * (256u * KS) + i * 128u, 0);
                    raw[i][4 * j] = v[0]; raw[i][4 * j + 1] = v[1]; raw[i][4 * j + 2] = v[2]; raw[i][4 * j + 3] = v[3]; }
        } else {
#pragma unroll
            for (int i = 0; i < 4; ++i)
#pragma unroll
                for (int j = 0; j < 16; ++j) raw[i][j] = __builtin_amdgcn_raw_buffer_load_b32(rw, wv, (i * 128u + h * 64u + j) * N + (p - 16u) * 256u, 0);
        }
    }
    if (tid == 0) *(LAS unsigned*)(lds + LDS_MISC + 4) = 0u;
    CALL_INIT();
    const __amdgpu_buffer_rsrc_t rin = __builtin_amdgcn_make_buffer_rsrc(myws + doff, 0, DATA_BYTES, 0x00020000);
    const __amdgpu_buffer_rsrc_t rfl = __builtin_amdgcn_make_buffer_rsrc(myws + WS_FLAGS, 0, FLAG_BYTES, 0x00020000);
    GAS unsigned* tmo = (GAS unsigned*)(myws + WS_TMO);
    if (lw >= 0) {
        const unsigned h = lw & 1, p = lw >> 1;
#pragma unroll
        for (int i = 0; i < 4; ++i) {
            const unsigned so = (i * NR + rank) * SLAB + p * BLK + h * 1024;
            if (p < 16) {
#pragma unroll
                for (int j = 0; j < 4; ++j) __builtin_amdgcn_raw_buffer_store_b128((u32x4){raw[i][4 * j], raw[i][4 * j + 1], raw[i][4 * j + 2], raw[i][4 * j + 3]}, rin, lane * 16, so + (wid * 4 + j) * 2048, AUX_SYS);
                ST_PAD();
            } else {
                u32x4 o[4];
#pragma unroll
                for (int b = 0; b < 4; ++b) {
                    const unsigned r0 = raw[i][4 * b], r1 = raw[i][4 * b + 1], r2 = raw[i][4 * b + 2], r3 = raw[i][4 * b + 3];
                    const unsigned t0 = __builtin_amdgcn_perm(r1, r0, 0x05010400), t1 = __builtin_amdgcn_perm(r1, r0, 0x07030602);
                    const unsigned t2 = __builtin_amdgcn_perm(r3, r2, 0x05010400), t3 = __builtin_amdgcn_perm(r3, r2, 0x07030602);
                    o[0][b] = __builtin_amdgcn_perm(t2, t0, 0x05040100); o[1][b] = __builtin_amdgcn_perm(t2, t0, 0x07060302);
                    o[2][b] = __builtin_amdgcn_perm(t3, t1, 0x05040100); o[3][b] = __builtin_amdgcn_perm(t3, t1, 0x07060302);
                }
#pragma unroll
                for (int c = 0; c < 4; ++c) __builtin_amdgcn_raw_buffer_store_b128(o[c], rin, lane * 16, so + wid * 8192 + c * 2048, AUX_SYS);
                ST_PAD();
            }
        }
    }
    asm volatile("s_waitcnt vmcnt(0) lgkmcnt(0)" ::: "memory"); BAR;
    CALL_ADVANCE();
    if (lw >= 0 && tid < 8)
        __hip_atomic_store((GAS unsigned*)(myws + WS_FLAGS) + (lw >> 1) * 128u + ((tid >> 1) * NR + rank) * 4u + (tid & 1) * 2 + (lw & 1), call, __ATOMIC_RELAXED, __HIP_MEMORY_SCOPE_SYSTEM);
    if (lw >= 0 && tid == 0) __hip_atomic_fetch_add((GAS unsigned*)(myws + WS_RETILE), 1u, __ATOMIC_RELAXED, __HIP_MEMORY_SCOPE_AGENT);
    unsigned flip = 0;
    if (gw == 0) {
        unsigned xcc; asm volatile("s_getreg_b32 %0, hwreg(HW_REG_XCC_ID)" : "=s"(xcc));
        if (tid == 0) __hip_atomic_store((GAS unsigned*)(myws + WS_FLIP), (call << 1) | (xcc & 1u), __ATOMIC_RELAXED, __HIP_MEMORY_SCOPE_AGENT);
        flip = (xcc & 1u);
    } else {
        const unsigned long long t0 = __builtin_amdgcn_s_memrealtime();
        for (;;) {
            flip = __builtin_amdgcn_readfirstlane(__hip_atomic_load((GAS unsigned*)(myws + WS_FLIP), __ATOMIC_RELAXED, __HIP_MEMORY_SCOPE_AGENT));
            if ((flip >> 1) == (call & 0x7fffffffu)) break;
            if (__builtin_amdgcn_readfirstlane(__hip_atomic_load(tmo, __ATOMIC_RELAXED, __HIP_MEMORY_SCOPE_AGENT)) == call || __builtin_amdgcn_s_memrealtime() - t0 > DEADLINE_TICKS) {
                __hip_atomic_store(tmo, call, __ATOMIC_RELAXED, __HIP_MEMORY_SCOPE_AGENT); *(volatile LAS unsigned*)(lds + LDS_MISC + 4) = 1u; break; }
            __builtin_amdgcn_s_sleep(8);
        }
        WAIT_L(0); BAR;
        flip = __builtin_amdgcn_readfirstlane(*(volatile LAS unsigned*)(lds + LDS_MISC + 4)) != 0u ? 0u : (flip & 1u);
    }
    const unsigned long long tl = a.tiles[gw ^ __builtin_amdgcn_readfirstlane(flip)];
    const int ntiles = (int)(tl >> 50) & 7;
    const unsigned koff0 = (unsigned)(tl >> 56) & 31u;
    const float scale = __builtin_bit_cast(float, __builtin_amdgcn_readfirstlane(__builtin_bit_cast(int, a.sx[0] * a.sw[0])));
    const unsigned voff = wid * 4096 + lane * 16, ldsw = wid * 4096;
    const unsigned ldsb = (unsigned)(unsigned long)lds;
    unsigned aoff = ldsb + wr * 8192 + lane * 16, boff = ldsb + 4 * HB + wc * 8192 + lane * 16;
    asm volatile("" : "+v"(aoff), "+v"(boff));
    const unsigned ovoff = (unsigned)(wr * 64 + 4 * fq) * 65536u + wc * 256 + fr * 16;
#define TILE(i) ((unsigned)(tl >> (10 * (i))) & 0x3ffu)
#define SA(b, h) (((b) * 2 + (h)) * HB)
#define SB(b, h) ((4 + (b) * 2 + (h)) * HB)
#define G1(bufoff, soff, i) __builtin_amdgcn_raw_ptr_buffer_load_lds(rin, (LAS void*)(lds + (bufoff) + ldsw), 16, voff, (soff), (i) * 1024, AUX_SYS)
#define STAGE4(bufoff, soff) do { G1(bufoff, soff, 0); G1(bufoff, soff, 1); G1(bufoff, soff, 2); G1(bufoff, soff, 3); } while (0)
#define RD(U, i, base, off) asm volatile("ds_read_b128 a[%c1:%c2], %0 offset:%c3" :: "v"(base), "i"((U) + 4 * (i)), "i"((U) + 4 * (i) + 3), "i"((off) + (i) * 1024) : "memory")
#define MF(Z, ai, bj, m, n, RA, RB) mfma<acc_idx(ai, bj, m, n), Z, (RA) + 8 * (m), (RB) + 8 * (n)>()
#define EPG(eq, g) do { if constexpr ((eq) >= 0) epi_gap<(eq) < 0 ? 0 : (eq), (g)>(esc, erout, ovoff, eobase); } while (0)
#define QUAD(Z, ai, bj, RA, RB, RN, rdb, rdoff, stoff, soff, vm, eq, g0, hook) do { u32x4 _fv; unsigned _pi; \
        if constexpr ((vm) >= 0) { asm volatile("s_waitcnt vmcnt(%0) lgkmcnt(0)" :: "n"((vm) < 0 ? 0 : (vm)) : "memory"); BAR; } else WAIT_L(0); SCHED; \
        MF(Z, ai, bj, 0, 0, RA, RB); RD(RN, 0, rdb, rdoff); EPG(eq, g0 + 0); SCHED; \
        MF(Z, ai, bj, 0, 1, RA, RB); RD(RN, 1, rdb, rdoff); EPG(eq, g0 + 1); SCHED; \
        MF(Z, ai, bj, 0, 2, RA, RB); RD(RN, 2, rdb, rdoff); EPG(eq, g0 + 2); if constexpr ((hook) >= 3) PICK_A(_pi); SCHED; \
        MF(Z, ai, bj, 0, 3, RA, RB); RD(RN, 3, rdb, rdoff); EPG(eq, g0 + 3); SCHED; \
        MF(Z, ai, bj, 1, 0, RA, RB); RD(RN, 4, rdb, rdoff); EPG(eq, g0 + 4); if constexpr ((hook) == 3) PICK_B(_pi, kaA, kaB); if constexpr ((hook) == 4) PICK_B(_pi, kbA, kbB); SCHED; \
        MF(Z, ai, bj, 1, 1, RA, RB); RD(RN, 5, rdb, rdoff); EPG(eq, g0 + 5); SCHED; \
        MF(Z, ai, bj, 1, 2, RA, RB); RD(RN, 6, rdb, rdoff); EPG(eq, g0 + 6); SCHED; \
        MF(Z, ai, bj, 1, 3, RA, RB); RD(RN, 7, rdb, rdoff); EPG(eq, g0 + 7); SCHED; \
        MF(Z, ai, bj, 2, 0, RA, RB); G1(stoff, soff, 0); EPG(eq, g0 + 8); SCHED; \
        MF(Z, ai, bj, 2, 1, RA, RB); EPG(eq, g0 + 9); if constexpr ((hook) == 2) _fv = FLAG_LOAD(0); SCHED; \
        MF(Z, ai, bj, 2, 2, RA, RB); G1(stoff, soff, 1); EPG(eq, g0 + 10); SCHED; \
        MF(Z, ai, bj, 2, 3, RA, RB); EPG(eq, g0 + 11); SCHED; \
        MF(Z, ai, bj, 3, 0, RA, RB); G1(stoff, soff, 2); EPG(eq, g0 + 12); SCHED; \
        MF(Z, ai, bj, 3, 1, RA, RB); EPG(eq, g0 + 13); if constexpr ((hook) == 2) FLAG_MASK(_fv); SCHED; \
        MF(Z, ai, bj, 3, 2, RA, RB); G1(stoff, soff, 3); EPG(eq, g0 + 14); SCHED; \
        MF(Z, ai, bj, 3, 3, RA, RB); EPG(eq, g0 + 15); if constexpr ((hook) == 1) FLAG_FETCH(0); SCHED; } while (0)

    unsigned rmask = 0, done = 0, kaA, kaB, kbA, kbB;
#define BASE_A(tile) (((tile) & 15u) * BLK)
#define BASE_B(tile) ((16u + ((tile) >> 4)) * BLK)
#define FLAG_SOFF(tile) ((wid < 2 ? ((tile) & 15u) : 16u + ((tile) >> 4)) * 512u + (wid & 1) * 256u)
    unsigned pA = BASE_A(TILE(0)), pB = BASE_B(TILE(0)), psoff = FLAG_SOFF(TILE(0)), fsoff = psoff;
    const unsigned fvoff = lane * 4;
    int froff = LDS_FLAGS + ((lane & 32) + ((lane + koff0) & 31)) * 16; asm volatile("" : "+v"(froff));
#define FLAG_FETCH_AT(slot, so) __builtin_amdgcn_raw_ptr_buffer_load_lds(rfl, (LAS void*)(lds + LDS_FLAGS + (slot) * 1024 + wid * 256), 4, fvoff, so, 0, AUX_SYS)
#define FLAG_FETCH(slot) FLAG_FETCH_AT(slot, fsoff)
#define FLAG_LOAD(slot) (*(volatile LAS u32x4*)(lds + froff + (slot) * 1024))
#define FLAG_MASK(_v) do { const unsigned _a = _v[0] < _v[1] ? _v[0] : _v[1], _b = _v[2] < _v[3] ? _v[2] : _v[3]; \
            const unsigned long long _m = __builtin_amdgcn_ballot_w64((_a < _b ? _a : _b) >= call); \
            rmask |= (unsigned)_m & (unsigned)(_m >> 32); } while (0)
#define PICK_A(_i) do { \
            if (__builtin_expect((rmask & ~done) == 0u, 0)) {        \
                const unsigned long long _t0 = __builtin_amdgcn_s_memrealtime(); \
                for (;;) { \
                    FLAG_FETCH_AT(1, psoff); \
                    if (wid == 0 && __builtin_amdgcn_s_memrealtime() - _t0 > 10000ull        \
                        && (__builtin_amdgcn_readfirstlane(__hip_atomic_load(tmo, __ATOMIC_RELAXED, __HIP_MEMORY_SCOPE_AGENT)) == call \
                            || __builtin_amdgcn_s_memrealtime() - _t0 > DEADLINE_TICKS)) { \
                        __hip_atomic_store(tmo, call, __ATOMIC_RELAXED, __HIP_MEMORY_SCOPE_AGENT); \
                        *(volatile LAS unsigned*)(lds + LDS_MISC + 4) = 1u; } \
                    asm volatile("s_waitcnt vmcnt(0) lgkmcnt(0)" ::: "memory"); BAR; \
                    const u32x4 _pv = FLAG_LOAD(1); FLAG_MASK(_pv); \
                    if (__builtin_amdgcn_readfirstlane(*(volatile LAS unsigned*)(lds + LDS_MISC + 4)) != 0u) rmask = ~0u; \
                    WAIT_L(0); BAR; \
                    if ((rmask & ~done) != 0u) break; \
                    __builtin_amdgcn_s_sleep(8); \
                } \
            } \
            _i = __builtin_ctz(rmask & ~done); done |= 1u << _i; } while (0)
#define PICK_B(_i, oA, oB) do { const unsigned _q = ((_i + koff0) & 31u) * SLAB; oA = _q + pA; oB = _q + pB; asm volatile("" : "+s"(oA), "+s"(oB)); } while (0)
#define PICK(oA, oB) do { unsigned _pi; PICK_A(_pi); PICK_B(_pi, oA, oB); } while (0)
#define ITER(KIND, sw, S1, S2, S3, S4) do { \
        if (sw) fsoff = nsoff; \
        QUAD(KIND == 0, 0, 0, S1, V_B0A, V_B0B, boff, SA(1, 0), SA(1, 0), kbA, (KIND == 0 ? 38 : KIND == 3 ? 27 : 16), (KIND == 0 ? 3 : -1), 0, 4); \
        QUAD(false, 0, 0, S2, V_B0B, S3, boff, SA(0, 1), SB(0, 0), kaB, -1, (KIND == 0 ? 3 : -1), 16, 1); \
        if constexpr (KIND == 2) { \
            esc = *(volatile LAS unsigned*)(lds + LDS_MISC + 4) != 0u ? __builtin_nanf("") : scale; \
            esc = __builtin_bit_cast(float, __builtin_amdgcn_readfirstlane(__builtin_bit_cast(int, esc))); } \
        QUAD(KIND == 0, 0, 1, S1, S3, S4, boff, SA(1, 1), SB(1, 0), kbB, (KIND == 0 ? 38 : 17), (KIND == 0 ? 3 : KIND == 2 ? 0 : -1), (KIND == 0 ? 32 : 0), 0); \
        QUAD(false, 0, 1, S2, S4, S1, aoff, SA(0, 1), SB(0, 1), kaB + HB, -1, (KIND == 0 ? 2 : KIND == 2 ? 0 : -1), (KIND == 0 ? 0 : 16), 0); \
        QUAD(KIND == 0, 1, 1, S1, S3, S2, aoff, SA(1, 1), SB(1, 1), kbB + HB, (KIND == 0 ? 38 : KIND == 2 ? 27 : 17), (KIND == 0 ? 2 : KIND == 2 ? 0 : -1), (KIND == 0 ? 16 : 32), 0); \
        QUAD(false, 1, 1, S2, S4, S3, aoff, SA(0, 0), SA(0, 1), kaA + HB, -1, (KIND == 0 ? 2 : KIND == 2 ? 1 : -1), (KIND == 0 ? 32 : 0), 0); \
        if (sw) { done = 0; rmask = nmask; pA = nA; pB = nB; psoff = nsoff; } \
        QUAD(KIND == 0, 1, 0, S1, V_B0A, S4, aoff, SA(1, 0), SA(1, 1), kbA + HB, (KIND == 0 ? 38 : KIND == 2 ? 37 : 16), (KIND == 2 ? 1 : -1), 16, 2); \
        QUAD(false, 1, 0, S2, V_B0B, V_B0A, boff, SA(0, 0), SA(0, 0), kaA, -1, (KIND == 2 ? 1 : -1), 32, 3); } while (0)

    PICK(kaA, kaB);
    STAGE4(SA(0, 0), kaA); STAGE4(SB(0, 0), kaB); STAGE4(SB(0, 1), kaB + HB); STAGE4(SA(0, 1), kaA + HB);
    PICK(kbA, kbB);
    STAGE4(SA(1, 0), kbA); STAGE4(SB(1, 0), kbB); STAGE4(SB(1, 1), kbB + HB); STAGE4(SA(1, 1), kbA + HB);
    PICK(kaA, kaB);
    WAIT_V(12); BAR;
    asm volatile("; GEMM_REGS_BEGIN" ::: "memory", RCLOB);
#define RD8(U, base, off) RD(U, 0, base, off); RD(U, 1, base, off); RD(U, 2, base, off); RD(U, 3, base, off); RD(U, 4, base, off); RD(U, 5, base, off); RD(U, 6, base, off); RD(U, 7, base, off)
    RD8(V_R1, aoff, SA(0, 0)); RD8(V_B0A, boff, SA(0, 0)); WAIT_L(0); RD8(V_R2, aoff, SA(1, 0));
    WAIT_L(0); BAR;
    STAGE4(SA(0, 0), kaA);
    WAIT_V(0);
    __amdgpu_buffer_rsrc_t erout = __builtin_amdgcn_make_buffer_rsrc((void*)a.out, 0, 0, 0x00020000);
    unsigned eobase = 0; float esc = 0.f;
    int ui = 0;
    do {
        const bool more = ui + 1 < ntiles; const unsigned nt = TILE(more ? ui + 1 : ui);
        const unsigned nA = BASE_A(nt), nB = BASE_B(nt), nsoff = FLAG_SOFF(nt), nmask = more ? 0u : ~0u;
        ITER(0, false, V_R1, V_R2, V_R3, V_R4);
        ITER(3, false, V_R3, V_R4, V_R1, V_R2);
        if (__builtin_expect(ui == (int)((tl >> 53) & 7u) && ui > 0, 0)) {
            WAIT_V(0); BAR;
            if (tid == 0) {
                __builtin_amdgcn_fence(__ATOMIC_RELEASE, "agent"); WAIT_V(0);
                __hip_atomic_store((GAS unsigned*)(myws + WS_TDONE) + (unsigned)(tl >> 61), call, __ATOMIC_RELAXED, __HIP_MEMORY_SCOPE_AGENT);
            }
        }
        for (int it = 2; it < 14; it += 2) { ITER(1, false, V_R1, V_R2, V_R3, V_R4); ITER(1, false, V_R3, V_R4, V_R1, V_R2); }
        ITER(1, true, V_R1, V_R2, V_R3, V_R4);
        erout = __builtin_amdgcn_make_buffer_rsrc((void*)a.out, 0, (unsigned)M * N * 4u, 0x00020000);
        eobase = (TILE(ui) & 15u) * (256u * 65536u) + (TILE(ui) >> 4) * 1024u;
        ITER(2, false, V_R3, V_R4, V_R1, V_R2);
    } while (++ui < ntiles);
#define EP3(q, g) EPG(q, g); EPG(q, g + 1); EPG(q, g + 2)
#define EP12(q, g) EP3(q, g); EP3(q, g + 3); EP3(q, g + 6); EP3(q, g + 9)
    EP12(3, 0); EP12(3, 12); EP12(3, 24); EP12(3, 36); EP12(2, 0); EP12(2, 12); EP12(2, 24); EP12(2, 36);
    WAIT_V(0);
    asm volatile("; GEMM_REGS_END" ::: "memory");
}

void make_tables(Args& a, int rank) {
    constexpr double P_G1X = -2.0, P_SX = -3.0, P_RX = 1.18, P_SPS = 10.0, P_LEN2 = 38.0, P_S3 = 44.0, P_TAU = 38.0, P_ZOFF = -8.0, P_KS0 = 2.0;
    constexpr int P_KZ = 25, P_RH = 1;
    double key[320]; int n = 0;
    constexpr int PZ = 15 - NZ;
    auto koff = [](int p) { return p < 4 ? 0 : p < PZ ? (int)(P_KS0 + (p - 4) * P_SPS / (PZ - 5) / 1.25 + 0.5) : P_KZ; };
    auto phase = [](int p) { return p < PZ ? p * P_SPS / (PZ - 5) : P_ZOFF; };
    auto add = [&](int panel, double start, double rate, int rot, int off) {
        for (int kt = 0; kt < 4; ++kt) {
            const double k = start + ((kt * NR + ((rank + rot) & (NR - 1)) - off) & 31) * rate + panel * 1e-6;
            int i = n++;
            for (; i > 0 && key[i - 1] > k; --i) { key[i] = key[i - 1]; a.order[i] = a.order[i - 1]; }
            key[i] = k; a.order[i] = (unsigned)((kt << 8) | panel);
        }
    };
    for (int i = 0; i < 8; ++i) { add(i, -1000.0 + P_G1X, 1.0, i, 0); add(16 + i, -1000.0, 1.0, i, 0); add(8 + i, P_SX, P_RX, i, 0); }
    for (int p = 4; p < PZ; ++p) add(16 + 8 + (p - 4), phase(p) - phase(4), P_LEN2 / 32.0, p, koff(p));
    for (int g = 0; g < 3; ++g)
        for (int p = 0; p < 15; ++p) add(16 + 19 + g * 15 + p, P_S3 + g * P_TAU + phase(p), 1.18, p, koff(p));
    for (int i = 0; i < NZ; ++i) add(16 + 19 - NZ + i, P_S3 + P_RH * P_TAU + phase(PZ), 1.18, 1, koff(PZ));
    const int H = 19 + P_RH * 15 + PZ;
    int smp[NSHARE], snp[NSHARE];
    for (int i = 0; i < 2 * NZ; ++i) { smp[i] = (2 * NZ * rank + i) & 15; snp[i] = 19 - NZ + ((2 * NZ * rank) >> 4); }
    smp[2 * NZ] = rank; snp[2 * NZ] = H;
    for (int i = 0; i < NSHARE; ++i) a.shared[i] = (unsigned)smp[i] * (256u * 65536u) + (unsigned)snp[i] * 1024u;
    for (int g = 0; g < NWG - NCOMM; ++g) {
        constexpr int seq[15] = {0, 2, 12, 11, 10, 9, 7, 5, 8, 6, 4, 14, 13, 3, 1};
        const int r = (g & 1) ? 59 - (g >> 3) : (g >> 3), p = seq[r >> 2], m = ((g & 7) >> 1) * 4 + (r & 3);
        int mp[5], np[5], nt = 0; unsigned long long tl = 0;
        if (p < 4) { mp[0] = m & 7; mp[1] = 8 + (m & 7); np[0] = np[1] = 2 * p + (m >> 3); nt = 2; }
        else if (p < PZ) { mp[0] = m; np[0] = 8 + p - 4; nt = 1; }
        for (int k = 0; k < 3; ++k) {
            if (19 + k * 15 + p == H && m < 8) {
                if (m < NSHARE) { mp[nt] = smp[m]; np[nt] = snp[m]; ++nt; tl |= ((unsigned long long)nt << 53) | ((unsigned long long)m << 61); }
            } else { mp[nt] = m; np[nt] = 19 + k * 15 + p; ++nt; }
        }
        tl |= ((unsigned long long)nt << 50) | ((unsigned long long)koff(p) << 56);
        for (int i = 0; i < nt; ++i) tl |= (unsigned long long)((np[i] << 4) | mp[i]) << (10 * i);
        a.tiles[g] = tl;
    }
}
}

extern "C" void kernel_launch(int rank, int nranks, ncclComm_t comm,
                              hipStream_t stream,
                              void* const* d_in, const int* in_sizes, int n_in,
                              void* d_out, int out_size,
                              const int* in_dtypes, int out_dtype,
                              void* const* d_out_all,
                              void* const* d_ws_all, size_t ws_bytes) {
    (void)comm; (void)in_dtypes; (void)out_dtype;
    if (nranks != NR || (unsigned)rank >= (unsigned)NR || n_in != 4 || in_sizes[0] != M * KS || in_sizes[1] != KS * N ||
        out_size != M * N || ws_bytes < WS_NEED) {
        fprintf(stderr, "kernel_launch: unexpected shapes\n");
        return;
    }
    Args a;
    a.x = (const unsigned char*)d_in[0]; a.w = (const unsigned char*)d_in[1];
    a.sx = (const float*)d_in[2]; a.sw = (const float*)d_in[3];
    a.out = (float*)d_out;
    a.myws = (char*)d_ws_all[rank];
    for (int i = 0; i < NR - 1; ++i) { a.peer[i] = (char*)d_ws_all[(rank + 1 + i) % NR]; a.peer_out[i] = (char*)d_out_all[(rank + 1 + i) % NR]; }
    a.rank = rank;
    make_tables(a, rank);
    hipLaunchKernelGGL(fused_gemm, dim3(NWG), dim3(256), 0, stream, a);
    HIP_CHECK(hipGetLastError());
}
```

```cpp
#include <hip/hip_runtime.h>
#include <rccl/rccl.h>
#include <cstdio>

#define HIP_CHECK(expr) do { hipError_t _e = (expr); if (_e != hipSuccess) { \
    fprintf(stderr, "HIP error %s at %s:%d\n", hipGetErrorString(_e), __FILE__, __LINE__); \
    return; } } while (0)

#define LAS __attribute__((address_space(3)))
#define GAS __attribute__((address_space(1)))
typedef unsigned u32x4 __attribute__((ext_vector_type(4)));

namespace {
constexpr int NR = 8;
constexpr int M = 4096, N = 16384, KS = 512;
constexpr unsigned BLK = 32768, HB = 16384;
constexpr unsigned NPANEL = 80;
constexpr unsigned SLAB = NPANEL * BLK;
constexpr unsigned DATA_BYTES = 32 * SLAB;
constexpr unsigned WS_FLAGS = 0;
constexpr unsigned FLAG_BYTES = NPANEL * 512;
constexpr unsigned WS_CTR = 65536;
constexpr unsigned WS_TMO = WS_CTR + 256 * 128;
constexpr unsigned WS_TICKET = WS_TMO + 128;
constexpr unsigned WS_TDONE = WS_TICKET + 128;
constexpr int NZ = 2;
static_assert(NZ == 2, "the senders' destination masks name column panels 17 and 18");
constexpr int NSHARE = 2 * NZ + 1;
constexpr unsigned WS_RETILE = WS_TDONE + 128;
constexpr unsigned WS_FLIP = WS_RETILE + 128;
constexpr unsigned WS_FWD = WS_FLIP + 128;
constexpr unsigned NBLOCK = 320, NUNIT = 4 * NBLOCK;
constexpr size_t WS_DATA = 1u << 20;
constexpr size_t WS_NEED = WS_DATA + 2 * (size_t)DATA_BYTES;
constexpr int NCOMM = 16, NWG = 256;
constexpr unsigned long long DEADLINE_TICKS = 50000000ull;
constexpr int AUX_SYS = 17;
constexpr unsigned RAW_UNITS = 64u, RAW_STEP = 8u;
constexpr unsigned LDS_FLAGS = 8 * HB;
constexpr unsigned LDS_MISC = LDS_FLAGS + 2 * 1024;
constexpr unsigned LDS_BYTES = LDS_MISC + 64;

struct Args {
    const unsigned char* x; const unsigned char* w; const float* sx; const float* sw; float* out;
    char* myws; char* peer[NR - 1];
    char* peer_out[NR - 1];
    int rank;
    unsigned shared[NSHARE];
    unsigned order[NBLOCK];
    unsigned long long tiles[NWG - NCOMM];
};

#define WAIT_V(n) asm volatile("s_waitcnt vmcnt(" #n ")" ::: "memory")
#define WAIT_L(n) asm volatile("s_waitcnt lgkmcnt(" #n ")" ::: "memory")
#define BAR __builtin_amdgcn_s_barrier()
#define SCHED __builtin_amdgcn_sched_barrier(0)
#define ST_PAD() do { SCHED; asm volatile("s_nop 1"); SCHED; } while (0)

#define RCLOB "a0","a1","a2","a3","a4","a5","a6","a7","a8","a9","a10","a11","a12","a13","a14","a15","a16","a17","a18","a19","a20","a21","a22","a23","a24","a25","a26","a27","a28","a29","a30","a31","a32","a33","a34","a35","a36","a37","a38","a39","a40","a41","a42","a43","a44","a45","a46","a47","a48","a49","a50","a51","a52","a53","a54","a55","a56","a57","a58","a59","a60","a61","a62","a63","a64","a65","a66","a67","a68","a69","a70","a71","a72","a73","a74","a75","a76","a77","a78","a79","a80","a81","a82","a83","a84","a85","a86","a87","a88","a89","a90","a91","a92","a93","a94","a95","a96","a97","a98","a99","a100","a101","a102","a103","a104","a105","a106","a107","a108","a109","a110","a111","a112","a113","a114","a115","a116","a117","a118","a119","a120","a121","a122","a123","a124","a125","a126","a127","a128","a129","a130","a131","a132","a133","a134","a135","a136","a137","a138","a139","a140","a141","a142","a143","a144","a145","a146","a147","a148","a149","a150","a151","a152","a153","a154","a155","a156","a157","a158","a159","a160","a161","a162","a163","a164","a165","a166","a167","a168","a169","a170","a171","a172","a173","a174","a175","a176","a177","a178","a179","a180","a181","a182","a183","a184","a185","a186","a187","a188","a189","a190","a191","a192","a193","a194","a195","a196","a197","a198","a199","a200","a201","a202","a203","a204","a205","a206","a207","a208","a209","a210","a211","a212","a213","a214","a215","a216","a217","a218","a219","a220","a221","a222","a223","a224","a225","a226","a227","a228","a229","a230","a231","a232","a233","a234","a235","a236","a237","a238","a239","v40","v41","v42","v43","v44","v45","v46","v47","v48","v49","v50","v51","v52","v53","v54","v55","v56","v57","v58","v59","v60","v61","v62","v63","v64","v65","v66","v67","v68","v69","v70","v71","v72","v73","v74","v75","v76","v77","v78","v79","v80","v81","v82","v83","v84","v85","v86","v87","v88","v89","v90","v91","v92","v93","v94","v95","v96","v97","v98","v99","v100","v101","v102","v103","v104","v105","v106","v107","v108","v109","v110","v111","v112","v113","v114","v115","v116","v117","v118","v119","v120","v121","v122","v123","v124","v125","v126","v127","v128","v129","v130","v131","v132","v133","v134","v135","v136","v137","v138","v139","v140","v141","v142","v143","v144","v145","v146","v147","v148","v149","v150","v151","v152","v153","v154","v155","v156","v157","v158","v159","v160","v161","v162","v163","v164","v165","v166","v167","v168","v169","v170","v171","v172","v173","v174","v175","v176","v177","v178","v179","v180","v181","v182","v183","v184","v185","v186","v187","v188","v189","v190","v191","v192","v193","v194","v195","v196","v197","v198","v199","v200","v201","v202","v203","v204","v205","v206","v207","v208","v209","v210","v211","v212","v213","v214","v215","v216","v217","v218","v219","v220","v221","v222","v223","v224","v225","v226","v227","v228","v229","v230","v231","v232","v233","v234","v235","v236","v237","v238","v239","v240","v241","v242","v243","v244","v245","v246","v247","v248","v249","v250","v251","v252","v253","v254","v255"
constexpr int V_EV = 40, V_R1 = 0, V_R2 = 32, V_R3 = 64, V_R4 = 96, V_B0A = 128, V_B0B = 160;
constexpr int acc_idx(int ai, int bj, int m, int n) { return ((ai * 2 + bj) * 4 + m) * 4 + n; }
constexpr bool acc_in_v(int f) { return f < 52; }
constexpr int acc_reg(int f) { return f < 52 ? 48 + 4 * f : 192 + 4 * (f - 52); }
#define MFMA_ASM(D, C, ...) asm volatile("v_mfma_f32_16x16x128_f8f6f4 " D "[%c0:%c1], a[%c2:%c3], a[%c4:%c5], " C " cbsz:1 blgp:1" \
                                         :: "i"(R), "i"(R + 3), "i"(X), "i"(X + 7), "i"(Y), "i"(Y + 7) __VA_ARGS__)
template <int F, bool ZERO, int X, int Y> __device__ __forceinline__ void mfma() {
    constexpr int R = acc_reg(F);
    if constexpr ((F & 15) == 0) { if constexpr (ZERO) MFMA_ASM("v", "0", : RCLOB); else MFMA_ASM("v", "v[%c0:%c1]", : RCLOB); }
    else if constexpr (acc_in_v(F)) { if constexpr (ZERO) MFMA_ASM("v", "0"); else MFMA_ASM("v", "v[%c0:%c1]"); }
    else { if constexpr (ZERO) MFMA_ASM("a", "0"); else MFMA_ASM("a", "a[%c0:%c1]"); }
}
template <int Q, int G> __device__ __forceinline__ void epi_gap(float sc, __amdgpu_buffer_rsrc_t ro, unsigned ovoff, unsigned obase) {
    constexpr int E = G / 3, part = G % 3, m = E >> 2, j = E & 3, ai = Q >> 1, bj = Q & 1, V = V_EV + (E & 1) * 4, R = acc_reg(acc_idx(ai, bj, m, 0)) + j;
    constexpr bool inv = acc_in_v(acc_idx(ai, bj, m, 0));
    if constexpr (part == 0 && inv)
        asm volatile("v_mul_f32 v%c1, %0, v%c5\n\tv_mul_f32 v%c2, %0, v%c6\n\tv_mul_f32 v%c3, %0, v%c7\n\tv_mul_f32 v%c4, %0, v%c8"
                     :: "s"(sc), "i"(V), "i"(V + 1), "i"(V + 2), "i"(V + 3), "i"(R), "i"(R + 4), "i"(R + 8), "i"(R + 12));
    else if constexpr (part == 0)
        asm volatile("v_accvgpr_read_b32 v%c1, a%c5\n\tv_accvgpr_read_b32 v%c2, a%c6\n\tv_accvgpr_read_b32 v%c3, a%c7\n\tv_accvgpr_read_b32 v%c4, a%c8\n\t"
                     "v_mul_f32 v%c1, %0, v%c1\n\tv_mul_f32 v%c2, %0, v%c2"
                     :: "s"(sc), "i"(V), "i"(V + 1), "i"(V + 2), "i"(V + 3), "i"(R), "i"(R + 4), "i"(R + 8), "i"(R + 12));
    else if constexpr (part == 1) {
        if constexpr (!inv) asm volatile("v_mul_f32 v%c1, %0, v%c1\n\tv_mul_f32 v%c2, %0, v%c2" :: "s"(sc), "i"(V + 2), "i"(V + 3));
        asm volatile("v_maximum3_f32 v%c0, v%c0, 0, 0\n\tv_maximum3_f32 v%c1, v%c1, 0, 0\n\tv_maximum3_f32 v%c2, v%c2, 0, 0\n\tv_maximum3_f32 v%c3, v%c3, 0, 0"
                     :: "i"(V), "i"(V + 1), "i"(V + 2), "i"(V + 3));
    } else
        asm volatile("buffer_store_dwordx4 v[%c3:%c4], %0, %1, %2 offen offset:%c5 nt"
                     :: "v"(ovoff), "s"(ro), "s"(obase + (unsigned)(ai * 128 + m * 16 + j) * 65536u), "i"(V), "i"(V + 3), "i"(bj * 512) : "memory");
}

__global__ __launch_bounds__(256, 1) void fused_gemm(Args a) {
    __shared__ __attribute__((aligned(16))) char shm[LDS_BYTES];
    LAS char* lds = (LAS char*)shm;
    const int wid = __builtin_amdgcn_readfirstlane(threadIdx.x >> 6);
#define LANE_INIT() int tid = threadIdx.x; asm volatile("" : "+v"(tid)); const int lane = tid & 63; \
    const unsigned wv = (unsigned)((lane >> 4) * 16) * N + wid * 64 + 4 * (lane & 15)
    const int bid = blockIdx.x;
    const unsigned rank = (unsigned)a.rank & (NR - 1);
    char* const myws = a.myws;
    GAS unsigned* const ctr = (GAS unsigned*)(myws + WS_CTR + bid * 128);
#define CALL_LOAD() asm volatile("" ::: "memory"); const unsigned ctr_old = __hip_atomic_load(ctr, __ATOMIC_RELAXED, __HIP_MEMORY_SCOPE_AGENT)
#define CALL_INIT() const unsigned call = __builtin_amdgcn_readfirstlane(ctr_old) + 1u; const size_t doff = WS_DATA + (size_t)(call & 1u) * DATA_BYTES
#define CALL_ADVANCE() do { if (tid == 0) __hip_atomic_store(ctr, call, __ATOMIC_RELAXED, __HIP_MEMORY_SCOPE_AGENT); } while (0)
    const __amdgpu_buffer_rsrc_t rx = __builtin_amdgcn_make_buffer_rsrc((void*)a.x, 0, (unsigned)M * KS, 0x00020000);
    const __amdgpu_buffer_rsrc_t rw = __builtin_amdgcn_make_buffer_rsrc((void*)a.w, 0, (unsigned)KS * N, 0x00020000);
    if (bid < NCOMM) {
        LANE_INIT(); CALL_LOAD();
        const unsigned xv = (unsigned)(wid * 64 + (lane & 15)) * KS + (lane >> 4) * 16;
        const unsigned ov = wid * 8192 + lane * 16;
        unsigned rawA[16], rawB[16]; u32x4 xrA[2], xrB[2];
        unsigned u0 = (bid << 16) | a.order[bid >> 2], u1 = ((bid + NCOMM) << 16) | a.order[(bid + NCOMM) >> 2];
        unsigned u2 = ((bid + 2 * NCOMM) << 16) | a.order[(bid + 2 * NCOMM) >> 2], u3 = ((bid + 3 * NCOMM) << 16) | a.order[(bid + 3 * NCOMM) >> 2], pfl = 0;
#define COMM_LOAD(raw, xr, u) do { const unsigned _p = (u) & 255u, _k = ((u) >> 8) & 3u, _h = ((u) >> 16) & 1u, _f = ((u) >> 17) & 1u; \
            const bool _x = _p < 16; \
            const unsigned sox = _x ? (_p * 256u + _f * 32u) * KS + _k * 128u + _h * 64u : 0u, sow = _x ? 0u : (_k * 128u + _h * 64u) * N + (_p - 16u) * 256u; \
            const __amdgpu_buffer_rsrc_t rsx = __builtin_amdgcn_make_buffer_rsrc((void*)a.x, 0, _x ? (unsigned)M * KS : 0u, 0x00020000); \
            const __amdgpu_buffer_rsrc_t rsw = __builtin_amdgcn_make_buffer_rsrc((void*)a.w, 0, _x ? 0u : (unsigned)KS * N, 0x00020000); \
            _Pragma("unroll") for (int i = 0; i < 2; ++i) xr[i] = __builtin_amdgcn_raw_buffer_load_b128(rsx, xv + i * 16 * KS, sox, 0); \
            _Pragma("unroll") for (int i = 0; i < 16; ++i) raw[i] = __builtin_amdgcn_raw_buffer_load_b32(rsw, wv, sow + i * N, 0); } while (0)
        COMM_LOAD(rawA, xrA, u0); COMM_LOAD(rawB, xrB, u1);
        { const unsigned e0 = a.order[tid], e1 = a.order[256 + tid < NBLOCK ? 256 + tid : NBLOCK - 1];
          *(LAS unsigned*)(lds + tid * 4) = e0; *(LAS unsigned*)(lds + 1024 + tid * 4) = e1; }
        CALL_INIT();
        char* dst[NR - 1]; unsigned low4 = 0;
#pragma unroll
        for (int i = 0; i < NR - 1; ++i) { const unsigned pi = (unsigned)(i + 2 * wid + bid) % (NR - 1);
            dst[i] = a.peer[pi] + doff; low4 |= (((rank + 1u + pi) & (NR - 1)) < 4u ? 1u : 0u) << i; }
        GAS unsigned* fl = (GAS unsigned*)(a.peer[lane < NR - 1 ? lane : 0] + WS_FLAGS) + rank * 4;
        GAS unsigned* const tmo = (GAS unsigned*)(myws + WS_TMO);
        char* const tickp = myws + WS_TICKET;
        const unsigned tbase = (call - 1u) * NUNIT - 4u * NCOMM, one = 1u;
        bool bad = false;
#define SPIN_UNTIL(cond) do { const unsigned long long _t0 = __builtin_amdgcn_s_memrealtime(); \
            while (__builtin_amdgcn_ballot_w64(!(cond)) != 0ull) { \
                if (__builtin_amdgcn_readfirstlane(__hip_atomic_load(tmo, __ATOMIC_RELAXED, __HIP_MEMORY_SCOPE_AGENT)) == call \
                    || __builtin_amdgcn_s_memrealtime() - _t0 > DEADLINE_TICKS) { \
                    __hip_atomic_store(tmo, call, __ATOMIC_RELAXED, __HIP_MEMORY_SCOPE_AGENT); bad = true; break; } \
                __builtin_amdgcn_s_sleep(8); } } while (0)
#define COMM_ST(o, vo, so) do { _Pragma("unroll") for (int d = 0; d < NR - 1; ++d) __builtin_amdgcn_raw_buffer_store_b128(o, rd[d], vo, so, AUX_SYS); ST_PAD(); } while (0)
#define UNIT_OFF(u) (((((u) >> 8) & 3u) * NR + rank) * SLAB + ((u) & 255u) * BLK + (((u) >> 16) & 2u) * 2048u + (((u) >> 16) & 1u) * 1024u)
        const __amdgpu_buffer_rsrc_t rl = __builtin_amdgcn_make_buffer_rsrc(myws + doff, 0, DATA_BYTES, 0x00020000);
#define COMM_STEP(raw, xr, first, word, craw, lraw) do { \
            u32x4 o0 = xr[0], o1 = xr[1]; \
            if (craw) { const unsigned sel = (u0 & 0x20000u) ? 0x07030602u : 0x05010400u; \
              _Pragma("unroll") for (int b = 0; b < 4; ++b) { \
                    const unsigned t0 = __builtin_amdgcn_perm(raw[4 * b + 1], raw[4 * b], sel), t2 = __builtin_amdgcn_perm(raw[4 * b + 3], raw[4 * b + 2], sel); \
                    o0[b] |= __builtin_amdgcn_perm(t2, t0, 0x05040100u); o1[b] |= __builtin_amdgcn_perm(t2, t0, 0x07060302u); } } \
            asm volatile("" : "+v"(o0), "+v"(o1)); \
            const unsigned sd = UNIT_OFF(u0); \
              \
            const unsigned _to = (craw) ? 127u : (u0 & 255u) == 16u + 17u ? low4 : (u0 & 255u) == 16u + 18u ? ~low4 : 127u; \
            __amdgpu_buffer_rsrc_t rd[NR - 1]; \
            _Pragma("unroll") for (int d = 0; d < NR - 1; ++d) rd[d] = __builtin_amdgcn_make_buffer_rsrc(dst[d], 0, ((_to >> d) & 1u) ? DATA_BYTES : 0u, 0x00020000); \
            if (lraw) COMM_LOAD(raw, xr, u2); \
            else { xr[0] = __builtin_amdgcn_raw_buffer_load_b128(rl, ov, UNIT_OFF(u2), AUX_SYS); xr[1] = __builtin_amdgcn_raw_buffer_load_b128(rl, ov + 2048, UNIT_OFF(u2), AUX_SYS); } \
            SCHED; COMM_ST(o0, ov, sd); if (lraw) WAIT_V(25); else WAIT_V(9); \
            if (!(first) && tid == 0) { unsigned _t; asm volatile("v_subrev_u32 %0, %1, v255" : "=v"(_t) : "s"(tbase)); \
                *(volatile LAS unsigned*)(lds + LDS_MISC + (word)) = (_t << 16) | *(volatile LAS unsigned*)(lds + ((_t & 2047u) & ~3u)); } \
            WAIT_L(0); BAR; \
            unsigned un = 0; \
            if (!(first)) { un = __builtin_amdgcn_readfirstlane(*(volatile LAS unsigned*)(lds + LDS_MISC + (word))); \
                if (tid < NR - 1) __hip_atomic_store(fl + pfl, call, __ATOMIC_RELAXED, __HIP_MEMORY_SCOPE_SYSTEM); } \
            else CALL_ADVANCE(); \
              \
            if (tid == 0) asm volatile("global_atomic_add v255, %0, %1, off sc0" :: "v"(tickp), "v"(one) : "memory", "v255"); \
            COMM_ST(o1, ov + 2048, sd); \
            pfl = (u0 & 255u) * 128u + ((u0 >> 8) & 3u) * (NR * 4u) + ((u0 >> 16) & 3u); \
            u0 = u1; u1 = u2; if (first) { u2 = u3; } else { u2 = un; } } while (0)
#define COMM_END ((u0 >> 16) >= NUNIT)
        COMM_STEP(rawA, xrA, true, 8, true, true);
        char* dout[NR - 1]; unsigned shoff[NSHARE]; float* outp = a.out;
#pragma unroll
        for (int i = 0; i < NR - 1; ++i) { dout[i] = a.peer_out[(unsigned)(i + 2 * wid + bid) % (NR - 1)]; asm volatile("" : "+s"(dout[i])); }
#pragma unroll
        for (int i = 0; i < NSHARE; ++i) { shoff[i] = a.shared[i]; asm volatile("" : "+s"(shoff[i])); }
        asm volatile("" : "+s"(outp));
        const __amdgpu_buffer_rsrc_t ro = __builtin_amdgcn_make_buffer_rsrc((void*)outp, 0, (unsigned)M * N * 4u, 0x00020000);
        do {
            for (;;) {
                COMM_STEP(rawB, xrB, false, 12, true, true); if (COMM_END) break;
                COMM_STEP(rawA, xrA, false, 8, true, true); if ((u0 >> 16) >= RAW_UNITS + RAW_STEP * bid) break;
            }
            if (COMM_END) break;
            SPIN_UNTIL(__hip_atomic_load((GAS unsigned*)(myws + WS_RETILE), __ATOMIC_RELAXED, __HIP_MEMORY_SCOPE_AGENT) >= call * (2u * NPANEL));
            COMM_STEP(rawB, xrB, false, 12, true, false); if (COMM_END) break;
            COMM_STEP(rawA, xrA, false, 8, true, false); if (COMM_END) break;
            for (;;) {
                COMM_STEP(rawB, xrB, false, 12, false, false); if (COMM_END) break;
                COMM_STEP(rawA, xrA, false, 8, false, false); if (COMM_END) break;
            }
        } while (0);
#define TDONE() (__hip_atomic_load((GAS unsigned*)(myws + WS_TDONE) + (lane < NSHARE ? lane : 0), __ATOMIC_RELAXED, __HIP_MEMORY_SCOPE_AGENT) >= call)
#define PUBLISH() do { BAR; if (tid < NR - 1) __hip_atomic_store(fl + pfl, call, __ATOMIC_RELAXED, __HIP_MEMORY_SCOPE_SYSTEM); } while (0)
        const bool ready = __builtin_amdgcn_ballot_w64(!TDONE()) == 0ull;
        WAIT_V(0);
        if (!ready) { PUBLISH(); SPIN_UNTIL(TDONE()); }
        __builtin_amdgcn_fence(__ATOMIC_ACQUIRE, "agent");
        const unsigned fv = (unsigned)((bid * 4 + wid) * 4) * 65536u + lane * 16;
        u32x4 fw[NSHARE][4];
#pragma unroll
        for (int t = 0; t < NSHARE; ++t)
#pragma unroll
            for (int i = 0; i < 4; ++i) { fw[t][i] = __builtin_amdgcn_raw_buffer_load_b128(ro, fv, shoff[t] + i * 65536u, AUX_SYS); SCHED; }
        if (ready) PUBLISH();
        if (bad) fw[0][0] = (u32x4){0x7fc00000u, 0x7fc00000u, 0x7fc00000u, 0x7fc00000u};
        __amdgpu_buffer_rsrc_t rd[NR - 1];
#pragma unroll
        for (int i = 0; i < NR - 1; ++i) rd[i] = __builtin_amdgcn_make_buffer_rsrc(dout[i], 0, (unsigned)M * N * 4u, 0x00020000);
#pragma unroll
        for (int t = 0; t < NSHARE; ++t)
#pragma unroll
            for (int i = 0; i < 4; ++i) { COMM_ST(fw[t][i], fv, shoff[t] + i * 65536u); }
        WAIT_V(0); BAR;
        if (wid == 0) {
            if (lane < NR - 1) __hip_atomic_store(fl + ((WS_FWD - WS_FLAGS) / 4 + rank * (NCOMM - 4) + bid), call, __ATOMIC_RELAXED, __HIP_MEMORY_SCOPE_SYSTEM);
            SPIN_UNTIL(__hip_atomic_load((GAS unsigned*)(myws + WS_FWD) + ((rank + 1u + (lane < NR - 1 ? lane : 0)) & (NR - 1)) * NCOMM + bid, __ATOMIC_RELAXED, __HIP_MEMORY_SCOPE_SYSTEM) >= call);
            if (bad && lane == 0) a.out[(shoff[0] + fv) / 4] = __builtin_nanf("");
        }
        return;
    }

    LANE_INIT(); CALL_LOAD();
    const int gw = bid - NCOMM;
    const int wr = wid >> 1, wc = wid & 1, fr = lane & 15, fq = lane >> 4;
    constexpr int NLOC = 2 * NPANEL;
    const int lw = gw - (NWG - NCOMM - NLOC);
    unsigned raw[4][16];
    if (lw >= 0) {
        const unsigned h = lw & 1, p = lw >> 1;
        if (p < 16) {
            const unsigned xv = (unsigned)(wid * 64 + (lane & 15)) * KS + h * 64 + (lane >> 4) * 16;
#pragma unroll
            for (int i = 0; i < 4; ++i)
#pragma unroll
                for (int j = 0; j < 4; ++j) { const u32x4 v = __builtin_amdgcn_raw_buffer_load_b128(rx, xv + j * 16 * KS, p * (256u * KS) + i * 128u, 0);
                    raw[i][4 * j] = v[0]; raw[i][4 * j + 1] = v[1]; raw[i][4 * j + 2] = v[2]; raw[i][4 * j + 3] = v[3]; }
        } else {
#pragma unroll
            for (int i = 0; i < 4; ++i)
#pragma unroll
                for (int j = 0; j < 16; ++j) raw[i][j] = __builtin_amdgcn_raw_buffer_load_b32(rw, wv, (i * 128u + h * 64u + j) * N + (p - 16u) * 256u, 0);
        }
    }
    if (tid == 0) *(LAS unsigned*)(lds + LDS_MISC + 4) = 0u;
    CALL_INIT();
    const __amdgpu_buffer_rsrc_t rin = __builtin_amdgcn_make_buffer_rsrc(myws + doff, 0, DATA_BYTES, 0x00020000);
    const __amdgpu_buffer_rsrc_t rfl = __builtin_amdgcn_make_buffer_rsrc(myws + WS_FLAGS, 0, FLAG_BYTES, 0x00020000);
    GAS unsigned* tmo = (GAS unsigned*)(myws + WS_TMO);
    if (lw >= 0) {
        const unsigned h = lw & 1, p = lw >> 1;
#pragma unroll
        for (int i = 0; i < 4; ++i) {
            const unsigned so = (i * NR + rank) * SLAB + p * BLK + h * 1024;
            if (p < 16) {
#pragma unroll
                for (int j = 0; j < 4; ++j) __builtin_amdgcn_raw_buffer_store_b128((u32x4){raw[i][4 * j], raw[i][4 * j + 1], raw[i][4 * j + 2], raw[i][4 * j + 3]}, rin, lane * 16, so + (wid * 4 + j) * 2048, AUX_SYS);
                ST_PAD();
            } else {
                u32x4 o[4];
#pragma unroll
                for (int b = 0; b < 4; ++b) {
                    const unsigned r0 = raw[i][4 * b], r1 = raw[i][4 * b + 1], r2 = raw[i][4 * b + 2], r3 = raw[i][4 * b + 3];
                    const unsigned t0 = __builtin_amdgcn_perm(r1, r0, 0x05010400), t1 = __builtin_amdgcn_perm(r1, r0, 0x07030602);
                    const unsigned t2 = __builtin_amdgcn_perm(r3, r2, 0x05010400), t3 = __builtin_amdgcn_perm(r3, r2, 0x07030602);
                    o[0][b] = __builtin_amdgcn_perm(t2, t0, 0x05040100); o[1][b] = __builtin_amdgcn_perm(t2, t0, 0x07060302);
                    o[2][b] = __builtin_amdgcn_perm(t3, t1, 0x05040100); o[3][b] = __builtin_amdgcn_perm(t3, t1, 0x07060302);
                }
#pragma unroll
                for (int c = 0; c < 4; ++c) __builtin_amdgcn_raw_buffer_store_b128(o[c], rin, lane * 16, so + wid * 8192 + c * 2048, AUX_SYS);
                ST_PAD();
            }
        }
    }
    asm volatile("s_waitcnt vmcnt(0) lgkmcnt(0)" ::: "memory"); BAR;
    CALL_ADVANCE();
    if (lw >= 0 && tid < 8)
        __hip_atomic_store((GAS unsigned*)(myws + WS_FLAGS) + (lw >> 1) * 128u + ((tid >> 1) * NR + rank) * 4u + (tid & 1) * 2 + (lw & 1), call, __ATOMIC_RELAXED, __HIP_MEMORY_SCOPE_SYSTEM);
    if (lw >= 0 && tid == 0) __hip_atomic_fetch_add((GAS unsigned*)(myws + WS_RETILE), 1u, __ATOMIC_RELAXED, __HIP_MEMORY_SCOPE_AGENT);
    unsigned flip = 0;
    if (gw == 0) {
        unsigned xcc; asm volatile("s_getreg_b32 %0, hwreg(HW_REG_XCC_ID)" : "=s"(xcc));
        if (tid == 0) __hip_atomic_store((GAS unsigned*)(myws + WS_FLIP), (call << 1) | (xcc & 1u), __ATOMIC_RELAXED, __HIP_MEMORY_SCOPE_AGENT);
        flip = (xcc & 1u);
    } else {
        const unsigned long long t0 = __builtin_amdgcn_s_memrealtime();
        for (;;) {
            flip = __builtin_amdgcn_readfirstlane(__hip_atomic_load((GAS unsigned*)(myws + WS_FLIP), __ATOMIC_RELAXED, __HIP_MEMORY_SCOPE_AGENT));
            if ((flip >> 1) == (call & 0x7fffffffu)) break;
            if (__builtin_amdgcn_readfirstlane(__hip_atomic_load(tmo, __ATOMIC_RELAXED, __HIP_MEMORY_SCOPE_AGENT)) == call || __builtin_amdgcn_s_memrealtime() - t0 > DEADLINE_TICKS) {
                __hip_atomic_store(tmo, call, __ATOMIC_RELAXED, __HIP_MEMORY_SCOPE_AGENT); *(volatile LAS unsigned*)(lds + LDS_MISC + 4) = 1u; break; }
            __builtin_amdgcn_s_sleep(8);
        }
        WAIT_L(0); BAR;
        flip = __builtin_amdgcn_readfirstlane(*(volatile LAS unsigned*)(lds + LDS_MISC + 4)) != 0u ? 0u : (flip & 1u);
    }
    const unsigned long long tl = a.tiles[gw ^ __builtin_amdgcn_readfirstlane(flip)];
    const int ntiles = (int)(tl >> 50) & 7;
    const unsigned koff0 = (unsigned)(tl >> 56) & 31u;
    const float scale = __builtin_bit_cast(float, __builtin_amdgcn_readfirstlane(__builtin_bit_cast(int, a.sx[0] * a.sw[0])));
    const unsigned voff = wid * 4096 + lane * 16, ldsw = wid * 4096;
    const unsigned ldsb = (unsigned)(unsigned long)lds;
    unsigned aoff = ldsb + wr * 8192 + lane * 16, boff = ldsb + 4 * HB + wc * 8192 + lane * 16;
    asm volatile("" : "+v"(aoff), "+v"(boff));
    const unsigned ovoff = (unsigned)(wr * 64 + 4 * fq) * 65536u + wc * 256 + fr * 16;
#define TILE(i) ((unsigned)(tl >> (10 * (i))) & 0x3ffu)
#define SA(b, h) (((b) * 2 + (h)) * HB)
#define SB(b, h) ((4 + (b) * 2 + (h)) * HB)
#define G1(bufoff, soff, i) __builtin_amdgcn_raw_ptr_buffer_load_lds(rin, (LAS void*)(lds + (bufoff) + ldsw), 16, voff, (soff), (i) * 1024, AUX_SYS)
#define STAGE4(bufoff, soff) do { G1(bufoff, soff, 0); G1(bufoff, soff, 1); G1(bufoff, soff, 2); G1(bufoff, soff, 3); } while (0)
#define RD(U, i, base, off) asm volatile("ds_read_b128 a[%c1:%c2], %0 offset:%c3" :: "v"(base), "i"((U) + 4 * (i)), "i"((U) + 4 * (i) + 3), "i"((off) + (i) * 1024) : "memory")
#define MF(Z, ai, bj, m, n, RA, RB) mfma<acc_idx(ai, bj, m, n), Z, (RA) + 8 * (m), (RB) + 8 * (n)>()
#define EPG(eq, g) do { if constexpr ((eq) >= 0) epi_gap<(eq) < 0 ? 0 : (eq), (g)>(esc, erout, ovoff, eobase); } while (0)
#define QUAD(Z, ai, bj, RA, RB, RN, rdb, rdoff, stoff, soff, vm, eq, g0, hook) do { u32x4 _fv; unsigned _pi; \
        if constexpr ((vm) >= 0) { asm volatile("s_waitcnt vmcnt(%0) lgkmcnt(0)" :: "n"((vm) < 0 ? 0 : (vm)) : "memory"); BAR; } else WAIT_L(0); SCHED; \
        MF(Z, ai, bj, 0, 0, RA, RB); RD(RN, 0, rdb, rdoff); EPG(eq, g0 + 0); SCHED; \
        MF(Z, ai, bj, 0, 1, RA, RB); RD(RN, 1, rdb, rdoff); EPG(eq, g0 + 1); SCHED; \
        MF(Z, ai, bj, 0, 2, RA, RB); RD(RN, 2, rdb, rdoff); EPG(eq, g0 + 2); if constexpr ((hook) >= 3) PICK_A(_pi); SCHED; \
        MF(Z, ai, bj, 0, 3, RA, RB); RD(RN, 3, rdb, rdoff); EPG(eq, g0 + 3); SCHED; \
        MF(Z, ai, bj, 1, 0, RA, RB); RD(RN, 4, rdb, rdoff); EPG(eq, g0 + 4); if constexpr ((hook) == 3) PICK_B(_pi, kaA, kaB); if constexpr ((hook) == 4) PICK_B(_pi, kbA, kbB); SCHED; \
        MF(Z, ai, bj, 1, 1, RA, RB); RD(RN, 5, rdb, rdoff); EPG(eq, g0 + 5); SCHED; \
        MF(Z, ai, bj, 1, 2, RA, RB); RD(RN, 6, rdb, rdoff); EPG(eq, g0 + 6); SCHED; \
        MF(Z, ai, bj, 1, 3, RA, RB); RD(RN, 7, rdb, rdoff); EPG(eq, g0 + 7); SCHED; \
        MF(Z, ai, bj, 2, 0, RA, RB); G1(stoff, soff, 0); EPG(eq, g0 + 8); SCHED; \
        MF(Z, ai, bj, 2, 1, RA, RB); EPG(eq, g0 + 9); if constexpr ((hook) == 2) _fv = FLAG_LOAD(0); SCHED; \
        MF(Z, ai, bj, 2, 2, RA, RB); G1(stoff, soff, 1); EPG(eq, g0 + 10); SCHED; \
        MF(Z, ai, bj, 2, 3, RA, RB); EPG(eq, g0 + 11); SCHED; \
        MF(Z, ai, bj, 3, 0, RA, RB); G1(stoff, soff, 2); EPG(eq, g0 + 12); SCHED; \
        MF(Z, ai, bj, 3, 1, RA, RB); EPG(eq, g0 + 13); if constexpr ((hook) == 2) FLAG_MASK(_fv); SCHED; \
        MF(Z, ai, bj, 3, 2, RA, RB); G1(stoff, soff, 3); EPG(eq, g0 + 14); SCHED; \
        MF(Z, ai, bj, 3, 3, RA, RB); EPG(eq, g0 + 15); if constexpr ((hook) == 1) FLAG_FETCH(0); SCHED; } while (0)

    unsigned rmask = 0, done = 0, kaA, kaB, kbA, kbB;
#define BASE_A(tile) (((tile) & 15u) * BLK)
#define BASE_B(tile) ((16u + ((tile) >> 4)) * BLK)
#define FLAG_SOFF(tile) ((wid < 2 ? ((tile) & 15u) : 16u + ((tile) >> 4)) * 512u + (wid & 1) * 256u)
    unsigned pA = BASE_A(TILE(0)), pB = BASE_B(TILE(0)), psoff = FLAG_SOFF(TILE(0)), fsoff = psoff;
    const unsigned fvoff = lane * 4;
    int froff = LDS_FLAGS + ((lane & 32) + ((lane + koff0) & 31)) * 16; asm volatile("" : "+v"(froff));
#define FLAG_FETCH_AT(slot, so) __builtin_amdgcn_raw_ptr_buffer_load_lds(rfl, (LAS void*)(lds + LDS_FLAGS + (slot) * 1024 + wid * 256), 4, fvoff, so, 0, AUX_SYS)
#define FLAG_FETCH(slot) FLAG_FETCH_AT(slot, fsoff)
#define FLAG_LOAD(slot) (*(volatile LAS u32x4*)(lds + froff + (slot) * 1024))
#define FLAG_MASK(_v) do { const unsigned _a = _v[0] < _v[1] ? _v[0] : _v[1], _b = _v[2] < _v[3] ? _v[2] : _v[3]; \
            const unsigned long long _m = __builtin_amdgcn_ballot_w64((_a < _b ? _a : _b) >= call); \
            rmask |= (unsigned)_m & (unsigned)(_m >> 32); } while (0)
#define PICK_A(_i) do { \
            if (__builtin_expect((rmask & ~done) == 0u, 0)) {        \
                const unsigned long long _t0 = __builtin_amdgcn_s_memrealtime(); \
                for (;;) { \
                    FLAG_FETCH_AT(1, psoff); \
                    if (wid == 0 && __builtin_amdgcn_s_memrealtime() - _t0 > 10000ull        \
                        && (__builtin_amdgcn_readfirstlane(__hip_atomic_load(tmo, __ATOMIC_RELAXED, __HIP_MEMORY_SCOPE_AGENT)) == call \
                            || __builtin_amdgcn_s_memrealtime() - _t0 > DEADLINE_TICKS)) { \
                        __hip_atomic_store(tmo, call, __ATOMIC_RELAXED, __HIP_MEMORY_SCOPE_AGENT); \
                        *(volatile LAS unsigned*)(lds + LDS_MISC + 4) = 1u; } \
                    asm volatile("s_waitcnt vmcnt(0) lgkmcnt(0)" ::: "memory"); BAR; \
                    const u32x4 _pv = FLAG_LOAD(1); FLAG_MASK(_pv); \
                    if (__builtin_amdgcn_readfirstlane(*(volatile LAS unsigned*)(lds + LDS_MISC + 4)) != 0u) rmask = ~0u; \
                    WAIT_L(0); BAR; \
                    if ((rmask & ~done) != 0u) break; \
                    __builtin_amdgcn_s_sleep(8); \
                } \
            } \
            _i = __builtin_ctz(rmask & ~done); done |= 1u << _i; } while (0)
#define PICK_B(_i, oA, oB) do { const unsigned _q = ((_i + koff0) & 31u) * SLAB; oA = _q + pA; oB = _q + pB; asm volatile("" : "+s"(oA), "+s"(oB)); } while (0)
#define PICK(oA, oB) do { unsigned _pi; PICK_A(_pi); PICK_B(_pi, oA, oB); } while (0)
#define ITER(KIND, sw, S1, S2, S3, S4) do { \
        if (sw) fsoff = nsoff; \
        QUAD(KIND == 0, 0, 0, S1, V_B0A, V_B0B, boff, SA(1, 0), SA(1, 0), kbA, (KIND == 0 ? 38 : KIND == 3 ? 27 : 16), (KIND == 0 ? 3 : -1), 0, 4); \
        QUAD(false, 0, 0, S2, V_B0B, S3, boff, SA(0, 1), SB(0, 0), kaB, -1, (KIND == 0 ? 3 : -1), 16, 1); \
        if constexpr (KIND == 2) { \
            esc = *(volatile LAS unsigned*)(lds + LDS_MISC + 4) != 0u ? __builtin_nanf("") : scale; \
            esc = __builtin_bit_cast(float, __builtin_amdgcn_readfirstlane(__builtin_bit_cast(int, esc))); } \
        QUAD(KIND == 0, 0, 1, S1, S3, S4, boff, SA(1, 1), SB(1, 0), kbB, (KIND == 0 ? 38 : 17), (KIND == 0 ? 3 : KIND == 2 ? 0 : -1), (KIND == 0 ? 32 : 0), 0); \
        QUAD(false, 0, 1, S2, S4, S1, aoff, SA(0, 1), SB(0, 1), kaB + HB, -1, (KIND == 0 ? 2 : KIND == 2 ? 0 : -1), (KIND == 0 ? 0 : 16), 0); \
        QUAD(KIND == 0, 1, 1, S1, S3, S2, aoff, SA(1, 1), SB(1, 1), kbB + HB, (KIND == 0 ? 38 : KIND == 2 ? 27 : 17), (KIND == 0 ? 2 : KIND == 2 ? 0 : -1), (KIND == 0 ? 16 : 32), 0); \
        QUAD(false, 1, 1, S2, S4, S3, aoff, SA(0, 0), SA(0, 1), kaA + HB, -1, (KIND == 0 ? 2 : KIND == 2 ? 1 : -1), (KIND == 0 ? 32 : 0), 0); \
        if (sw) { done = 0; rmask = nmask; pA = nA; pB = nB; psoff = nsoff; } \
        QUAD(KIND == 0, 1, 0, S1, V_B0A, S4, aoff, SA(1, 0), SA(1, 1), kbA + HB, (KIND == 0 ? 38 : KIND == 2 ? 37 : 16), (KIND == 2 ? 1 : -1), 16, 2); \
        QUAD(false, 1, 0, S2, V_B0B, V_B0A, boff, SA(0, 0), SA(0, 0), kaA, -1, (KIND == 2 ? 1 : -1), 32, 3); } while (0)

    PICK(kaA, kaB);
    STAGE4(SA(0, 0), kaA); STAGE4(SB(0, 0), kaB); STAGE4(SB(0, 1), kaB + HB); STAGE4(SA(0, 1), kaA + HB);
    PICK(kbA, kbB);
    STAGE4(SA(1, 0), kbA); STAGE4(SB(1, 0), kbB); STAGE4(SB(1, 1), kbB + HB); STAGE4(SA(1, 1), kbA + HB);
    PICK(kaA, kaB);
    WAIT_V(12); BAR;
    asm volatile("; GEMM_REGS_BEGIN" ::: "memory", RCLOB);
#define RD8(U, base, off) RD(U, 0, base, off); RD(U, 1, base, off); RD(U, 2, base, off); RD(U, 3, base, off); RD(U, 4, base, off); RD(U, 5, base, off); RD(U, 6, base, off); RD(U, 7, base, off)
    RD8(V_R1, aoff, SA(0, 0)); RD8(V_B0A, boff, SA(0, 0)); WAIT_L(0); RD8(V_R2, aoff, SA(1, 0));
    WAIT_L(0); BAR;
    STAGE4(SA(0, 0), kaA);
    WAIT_V(0);
    __amdgpu_buffer_rsrc_t erout = __builtin_amdgcn_make_buffer_rsrc((void*)a.out, 0, 0, 0x00020000);
    unsigned eobase = 0; float esc = 0.f;
    int ui = 0;
    do {
        const bool more = ui + 1 < ntiles; const unsigned nt = TILE(more ? ui + 1 : ui);
        const unsigned nA = BASE_A(nt), nB = BASE_B(nt), nsoff = FLAG_SOFF(nt), nmask = more ? 0u : ~0u;
        ITER(0, false, V_R1, V_R2, V_R3, V_R4);
        ITER(3, false, V_R3, V_R4, V_R1, V_R2);
        if (__builtin_expect(ui == (int)((tl >> 53) & 7u) && ui > 0, 0)) {
            WAIT_V(0); BAR;
            if (tid == 0) {
                __builtin_amdgcn_fence(__ATOMIC_RELEASE, "agent"); WAIT_V(0);
                __hip_atomic_store((GAS unsigned*)(myws + WS_TDONE) + (unsigned)(tl >> 61), call, __ATOMIC_RELAXED, __HIP_MEMORY_SCOPE_AGENT);
            }
        }
        for (int it = 2; it < 14; it += 2) { ITER(1, false, V_R1, V_R2, V_R3, V_R4); ITER(1, false, V_R3, V_R4, V_R1, V_R2); }
        ITER(1, true, V_R1, V_R2, V_R3, V_R4);
        erout = __builtin_amdgcn_make_buffer_rsrc((void*)a.out, 0, (unsigned)M * N * 4u, 0x00020000);
        eobase = (TILE(ui) & 15u) * (256u * 65536u) + (TILE(ui) >> 4) * 1024u;
        ITER(2, false, V_R3, V_R4, V_R1, V_R2);
    } while (++ui < ntiles);
#define EP3(q, g) EPG(q, g); EPG(q, g + 1); EPG(q, g + 2)
#define EP12(q, g) EP3(q, g); EP3(q, g + 3); EP3(q, g + 6); EP3(q, g + 9)
    EP12(3, 0); EP12(3, 12); EP12(3, 24); EP12(3, 36); EP12(2, 0); EP12(2, 12); EP12(2, 24); EP12(2, 36);
    WAIT_V(0);
    asm volatile("; GEMM_REGS_END" ::: "memory");
}

void make_tables(Args& a, int rank) {
    constexpr int NE = 2, PH = 2;
    double key[NBLOCK]; int n = 0;
    auto koff = [](int p) { return p < NE ? 0 : p - NE; };
    auto fin = [](int p) { return p < NE ? 34.0 : 37.0 + (p - NE) * 1.5; };
    auto add = [&](int panel, double end, double rate, int rot, int off) {
        for (int kt = 0; kt < 4; ++kt) {
            const double k = end - (31 - ((kt * NR + ((rank + rot) & (NR - 1)) - off) & 31)) * rate + panel * 1e-6;
            int i = n++;
            for (; i > 0 && key[i - 1] > k; --i) { key[i] = key[i - 1]; a.order[i] = a.order[i - 1]; }
            key[i] = k; a.order[i] = (unsigned)((kt << 8) | panel);
        }
    };
    for (int i = 0; i < 8; ++i) { add(i, 1.0, 1.1, i, 0); add(8 + i, 41.0, 0.97, i, 0); }
    for (int i = 0; i < 2 * NE; ++i) add(16 + i, 4.0, 1.1, 2 * i, 0);
    for (int p = NE; p < 15; ++p) add(16 + NE + p, fin(p), 1.13, p, koff(p));
    for (int g = 0; g < 3; ++g)
        for (int p = 0; p < 15; ++p) add(16 + 19 + g * 15 + p, fin(p) + (g + 1) * 38.0 + 7.3, 1.18, p, koff(p));
    for (int i = 0; i < NZ; ++i) add(16 + 17 + i, fin(PH) + 2 * 38.0 + 7.3, 1.18, 1, koff(PH));
    const int H = 19 + 15 + PH;
    int smp[NSHARE], snp[NSHARE];
    for (int i = 0; i < 2 * NZ; ++i) { smp[i] = (2 * NZ * rank + i) & 15; snp[i] = 17 + ((2 * NZ * rank) >> 4); }
    smp[2 * NZ] = rank; snp[2 * NZ] = H;
    for (int i = 0; i < NSHARE; ++i) a.shared[i] = (unsigned)smp[i] * (256u * 65536u) + (unsigned)snp[i] * 1024u;
    for (int g = 0; g < NWG - NCOMM; ++g) {
        constexpr int seq[15] = {0, 1, 14, 13, 12, 11, 9, 7, 10, 8, 6, 5, 4, 3, 2};
        const int r = (g & 1) ? 59 - (g >> 3) : (g >> 3), p = seq[r >> 2], m = ((g & 7) >> 1) * 4 + (r & 3);
        int mp[5], np[5], nt = 0; unsigned long long tl = 0;
        if (p < NE) { mp[0] = m & 7; mp[1] = 8 + (m & 7); np[0] = np[1] = 2 * p + (m >> 3); nt = 2; }
        else { mp[0] = m; np[0] = NE + p; nt = 1; }
        for (int k = 0; k < 3; ++k) {
            if (19 + k * 15 + p == H && m < 8) {
                if (m < NSHARE) { mp[nt] = smp[m]; np[nt] = snp[m]; ++nt; tl |= ((unsigned long long)nt << 53) | ((unsigned long long)m << 61); }
            } else { mp[nt] = m; np[nt] = 19 + k * 15 + p; ++nt; }
        }
        tl |= ((unsigned long long)nt << 50) | ((unsigned long long)koff(p) << 56);
        for (int i = 0; i < nt; ++i) tl |= (unsigned long long)((np[i] << 4) | mp[i]) << (10 * i);
        a.tiles[g] = tl;
    }
}
}

extern "C" void kernel_launch(int rank, int nranks, ncclComm_t comm,
                              hipStream_t stream,
                              void* const* d_in, const int* in_sizes, int n_in,
                              void* d_out, int out_size,
                              const int* in_dtypes, int out_dtype,
                              void* const* d_out_all,
                              void* const* d_ws_all, size_t ws_bytes) {
    (void)comm; (void)in_dtypes; (void)out_dtype;
    if (nranks != NR || (unsigned)rank >= (unsigned)NR || n_in != 4 || in_sizes[0] != M * KS || in_sizes[1] != KS * N ||
        out_size != M * N || ws_bytes < WS_NEED) {
        fprintf(stderr, "kernel_launch: unexpected shapes\n");
        return;
    }
    Args a;
    a.x = (const unsigned char*)d_in[0]; a.w = (const unsigned char*)d_in[1];
    a.sx = (const float*)d_in[2]; a.sw = (const float*)d_in[3];
    a.out = (float*)d_out;
    a.myws = (char*)d_ws_all[rank];
    for (int i = 0; i < NR - 1; ++i) { a.peer[i] = (char*)d_ws_all[(rank + 1 + i) % NR]; a.peer_out[i] = (char*)d_out_all[(rank + 1 + i) % NR]; }
    a.rank = rank;
    make_tables(a, rank);
    hipLaunchKernelGGL(fused_gemm, dim3(NWG), dim3(256), 0, stream, a);
    HIP_CHECK(hipGetLastError());
}
```
